```python
import numpy as np

kernel_name = "gpu_a2a_gemm_m4096_k4096_n16384_w8a8_dqsilu"


ABSMAX_RTOL = 0.001

COMPUTE_DTYPE = "int8"
EPILOGUE = "dequant_silu"
IN_QUANT = "int8"
OUT_QUANT = None
IN_DTYPES = {"x": "int8", "w_mat": "int8",
             "scale_x": "float32", "scale_w": "float32"}
WORLD_SIZE = 8

def setup_inputs():

    x = np.random.randint(-127, 128, size=(4096, 4096)).astype(np.int8)
    w_mat = np.random.randint(-127, 128, size=(4096, 16384)).astype(np.int8)
    scale_x = (0.02 * (1.0 + np.random.rand(1))).astype(np.float32)
    scale_w = (0.02 * (1.0 + np.random.rand(1))).astype(np.float32)
    return {"x": x, "w_mat": w_mat, "scale_x": scale_x, "scale_w": scale_w}

def reference(x, w_mat, scale_x, scale_w):
    acc = x.astype(np.float64) @ w_mat.astype(np.float64)
    y = (acc.astype(np.float32) * (scale_x * scale_w)).astype(np.float32)
    return (y / (1.0 + np.exp(-np.clip(y, -60.0, 60.0)))).astype(np.float32)
```

```cpp
#include <hip/hip_runtime.h>
#include <rccl/rccl.h>
#include <cstdio>

#define PACE_TICKS 80
#define ODD_TICKS 60
#define POLL_START 200
#define POLL_AHEAD 16
#define POLL_SLEEP 4
#define LPRIO 0
#define LROT 1
#define NOPUSH 0
#define L34_SLEEP 12
#define T34_SLEEP 20
#define CL 2
#define STAGGER 0
#define THR_A 8
#define THR_B 5
#define XTOUCH 0
#define SPF 1
#define SPF_FIRST 5
#define WAUX ""
#define LD_SLEEP 0

typedef int v4i __attribute__((ext_vector_type(4)));
#define LAS __attribute__((address_space(3)))

namespace {
constexpr int NR = 8;
constexpr int RING = 9, SLOT = 16896, SYNC_BYTES = 3072, LDS_BYTES = SYNC_BYTES + RING * SLOT;
constexpr unsigned XBUF_BYTES = 2u << 20;
constexpr unsigned FLAG_OFF = 2 * XBUF_BYTES;
constexpr unsigned CTR_OFF = FLAG_OFF + 16384;
constexpr unsigned TMO_OFF = CTR_OFF + 256 * 512;
constexpr unsigned WS_USED = TMO_OFF + 128;
constexpr unsigned STAMP_OFF = 5u << 20;
constexpr unsigned DEADLINE_TICKS = 50000000;
struct Args {
  const signed char* x; const signed char* w; const float* sx; const float* sw; float* out;
  char* myws;
  int rank, pad;
  char* ws[NR];
};
__device__ __forceinline__ unsigned kblock(int u, int rank, int rot) {
  if (u < 8) return rank * 8 + ((u + 2 * rot) & 7);
  int v = u - 8, c = v / 7, i = v - c * 7;
  return (unsigned)((((rank + 1 + i) & 7) << 3) + c);
}
__device__ __forceinline__ bool slow_line(unsigned idx) { return (idx & 7u) == 3u || (idx & 127u) == 63u; }
__device__ __forceinline__ v4i rsrc(unsigned long long a, unsigned bytes) {
  return v4i{(int)(unsigned)a, (int)((a >> 32) & 0xffff), (int)bytes, 0x00020000};
}
}

__global__ __launch_bounds__(512) void a2a_gemm_kernel(Args a) {
  extern __shared__ __attribute__((aligned(16))) char smem[];
#define LREC(i)
  const int b = blockIdx.x;
  const int tm = (b >> 3) & 1, jn = b >> 4, tn = (b & 7) * 16 + jn;
  const int w = __builtin_amdgcn_readfirstlane(threadIdx.x >> 6), l = threadIdx.x & 63;
  const unsigned lds0 = (unsigned)(size_t)(LAS char*)smem;
  if (threadIdx.x < 8) ((volatile LAS unsigned*)smem)[threadIdx.x * 64] = 0u;
  const unsigned dest = (unsigned)(b + (b >> 3)) & 7u;
  unsigned long long px = (unsigned long long)a.x, pw = (unsigned long long)a.w, po = (unsigned long long)a.out,
                     pd = (unsigned long long)a.ws[dest];
  asm volatile("s_waitcnt lgkmcnt(0)\n\ts_barrier" : "+s"(px), "+s"(pw), "+s"(po), "+s"(pd)::"memory");
  const int rank = a.rank & 7;
  const int rot = LROT ? jn & 3 : 0;
  char* myws = a.myws;

  if (w < 4) {
    const int g = l >> 4, q = (l & 15) >> 1, p = l & 1;
    unsigned ad[4];
#pragma unroll
    for (int c = 0; c < 4; c++) ad[c] = lds0 + SYNC_BYTES + g * 2112 + q * 128 + ((c ^ ((q >> 1) & 3)) << 4) + 8 * p;
    unsigned vl = (tm * 4 + w) * 4096 + l * 16;
    unsigned vx = (l >> 3) * 512 + (((l & 7) ^ (l >> 3)) << 4);
    const unsigned stg = lds0 + SYNC_BYTES + 5 * SLOT + w * 8192;
    unsigned vs = stg + (l & 15) * 128 + ((g ^ (l & 7)) << 4);
    unsigned vt = kblock(2 * (l & 31), rank, 0) | (kblock(2 * (l & 31) + 1, rank, 0) << 26);
    v4i rx = rsrc(px, 4096 * 512), rw = rsrc((unsigned long long)myws, XBUF_BYTES), ro = rsrc(po, 512 * 16384 * 4);
    unsigned xo = (rank * 512 + tm * 256 + w * 64) * 512;
    unsigned oo = ((tm * 256 + w * 64) * 16384 + tn * 128) * 4;
    v4i xso = {(int)(xo + rot * 128), (int)(xo + ((rot + 1) & 3) * 128), (int)(xo + ((rot + 2) & 3) * 128), (int)(xo + ((rot + 3) & 3) * 128)};
    v4i oso = {(int)oo, (int)(oo + (1 << 20)), (int)(oo + (2 << 20)), (int)(oo + (3 << 20))};
    asm volatile(
      "s_mov_b32 s50, %[dl]\n\t"
      "s_mov_b32 s74, 0x42ad1f97\n\t"
      "v_mov_b32 v0, 0\n\t"
      "v_mov_b32 v1, 0\n\t"
      "v_mov_b32 v2, 0\n\t"
      "v_mov_b32 v3, 0\n\t"
      "v_mov_b32 v4, 0\n\t"
      "v_mov_b32 v5, 0\n\t"
      "v_mov_b32 v6, 0\n\t"
      "v_mov_b32 v7, 0\n\t"
      "v_mov_b32 v8, 0\n\t"
      "v_mov_b32 v9, 0\n\t"
      "v_mov_b32 v10, 0\n\t"
      "v_mov_b32 v11, 0\n\t"
      "v_mov_b32 v12, 0\n\t"
      "v_mov_b32 v13, 0\n\t"
      "v_mov_b32 v14, 0\n\t"
      "v_mov_b32 v15, 0\n\t"
      "v_mov_b32 v16, 0\n\t"
      "v_mov_b32 v17, 0\n\t"
      "v_mov_b32 v18, 0\n\t"
      "v_mov_b32 v19, 0\n\t"
      "v_mov_b32 v20, 0\n\t"
      "v_mov_b32 v21, 0\n\t"
      "v_mov_b32 v22, 0\n\t"
      "v_mov_b32 v23, 0\n\t"
      "v_mov_b32 v24, 0\n\t"
      "v_mov_b32 v25, 0\n\t"
      "v_mov_b32 v26, 0\n\t"
      "v_mov_b32 v27, 0\n\t"
      "v_mov_b32 v28, 0\n\t"
      "v_mov_b32 v29, 0\n\t"
      "v_mov_b32 v30, 0\n\t"
      "v_mov_b32 v31, 0\n\t"
      "v_mov_b32 v32, 0\n\t"
      "v_mov_b32 v33, 0\n\t"
      "v_mov_b32 v34, 0\n\t"
      "v_mov_b32 v35, 0\n\t"
      "v_mov_b32 v36, 0\n\t"
      "v_mov_b32 v37, 0\n\t"
      "v_mov_b32 v38, 0\n\t"
      "v_mov_b32 v39, 0\n\t"
      "v_mov_b32 v40, 0\n\t"
      "v_mov_b32 v41, 0\n\t"
      "v_mov_b32 v42, 0\n\t"
      "v_mov_b32 v43, 0\n\t"
      "v_mov_b32 v44, 0\n\t"
      "v_mov_b32 v45, 0\n\t"
      "v_mov_b32 v46, 0\n\t"
      "v_mov_b32 v47, 0\n\t"
      "v_mov_b32 v48, 0\n\t"
      "v_mov_b32 v49, 0\n\t"
      "v_mov_b32 v50, 0\n\t"
      "v_mov_b32 v51, 0\n\t"
      "v_mov_b32 v52, 0\n\t"
      "v_mov_b32 v53, 0\n\t"
      "v_mov_b32 v54, 0\n\t"
      "v_mov_b32 v55, 0\n\t"
      "v_mov_b32 v56, 0\n\t"
      "v_mov_b32 v57, 0\n\t"
      "v_mov_b32 v58, 0\n\t"
      "v_mov_b32 v59, 0\n\t"
      "v_mov_b32 v60, 0\n\t"
      "v_mov_b32 v61, 0\n\t"
      "v_mov_b32 v62, 0\n\t"
      "v_mov_b32 v63, 0\n\t"
      "v_mov_b32 v64, 0\n\t"
      "v_mov_b32 v65, 0\n\t"
      "v_mov_b32 v66, 0\n\t"
      "v_mov_b32 v67, 0\n\t"
      "v_mov_b32 v68, 0\n\t"
      "v_mov_b32 v69, 0\n\t"
      "v_mov_b32 v70, 0\n\t"
      "v_mov_b32 v71, 0\n\t"
      "v_mov_b32 v72, 0\n\t"
      "v_mov_b32 v73, 0\n\t"
      "v_mov_b32 v74, 0\n\t"
      "v_mov_b32 v75, 0\n\t"
      "v_mov_b32 v76, 0\n\t"
      "v_mov_b32 v77, 0\n\t"
      "v_mov_b32 v78, 0\n\t"
      "v_mov_b32 v79, 0\n\t"
      "v_mov_b32 v80, 0\n\t"
      "v_mov_b32 v81, 0\n\t"
      "v_mov_b32 v82, 0\n\t"
      "v_mov_b32 v83, 0\n\t"
      "v_mov_b32 v84, 0\n\t"
      "v_mov_b32 v85, 0\n\t"
      "v_mov_b32 v86, 0\n\t"
      "v_mov_b32 v87, 0\n\t"
      "v_mov_b32 v88, 0\n\t"
      "v_mov_b32 v89, 0\n\t"
      "v_mov_b32 v90, 0\n\t"
      "v_mov_b32 v91, 0\n\t"
      "v_mov_b32 v92, 0\n\t"
      "v_mov_b32 v93, 0\n\t"
      "v_mov_b32 v94, 0\n\t"
      "v_mov_b32 v95, 0\n\t"
      "s_setprio 1\n\t"
      "s_add_u32 m0, s75, 0\n\t"
      "s_add_u32 s70, s20, 0\n\t"
      "buffer_load_dwordx4 v254, s[36:39], s70 offen lds\n\t"
      "s_add_u32 m0, s75, 1024\n\t"
      "s_add_u32 s70, s20, 4096\n\t"
      "buffer_load_dwordx4 v254, s[36:39], s70 offen lds\n\t"
      "s_add_u32 m0, s75, 2048\n\t"
      "s_add_u32 s70, s20, 8192\n\t"
      "buffer_load_dwordx4 v254, s[36:39], s70 offen lds\n\t"
      "s_add_u32 m0, s75, 3072\n\t"
      "s_add_u32 s70, s20, 12288\n\t"
      "buffer_load_dwordx4 v254, s[36:39], s70 offen lds\n\t"
      "s_add_u32 m0, s75, 4096\n\t"
      "s_add_u32 s70, s20, 16384\n\t"
      "buffer_load_dwordx4 v254, s[36:39], s70 offen lds\n\t"
      "s_add_u32 m0, s75, 5120\n\t"
      "s_add_u32 s70, s20, 20480\n\t"
      "buffer_load_dwordx4 v254, s[36:39], s70 offen lds\n\t"
      "s_add_u32 m0, s75, 6144\n\t"
      "s_add_u32 s70, s20, 24576\n\t"
      "buffer_load_dwordx4 v254, s[36:39], s70 offen lds\n\t"
      "s_add_u32 m0, s75, 7168\n\t"
      "s_add_u32 s70, s20, 28672\n\t"
      "buffer_load_dwordx4 v254, s[36:39], s70 offen lds\n\t"
      "s_setprio 0\n\t"
      "s_add_u32 m0, s75, 33792\n\t"
      "s_add_u32 s70, s21, 0\n\t"
      "buffer_load_dwordx4 v254, s[36:39], s70 offen lds\n\t"
      "s_add_u32 m0, s75, 34816\n\t"
      "s_add_u32 s70, s21, 4096\n\t"
      "buffer_load_dwordx4 v254, s[36:39], s70 offen lds\n\t"
      "s_add_u32 m0, s75, 35840\n\t"
      "s_add_u32 s70, s21, 8192\n\t"
      "buffer_load_dwordx4 v254, s[36:39], s70 offen lds\n\t"
      "s_add_u32 m0, s75, 36864\n\t"
      "s_add_u32 s70, s21, 12288\n\t"
      "buffer_load_dwordx4 v254, s[36:39], s70 offen lds\n\t"
      "s_add_u32 m0, s75, 37888\n\t"
      "s_add_u32 s70, s21, 16384\n\t"
      "buffer_load_dwordx4 v254, s[36:39], s70 offen lds\n\t"
      "s_add_u32 m0, s75, 38912\n\t"
      "s_add_u32 s70, s21, 20480\n\t"
      "buffer_load_dwordx4 v254, s[36:39], s70 offen lds\n\t"
      "s_add_u32 m0, s75, 39936\n\t"
      "s_add_u32 s70, s21, 24576\n\t"
      "buffer_load_dwordx4 v254, s[36:39], s70 offen lds\n\t"
      "s_add_u32 m0, s75, 40960\n\t"
      "s_add_u32 s70, s21, 28672\n\t"
      "buffer_load_dwordx4 v254, s[36:39], s70 offen lds\n\t"
      "s_mov_b32 m0, s52\n\t"
      "s_mov_b32 s60, -135168\n\t"
      "ds_write_addtid_b32 v255 offset:2048\n\t"
      "s_mov_b32 s61, 16896\n\t"
      "s_mov_b32 s46, 0\n\t"
      "s_mov_b32 s51, 0\n\t"
      "s_mov_b32 s63, 4\n\t"
      "s_mov_b32 s45, 8\n\t"
      "v_mov_b32 v255, v253\n\t"
      "v_mov_b32 v96, 0\n\t"
      "v_mov_b32 v97, 0\n\t"
      "v_mov_b32 v98, 0\n\t"
      "v_mov_b32 v99, 0\n\t"
      "v_mov_b32 v100, 0\n\t"
      "v_mov_b32 v101, 0\n\t"
      "v_mov_b32 v102, 0\n\t"
      "v_mov_b32 v103, 0\n\t"
      "v_mov_b32 v104, 0\n\t"
      "v_mov_b32 v105, 0\n\t"
      "v_mov_b32 v106, 0\n\t"
      "v_mov_b32 v107, 0\n\t"
      "v_mov_b32 v108, 0\n\t"
      "v_mov_b32 v109, 0\n\t"
      "v_mov_b32 v110, 0\n\t"
      "v_mov_b32 v111, 0\n\t"
      "v_mov_b32 v112, 0\n\t"
      "v_mov_b32 v113, 0\n\t"
      "v_mov_b32 v114, 0\n\t"
      "v_mov_b32 v115, 0\n\t"
      "v_mov_b32 v116, 0\n\t"
      "v_mov_b32 v117, 0\n\t"
      "v_mov_b32 v118, 0\n\t"
      "v_mov_b32 v119, 0\n\t"
      "v_mov_b32 v120, 0\n\t"
      "v_mov_b32 v121, 0\n\t"
      "v_mov_b32 v122, 0\n\t"
      "v_mov_b32 v123, 0\n\t"
      "v_mov_b32 v124, 0\n\t"
      "v_mov_b32 v125, 0\n\t"
      "v_mov_b32 v126, 0\n\t"
      "v_mov_b32 v127, 0\n\t"
      "s_waitcnt vmcnt(8)\n\t"
      "ds_read_b128 v[128:131], v255 offset:0\n\t"
      "ds_read_b128 v[132:135], v255 offset:2048\n\t"
      "ds_read_b128 v[136:139], v255 offset:4096\n\t"
      "ds_read_b128 v[140:143], v255 offset:6144\n\t"
      "v_xor_b32 v255, 64, v255\n\t"
      "ds_read_b128 v[144:147], v255 offset:0\n\t"
      "ds_read_b128 v[148:151], v255 offset:2048\n\t"
      "ds_read_b128 v[152:155], v255 offset:4096\n\t"
      "ds_read_b128 v[156:159], v255 offset:6144\n\t"
      "v_xor_b32 v255, 64, v255\n\t"
      "ds_read_addtid_b32 v253 offset:0\n\t"
      "s_waitcnt lgkmcnt(0)\n\t"
      "v_readfirstlane_b32 s53, v253\n\t"
      "s_mov_b32 s48, 0x10001\n\t"
      "s_or_b32 s49, s53, 0x808080\n\t"
      "s_sub_u32 s49, s49, s48\n\t"
      "s_and_b32 s49, s49, 0x808080\n\t"
      "s_cmp_eq_u32 s49, 0x808080\n\t"
      "s_cbranch_scc0 Lslow0_%=\n\t"
      "Lback0_%=:\n\t"
      "ds_read_b64_tr_b8 v[224:225], v248 offset:0\n\t"
      "ds_read_b64_tr_b8 v[226:227], v248 offset:1024\n\t"
      "ds_read_b64_tr_b8 v[228:229], v248 offset:8448\n\t"
      "ds_read_b64_tr_b8 v[230:231], v248 offset:9472\n\t"
      "ds_read_b64_tr_b8 v[232:233], v249 offset:0\n\t"
      "ds_read_b64_tr_b8 v[234:235], v249 offset:1024\n\t"
      "ds_read_b64_tr_b8 v[236:237], v249 offset:8448\n\t"
      "ds_read_b64_tr_b8 v[238:239], v249 offset:9472\n\t"
      "s_waitcnt lgkmcnt(4)\n\t"
      "v_mfma_i32_16x16x64_i8 v[0:3], v[224:227], v[128:131], v[0:3]\n\t"
      "ds_read_b64_tr_b8 v[240:241], v250 offset:0\n\t"
      "v_mfma_i32_16x16x64_i8 v[0:3], v[228:231], v[144:147], v[0:3]\n\t"
      "ds_read_b64_tr_b8 v[242:243], v250 offset:1024\n\t"
      "v_mfma_i32_16x16x64_i8 v[32:35], v[224:227], v[132:135], v[32:35]\n\t"
      "ds_read_b64_tr_b8 v[244:245], v250 offset:8448\n\t"
      "v_mfma_i32_16x16x64_i8 v[32:35], v[228:231], v[148:151], v[32:35]\n\t"
      "ds_read_b64_tr_b8 v[246:247], v250 offset:9472\n\t"
      "v_mfma_i32_16x16x64_i8 v[64:67], v[224:227], v[136:139], v[64:67]\n\t"
      "s_add_u32 s46, s46, 1\n\t"
      "s_cmp_eq_u32 s46, 9\n\t"
      "v_mfma_i32_16x16x64_i8 v[64:67], v[228:231], v[152:155], v[64:67]\n\t"
      "s_cselect_b32 s47, s60, s61\n\t"
      "v_mfma_i32_16x16x64_i8 v[96:99], v[224:227], v[140:143], v[96:99]\n\t"
      "s_cselect_b32 s46, 0, s46\n\t"
      "v_mfma_i32_16x16x64_i8 v[96:99], v[228:231], v[156:159], v[96:99]\n\t"
      "s_waitcnt lgkmcnt(4)\n\t"
      "v_mfma_i32_16x16x64_i8 v[4:7], v[232:235], v[128:131], v[4:7]\n\t"
      "ds_read_b64_tr_b8 v[224:225], v251 offset:0\n\t"
      "v_mfma_i32_16x16x64_i8 v[4:7], v[236:239], v[144:147], v[4:7]\n\t"
      "ds_read_b64_tr_b8 v[226:227], v251 offset:1024\n\t"
      "v_mfma_i32_16x16x64_i8 v[36:39], v[232:235], v[132:135], v[36:39]\n\t"
      "ds_read_b64_tr_b8 v[228:229], v251 offset:8448\n\t"
      "v_mfma_i32_16x16x64_i8 v[36:39], v[236:239], v[148:151], v[36:39]\n\t"
      "ds_read_b64_tr_b8 v[230:231], v251 offset:9472\n\t"
      "v_mfma_i32_16x16x64_i8 v[68:71], v[232:235], v[136:139], v[68:71]\n\t"
      "s_add_u32 m0, s75, 0\n\t"
      "s_add_u32 s70, s22, 0\n\t"
      "buffer_load_dwordx4 v254, s[36:39], s70 offen lds\n\t"
      "v_mfma_i32_16x16x64_i8 v[68:71], v[236:239], v[152:155], v[68:71]\n\t"
      "s_add_u32 m0, s75, 1024\n\t"
      "s_add_u32 s70, s22, 4096\n\t"
      "buffer_load_dwordx4 v254, s[36:39], s70 offen lds\n\t"
      "v_mfma_i32_16x16x64_i8 v[100:103], v[232:235], v[140:143], v[100:103]\n\t"
      "s_add_u32 m0, s75, 2048\n\t"
      "s_add_u32 s70, s22, 8192\n\t"
      "buffer_load_dwordx4 v254, s[36:39], s70 offen lds\n\t"
      "v_mfma_i32_16x16x64_i8 v[100:103], v[236:239], v[156:159], v[100:103]\n\t"
      "s_add_u32 m0, s75, 3072\n\t"
      "s_add_u32 s70, s22, 12288\n\t"
      "buffer_load_dwordx4 v254, s[36:39], s70 offen lds\n\t"
      "s_waitcnt lgkmcnt(4)\n\t"
      "v_mfma_i32_16x16x64_i8 v[8:11], v[240:243], v[128:131], v[8:11]\n\t"
      "ds_read_b64_tr_b8 v[232:233], v248 offset:64\n\t"
      "v_mfma_i32_16x16x64_i8 v[8:11], v[244:247], v[144:147], v[8:11]\n\t"
      "ds_read_b64_tr_b8 v[234:235], v248 offset:1088\n\t"
      "v_mfma_i32_16x16x64_i8 v[40:43], v[240:243], v[132:135], v[40:43]\n\t"
      "ds_read_b64_tr_b8 v[236:237], v248 offset:8512\n\t"
      "v_mfma_i32_16x16x64_i8 v[40:43], v[244:247], v[148:151], v[40:43]\n\t"
      "ds_read_b64_tr_b8 v[238:239], v248 offset:9536\n\t"
      "v_mfma_i32_16x16x64_i8 v[72:75], v[240:243], v[136:139], v[72:75]\n\t"
      "v_add_u32 v248, s47, v248\n\t"
      "s_add_u32 m0, s75, 4096\n\t"
      "s_add_u32 s70, s22, 16384\n\t"
      "buffer_load_dwordx4 v254, s[36:39], s70 offen lds\n\t"
      "v_mfma_i32_16x16x64_i8 v[72:75], v[244:247], v[152:155], v[72:75]\n\t"
      "s_add_u32 m0, s75, 5120\n\t"
      "s_add_u32 s70, s22, 20480\n\t"
      "buffer_load_dwordx4 v254, s[36:39], s70 offen lds\n\t"
      "v_mfma_i32_16x16x64_i8 v[104:107], v[240:243], v[140:143], v[104:107]\n\t"
      "s_add_u32 m0, s75, 6144\n\t"
      "s_add_u32 s70, s22, 24576\n\t"
      "buffer_load_dwordx4 v254, s[36:39], s70 offen lds\n\t"
      "v_mfma_i32_16x16x64_i8 v[104:107], v[244:247], v[156:159], v[104:107]\n\t"
      "s_add_u32 m0, s75, 7168\n\t"
      "s_add_u32 s70, s22, 28672\n\t"
      "buffer_load_dwordx4 v254, s[36:39], s70 offen lds\n\t"
      "s_mov_b32 m0, s52\n\t"
      "s_waitcnt lgkmcnt(4)\n\t"
      "v_mfma_i32_16x16x64_i8 v[12:15], v[224:227], v[128:131], v[12:15]\n\t"
      "ds_read_b64_tr_b8 v[240:241], v249 offset:64\n\t"
      "v_mfma_i32_16x16x64_i8 v[12:15], v[228:231], v[144:147], v[12:15]\n\t"
      "ds_read_b64_tr_b8 v[242:243], v249 offset:1088\n\t"
      "v_mfma_i32_16x16x64_i8 v[44:47], v[224:227], v[132:135], v[44:47]\n\t"
      "ds_read_b64_tr_b8 v[244:245], v249 offset:8512\n\t"
      "v_mfma_i32_16x16x64_i8 v[44:47], v[228:231], v[148:151], v[44:47]\n\t"
      "ds_read_b64_tr_b8 v[246:247], v249 offset:9536\n\t"
      "v_mfma_i32_16x16x64_i8 v[76:79], v[224:227], v[136:139], v[76:79]\n\t"
      "v_add_u32 v249, s47, v249\n\t"
      "ds_read_addtid_b32 v253 offset:0\n\t"
      "v_mfma_i32_16x16x64_i8 v[76:79], v[228:231], v[152:155], v[76:79]\n\t"
      "v_mfma_i32_16x16x64_i8 v[108:111], v[224:227], v[140:143], v[108:111]\n\t"
      "v_mfma_i32_16x16x64_i8 v[108:111], v[228:231], v[156:159], v[108:111]\n\t"
      "s_waitcnt lgkmcnt(4)\n\t"
      "v_mfma_i32_16x16x64_i8 v[16:19], v[232:235], v[128:131], v[16:19]\n\t"
      "ds_read_b64_tr_b8 v[224:225], v250 offset:64\n\t"
      "v_mfma_i32_16x16x64_i8 v[16:19], v[236:239], v[144:147], v[16:19]\n\t"
      "ds_read_b64_tr_b8 v[226:227], v250 offset:1088\n\t"
      "v_mfma_i32_16x16x64_i8 v[48:51], v[232:235], v[132:135], v[48:51]\n\t"
      "ds_read_b64_tr_b8 v[228:229], v250 offset:8512\n\t"
      "v_mfma_i32_16x16x64_i8 v[48:51], v[236:239], v[148:151], v[48:51]\n\t"
      "ds_read_b64_tr_b8 v[230:231], v250 offset:9536\n\t"
      "s_waitcnt vmcnt(8)\n\t"
      "v_mfma_i32_16x16x64_i8 v[80:83], v[232:235], v[136:139], v[80:83]\n\t"
      "v_add_u32 v250, s47, v250\n\t"
      "ds_read_b128 v[160:163], v255 offset:33792\n\t"
      "v_mfma_i32_16x16x64_i8 v[80:83], v[236:239], v[152:155], v[80:83]\n\t"
      "ds_read_b128 v[164:167], v255 offset:35840\n\t"
      "v_mfma_i32_16x16x64_i8 v[112:115], v[232:235], v[140:143], v[112:115]\n\t"
      "ds_read_b128 v[168:171], v255 offset:37888\n\t"
      "v_mfma_i32_16x16x64_i8 v[112:115], v[236:239], v[156:159], v[112:115]\n\t"
      "ds_read_b128 v[172:175], v255 offset:39936\n\t"
      "v_xor_b32 v255, 64, v255\n\t"
      "s_waitcnt lgkmcnt(4)\n\t"
      "v_mfma_i32_16x16x64_i8 v[20:23], v[240:243], v[128:131], v[20:23]\n\t"
      "ds_read_b64_tr_b8 v[232:233], v251 offset:64\n\t"
      "v_readfirstlane_b32 s53, v253\n\t"
      "v_mfma_i32_16x16x64_i8 v[20:23], v[244:247], v[144:147], v[20:23]\n\t"
      "ds_read_b64_tr_b8 v[234:235], v251 offset:1088\n\t"
      "s_add_u32 s48, s48, 0x10001\n\t"
      "v_mfma_i32_16x16x64_i8 v[52:55], v[240:243], v[132:135], v[52:55]\n\t"
      "ds_read_b64_tr_b8 v[236:237], v251 offset:8512\n\t"
      "s_or_b32 s49, s53, 0x808080\n\t"
      "v_mfma_i32_16x16x64_i8 v[52:55], v[244:247], v[148:151], v[52:55]\n\t"
      "ds_read_b64_tr_b8 v[238:239], v251 offset:9536\n\t"
      "s_sub_u32 s49, s49, s48\n\t"
      "v_mfma_i32_16x16x64_i8 v[84:87], v[240:243], v[136:139], v[84:87]\n\t"
      "v_add_u32 v251, s47, v251\n\t"
      "s_and_b32 s49, s49, 0x808080\n\t"
      "v_mfma_i32_16x16x64_i8 v[84:87], v[244:247], v[152:155], v[84:87]\n\t"
      "v_mfma_i32_16x16x64_i8 v[116:119], v[240:243], v[140:143], v[116:119]\n\t"
      "s_cmp_eq_u32 s49, 0x808080\n\t"
      "s_cbranch_scc0 Lslow1_%=\n\t"
      "Lback1_%=:\n\t"
      "v_mfma_i32_16x16x64_i8 v[116:119], v[244:247], v[156:159], v[116:119]\n\t"
      "s_waitcnt lgkmcnt(4)\n\t"
      "v_mfma_i32_16x16x64_i8 v[24:27], v[224:227], v[128:131], v[24:27]\n\t"
      "ds_read_b64_tr_b8 v[240:241], v248 offset:0\n\t"
      "v_mfma_i32_16x16x64_i8 v[24:27], v[228:231], v[144:147], v[24:27]\n\t"
      "ds_read_b64_tr_b8 v[242:243], v248 offset:1024\n\t"
      "v_mfma_i32_16x16x64_i8 v[56:59], v[224:227], v[132:135], v[56:59]\n\t"
      "ds_read_b64_tr_b8 v[244:245], v248 offset:8448\n\t"
      "v_mfma_i32_16x16x64_i8 v[56:59], v[228:231], v[148:151], v[56:59]\n\t"
      "ds_read_b64_tr_b8 v[246:247], v248 offset:9472\n\t"
      "v_mfma_i32_16x16x64_i8 v[88:91], v[224:227], v[136:139], v[88:91]\n\t"
      "ds_read_b128 v[176:179], v255 offset:33792\n\t"
      "v_mfma_i32_16x16x64_i8 v[88:91], v[228:231], v[152:155], v[88:91]\n\t"
      "ds_read_b128 v[180:183], v255 offset:35840\n\t"
      "v_mfma_i32_16x16x64_i8 v[120:123], v[224:227], v[140:143], v[120:123]\n\t"
      "ds_read_b128 v[184:187], v255 offset:37888\n\t"
      "v_mfma_i32_16x16x64_i8 v[120:123], v[228:231], v[156:159], v[120:123]\n\t"
      "ds_read_b128 v[188:191], v255 offset:39936\n\t"
      "v_xor_b32 v255, 64, v255\n\t"
      "s_waitcnt lgkmcnt(4)\n\t"
      "v_mfma_i32_16x16x64_i8 v[28:31], v[232:235], v[128:131], v[28:31]\n\t"
      "ds_read_b64_tr_b8 v[224:225], v249 offset:0\n\t"
      "v_mfma_i32_16x16x64_i8 v[28:31], v[236:239], v[144:147], v[28:31]\n\t"
      "ds_read_b64_tr_b8 v[226:227], v249 offset:1024\n\t"
      "v_mfma_i32_16x16x64_i8 v[60:63], v[232:235], v[132:135], v[60:63]\n\t"
      "ds_read_b64_tr_b8 v[228:229], v249 offset:8448\n\t"
      "v_mfma_i32_16x16x64_i8 v[60:63], v[236:239], v[148:151], v[60:63]\n\t"
      "ds_read_b64_tr_b8 v[230:231], v249 offset:9472\n\t"
      "v_mfma_i32_16x16x64_i8 v[92:95], v[232:235], v[136:139], v[92:95]\n\t"
      "s_add_u32 s51, s51, 1\n\t"
      "v_mfma_i32_16x16x64_i8 v[92:95], v[236:239], v[152:155], v[92:95]\n\t"
      "v_mfma_i32_16x16x64_i8 v[124:127], v[232:235], v[140:143], v[124:127]\n\t"
      "v_mov_b32 v253, s51\n\t"
      "v_mfma_i32_16x16x64_i8 v[124:127], v[236:239], v[156:159], v[124:127]\n\t"
      "ds_write_addtid_b32 v253 offset:1024\n\t"
      "s_waitcnt lgkmcnt(4)\n\t"
      "v_mfma_i32_16x16x64_i8 v[0:3], v[240:243], v[160:163], v[0:3]\n\t"
      "ds_read_b64_tr_b8 v[232:233], v250 offset:0\n\t"
      "v_mfma_i32_16x16x64_i8 v[0:3], v[244:247], v[176:179], v[0:3]\n\t"
      "ds_read_b64_tr_b8 v[234:235], v250 offset:1024\n\t"
      "v_mfma_i32_16x16x64_i8 v[32:35], v[240:243], v[164:167], v[32:35]\n\t"
      "ds_read_b64_tr_b8 v[236:237], v250 offset:8448\n\t"
      "v_mfma_i32_16x16x64_i8 v[32:35], v[244:247], v[180:183], v[32:35]\n\t"
      "ds_read_b64_tr_b8 v[238:239], v250 offset:9472\n\t"
      "v_mfma_i32_16x16x64_i8 v[64:67], v[240:243], v[168:171], v[64:67]\n\t"
      "s_add_u32 s46, s46, 1\n\t"
      "s_cmp_eq_u32 s46, 9\n\t"
      "v_mfma_i32_16x16x64_i8 v[64:67], v[244:247], v[184:187], v[64:67]\n\t"
      "s_cselect_b32 s47, s60, s61\n\t"
      "v_mfma_i32_16x16x64_i8 v[96:99], v[240:243], v[172:175], v[96:99]\n\t"
      "s_cselect_b32 s46, 0, s46\n\t"
      "v_mfma_i32_16x16x64_i8 v[96:99], v[244:247], v[188:191], v[96:99]\n\t"
      "s_waitcnt lgkmcnt(4)\n\t"
      "v_mfma_i32_16x16x64_i8 v[4:7], v[224:227], v[160:163], v[4:7]\n\t"
      "ds_read_b64_tr_b8 v[240:241], v251 offset:0\n\t"
      "v_mfma_i32_16x16x64_i8 v[4:7], v[228:231], v[176:179], v[4:7]\n\t"
      "ds_read_b64_tr_b8 v[242:243], v251 offset:1024\n\t"
      "v_mfma_i32_16x16x64_i8 v[36:39], v[224:227], v[164:167], v[36:39]\n\t"
      "ds_read_b64_tr_b8 v[244:245], v251 offset:8448\n\t"
      "v_mfma_i32_16x16x64_i8 v[36:39], v[228:231], v[180:183], v[36:39]\n\t"
      "ds_read_b64_tr_b8 v[246:247], v251 offset:9472\n\t"
      "v_mfma_i32_16x16x64_i8 v[68:71], v[224:227], v[168:171], v[68:71]\n\t"
      "s_add_u32 m0, s75, 33792\n\t"
      "s_add_u32 s70, s23, 0\n\t"
      "buffer_load_dwordx4 v254, s[36:39], s70 offen lds\n\t"
      "v_mfma_i32_16x16x64_i8 v[68:71], v[228:231], v[184:187], v[68:71]\n\t"
      "s_add_u32 m0, s75, 34816\n\t"
      "s_add_u32 s70, s23, 4096\n\t"
      "buffer_load_dwordx4 v254, s[36:39], s70 offen lds\n\t"
      "v_mfma_i32_16x16x64_i8 v[100:103], v[224:227], v[172:175], v[100:103]\n\t"
      "s_add_u32 m0, s75, 35840\n\t"
      "s_add_u32 s70, s23, 8192\n\t"
      "buffer_load_dwordx4 v254, s[36:39], s70 offen lds\n\t"
      "v_mfma_i32_16x16x64_i8 v[100:103], v[228:231], v[188:191], v[100:103]\n\t"
      "s_add_u32 m0, s75, 36864\n\t"
      "s_add_u32 s70, s23, 12288\n\t"
      "buffer_load_dwordx4 v254, s[36:39], s70 offen lds\n\t"
      "s_waitcnt lgkmcnt(4)\n\t"
      "v_mfma_i32_16x16x64_i8 v[8:11], v[232:235], v[160:163], v[8:11]\n\t"
      "ds_read_b64_tr_b8 v[224:225], v248 offset:64\n\t"
      "v_mfma_i32_16x16x64_i8 v[8:11], v[236:239], v[176:179], v[8:11]\n\t"
      "ds_read_b64_tr_b8 v[226:227], v248 offset:1088\n\t"
      "v_mfma_i32_16x16x64_i8 v[40:43], v[232:235], v[164:167], v[40:43]\n\t"
      "ds_read_b64_tr_b8 v[228:229], v248 offset:8512\n\t"
      "v_mfma_i32_16x16x64_i8 v[40:43], v[236:239], v[180:183], v[40:43]\n\t"
      "ds_read_b64_tr_b8 v[230:231], v248 offset:9536\n\t"
      "v_mfma_i32_16x16x64_i8 v[72:75], v[232:235], v[168:171], v[72:75]\n\t"
      "v_add_u32 v248, s47, v248\n\t"
      "s_add_u32 m0, s75, 37888\n\t"
      "s_add_u32 s70, s23, 16384\n\t"
      "buffer_load_dwordx4 v254, s[36:39], s70 offen lds\n\t"
      "v_mfma_i32_16x16x64_i8 v[72:75], v[236:239], v[184:187], v[72:75]\n\t"
      "s_add_u32 m0, s75, 38912\n\t"
      "s_add_u32 s70, s23, 20480\n\t"
      "buffer_load_dwordx4 v254, s[36:39], s70 offen lds\n\t"
      "v_mfma_i32_16x16x64_i8 v[104:107], v[232:235], v[172:175], v[104:107]\n\t"
      "s_add_u32 m0, s75, 39936\n\t"
      "s_add_u32 s70, s23, 24576\n\t"
      "buffer_load_dwordx4 v254, s[36:39], s70 offen lds\n\t"
      "v_mfma_i32_16x16x64_i8 v[104:107], v[236:239], v[188:191], v[104:107]\n\t"
      "s_add_u32 m0, s75, 40960\n\t"
      "s_add_u32 s70, s23, 28672\n\t"
      "buffer_load_dwordx4 v254, s[36:39], s70 offen lds\n\t"
      "s_mov_b32 m0, s52\n\t"
      "s_waitcnt lgkmcnt(4)\n\t"
      "v_mfma_i32_16x16x64_i8 v[12:15], v[240:243], v[160:163], v[12:15]\n\t"
      "ds_read_b64_tr_b8 v[232:233], v249 offset:64\n\t"
      "v_mfma_i32_16x16x64_i8 v[12:15], v[244:247], v[176:179], v[12:15]\n\t"
      "ds_read_b64_tr_b8 v[234:235], v249 offset:1088\n\t"
      "v_mfma_i32_16x16x64_i8 v[44:47], v[240:243], v[164:167], v[44:47]\n\t"
      "ds_read_b64_tr_b8 v[236:237], v249 offset:8512\n\t"
      "v_mfma_i32_16x16x64_i8 v[44:47], v[244:247], v[180:183], v[44:47]\n\t"
      "ds_read_b64_tr_b8 v[238:239], v249 offset:9536\n\t"
      "v_mfma_i32_16x16x64_i8 v[76:79], v[240:243], v[168:171], v[76:79]\n\t"
      "v_add_u32 v249, s47, v249\n\t"
      "ds_read_addtid_b32 v253 offset:0\n\t"
      "v_mfma_i32_16x16x64_i8 v[76:79], v[244:247], v[184:187], v[76:79]\n\t"
      "v_mfma_i32_16x16x64_i8 v[108:111], v[240:243], v[172:175], v[108:111]\n\t"
      "v_mfma_i32_16x16x64_i8 v[108:111], v[244:247], v[188:191], v[108:111]\n\t"
      "s_waitcnt lgkmcnt(4)\n\t"
      "v_mfma_i32_16x16x64_i8 v[16:19], v[224:227], v[160:163], v[16:19]\n\t"
      "ds_read_b64_tr_b8 v[240:241], v250 offset:64\n\t"
      "v_mfma_i32_16x16x64_i8 v[16:19], v[228:231], v[176:179], v[16:19]\n\t"
      "ds_read_b64_tr_b8 v[242:243], v250 offset:1088\n\t"
      "v_mfma_i32_16x16x64_i8 v[48:51], v[224:227], v[164:167], v[48:51]\n\t"
      "ds_read_b64_tr_b8 v[244:245], v250 offset:8512\n\t"
      "v_mfma_i32_16x16x64_i8 v[48:51], v[228:231], v[180:183], v[48:51]\n\t"
      "ds_read_b64_tr_b8 v[246:247], v250 offset:9536\n\t"
      "s_waitcnt vmcnt(8)\n\t"
      "v_mfma_i32_16x16x64_i8 v[80:83], v[224:227], v[168:171], v[80:83]\n\t"
      "v_add_u32 v250, s47, v250\n\t"
      "ds_read_b128 v[192:195], v255 offset:0\n\t"
      "v_mfma_i32_16x16x64_i8 v[80:83], v[228:231], v[184:187], v[80:83]\n\t"
      "ds_read_b128 v[196:199], v255 offset:2048\n\t"
      "v_mfma_i32_16x16x64_i8 v[112:115], v[224:227], v[172:175], v[112:115]\n\t"
      "ds_read_b128 v[200:203], v255 offset:4096\n\t"
      "v_mfma_i32_16x16x64_i8 v[112:115], v[228:231], v[188:191], v[112:115]\n\t"
      "ds_read_b128 v[204:207], v255 offset:6144\n\t"
      "v_xor_b32 v255, 64, v255\n\t"
      "s_waitcnt lgkmcnt(4)\n\t"
      "v_mfma_i32_16x16x64_i8 v[20:23], v[232:235], v[160:163], v[20:23]\n\t"
      "ds_read_b64_tr_b8 v[224:225], v251 offset:64\n\t"
      "v_readfirstlane_b32 s53, v253\n\t"
      "v_mfma_i32_16x16x64_i8 v[20:23], v[236:239], v[176:179], v[20:23]\n\t"
      "ds_read_b64_tr_b8 v[226:227], v251 offset:1088\n\t"
      "s_add_u32 s48, s48, 0x10001\n\t"
      "v_mfma_i32_16x16x64_i8 v[52:55], v[232:235], v[164:167], v[52:55]\n\t"
      "ds_read_b64_tr_b8 v[228:229], v251 offset:8512\n\t"
      "s_or_b32 s49, s53, 0x808080\n\t"
      "v_mfma_i32_16x16x64_i8 v[52:55], v[236:239], v[180:183], v[52:55]\n\t"
      "ds_read_b64_tr_b8 v[230:231], v251 offset:9536\n\t"
      "s_sub_u32 s49, s49, s48\n\t"
      "v_mfma_i32_16x16x64_i8 v[84:87], v[232:235], v[168:171], v[84:87]\n\t"
      "v_add_u32 v251, s47, v251\n\t"
      "s_and_b32 s49, s49, 0x808080\n\t"
      "v_mfma_i32_16x16x64_i8 v[84:87], v[236:239], v[184:187], v[84:87]\n\t"
      "v_mfma_i32_16x16x64_i8 v[116:119], v[232:235], v[172:175], v[116:119]\n\t"
      "s_cmp_eq_u32 s49, 0x808080\n\t"
      "s_cbranch_scc0 Lslow2_%=\n\t"
      "Lback2_%=:\n\t"
      "v_mfma_i32_16x16x64_i8 v[116:119], v[236:239], v[188:191], v[116:119]\n\t"
      "s_waitcnt lgkmcnt(4)\n\t"
      "v_mfma_i32_16x16x64_i8 v[24:27], v[240:243], v[160:163], v[24:27]\n\t"
      "ds_read_b64_tr_b8 v[232:233], v248 offset:0\n\t"
      "v_mfma_i32_16x16x64_i8 v[24:27], v[244:247], v[176:179], v[24:27]\n\t"
      "ds_read_b64_tr_b8 v[234:235], v248 offset:1024\n\t"
      "v_mfma_i32_16x16x64_i8 v[56:59], v[240:243], v[164:167], v[56:59]\n\t"
      "ds_read_b64_tr_b8 v[236:237], v248 offset:8448\n\t"
      "v_mfma_i32_16x16x64_i8 v[56:59], v[244:247], v[180:183], v[56:59]\n\t"
      "ds_read_b64_tr_b8 v[238:239], v248 offset:9472\n\t"
      "v_mfma_i32_16x16x64_i8 v[88:91], v[240:243], v[168:171], v[88:91]\n\t"
      "ds_read_b128 v[208:211], v255 offset:0\n\t"
      "v_mfma_i32_16x16x64_i8 v[88:91], v[244:247], v[184:187], v[88:91]\n\t"
      "ds_read_b128 v[212:215], v255 offset:2048\n\t"
      "v_mfma_i32_16x16x64_i8 v[120:123], v[240:243], v[172:175], v[120:123]\n\t"
      "ds_read_b128 v[216:219], v255 offset:4096\n\t"
      "v_mfma_i32_16x16x64_i8 v[120:123], v[244:247], v[188:191], v[120:123]\n\t"
      "ds_read_b128 v[220:223], v255 offset:6144\n\t"
      "v_xor_b32 v255, 64, v255\n\t"
      "s_waitcnt lgkmcnt(4)\n\t"
      "v_mfma_i32_16x16x64_i8 v[28:31], v[224:227], v[160:163], v[28:31]\n\t"
      "ds_read_b64_tr_b8 v[240:241], v249 offset:0\n\t"
      "v_mfma_i32_16x16x64_i8 v[28:31], v[228:231], v[176:179], v[28:31]\n\t"
      "ds_read_b64_tr_b8 v[242:243], v249 offset:1024\n\t"
      "v_mfma_i32_16x16x64_i8 v[60:63], v[224:227], v[164:167], v[60:63]\n\t"
      "ds_read_b64_tr_b8 v[244:245], v249 offset:8448\n\t"
      "v_mfma_i32_16x16x64_i8 v[60:63], v[228:231], v[180:183], v[60:63]\n\t"
      "ds_read_b64_tr_b8 v[246:247], v249 offset:9472\n\t"
      "v_mfma_i32_16x16x64_i8 v[92:95], v[224:227], v[168:171], v[92:95]\n\t"
      "s_add_u32 s51, s51, 1\n\t"
      "v_mfma_i32_16x16x64_i8 v[92:95], v[228:231], v[184:187], v[92:95]\n\t"
      "s_waitcnt lgkmcnt(4)\n\t"
      "v_mfma_i32_16x16x64_i8 v[124:127], v[224:227], v[172:175], v[124:127]\n\t"
      "v_mov_b32 v253, s51\n\t"
      "v_mfma_i32_16x16x64_i8 v[124:127], v[228:231], v[188:191], v[124:127]\n\t"
      "ds_write_addtid_b32 v253 offset:1024\n\t"
      "s_waitcnt lgkmcnt(4)\n\t"
      "v_mfma_i32_16x16x64_i8 v[0:3], v[232:235], v[192:195], v[0:3]\n\t"
      "ds_read_b64_tr_b8 v[224:225], v250 offset:0\n\t"
      "v_mfma_i32_16x16x64_i8 v[0:3], v[236:239], v[208:211], v[0:3]\n\t"
      "ds_read_b64_tr_b8 v[226:227], v250 offset:1024\n\t"
      "v_mfma_i32_16x16x64_i8 v[32:35], v[232:235], v[196:199], v[32:35]\n\t"
      "ds_read_b64_tr_b8 v[228:229], v250 offset:8448\n\t"
      "v_mfma_i32_16x16x64_i8 v[32:35], v[236:239], v[212:215], v[32:35]\n\t"
      "ds_read_b64_tr_b8 v[230:231], v250 offset:9472\n\t"
      "s_waitcnt vmcnt(0)\n\t"
      "v_mfma_i32_16x16x64_i8 v[64:67], v[232:235], v[200:203], v[64:67]\n\t"
      "s_add_u32 s46, s46, 1\n\t"
      "s_cmp_eq_u32 s46, 9\n\t"
      "v_mfma_i32_16x16x64_i8 v[64:67], v[236:239], v[216:219], v[64:67]\n\t"
      "s_cselect_b32 s47, s60, s61\n\t"
      "v_mfma_i32_16x16x64_i8 v[96:99], v[232:235], v[204:207], v[96:99]\n\t"
      "s_cselect_b32 s46, 0, s46\n\t"
      "v_mfma_i32_16x16x64_i8 v[96:99], v[236:239], v[220:223], v[96:99]\n\t"
      "s_waitcnt lgkmcnt(4)\n\t"
      "v_mfma_i32_16x16x64_i8 v[4:7], v[240:243], v[192:195], v[4:7]\n\t"
      "ds_read_b64_tr_b8 v[232:233], v251 offset:0\n\t"
      "v_mfma_i32_16x16x64_i8 v[4:7], v[244:247], v[208:211], v[4:7]\n\t"
      "ds_read_b64_tr_b8 v[234:235], v251 offset:1024\n\t"
      "v_mfma_i32_16x16x64_i8 v[36:39], v[240:243], v[196:199], v[36:39]\n\t"
      "ds_read_b64_tr_b8 v[236:237], v251 offset:8448\n\t"
      "v_mfma_i32_16x16x64_i8 v[36:39], v[244:247], v[212:215], v[36:39]\n\t"
      "ds_read_b64_tr_b8 v[238:239], v251 offset:9472\n\t"
      "v_mfma_i32_16x16x64_i8 v[68:71], v[240:243], v[200:203], v[68:71]\n\t"
      "ds_read_b128 v[128:131], v255 offset:33792\n\t"
      "v_mfma_i32_16x16x64_i8 v[68:71], v[244:247], v[216:219], v[68:71]\n\t"
      "ds_read_b128 v[132:135], v255 offset:35840\n\t"
      "v_mfma_i32_16x16x64_i8 v[100:103], v[240:243], v[204:207], v[100:103]\n\t"
      "ds_read_b128 v[136:139], v255 offset:37888\n\t"
      "v_mfma_i32_16x16x64_i8 v[100:103], v[244:247], v[220:223], v[100:103]\n\t"
      "ds_read_b128 v[140:143], v255 offset:39936\n\t"
      "v_xor_b32 v255, 64, v255\n\t"
      "s_waitcnt lgkmcnt(4)\n\t"
      "v_mfma_i32_16x16x64_i8 v[8:11], v[224:227], v[192:195], v[8:11]\n\t"
      "ds_read_b64_tr_b8 v[240:241], v248 offset:64\n\t"
      "v_mfma_i32_16x16x64_i8 v[8:11], v[228:231], v[208:211], v[8:11]\n\t"
      "ds_read_b64_tr_b8 v[242:243], v248 offset:1088\n\t"
      "v_mfma_i32_16x16x64_i8 v[40:43], v[224:227], v[196:199], v[40:43]\n\t"
      "ds_read_b64_tr_b8 v[244:245], v248 offset:8512\n\t"
      "v_mfma_i32_16x16x64_i8 v[40:43], v[228:231], v[212:215], v[40:43]\n\t"
      "ds_read_b64_tr_b8 v[246:247], v248 offset:9536\n\t"
      "v_mfma_i32_16x16x64_i8 v[72:75], v[224:227], v[200:203], v[72:75]\n\t"
      "v_add_u32 v248, s47, v248\n\t"
      "ds_read_b128 v[144:147], v255 offset:33792\n\t"
      "v_mfma_i32_16x16x64_i8 v[72:75], v[228:231], v[216:219], v[72:75]\n\t"
      "ds_read_b128 v[148:151], v255 offset:35840\n\t"
      "v_mfma_i32_16x16x64_i8 v[104:107], v[224:227], v[204:207], v[104:107]\n\t"
      "ds_read_b128 v[152:155], v255 offset:37888\n\t"
      "v_mfma_i32_16x16x64_i8 v[104:107], v[228:231], v[220:223], v[104:107]\n\t"
      "ds_read_b128 v[156:159], v255 offset:39936\n\t"
      "v_xor_b32 v255, 64, v255\n\t"
      "s_waitcnt lgkmcnt(4)\n\t"
      "v_mfma_i32_16x16x64_i8 v[12:15], v[232:235], v[192:195], v[12:15]\n\t"
      "ds_read_b64_tr_b8 v[224:225], v249 offset:64\n\t"
      "v_mfma_i32_16x16x64_i8 v[12:15], v[236:239], v[208:211], v[12:15]\n\t"
      "ds_read_b64_tr_b8 v[226:227], v249 offset:1088\n\t"
      "v_mfma_i32_16x16x64_i8 v[44:47], v[232:235], v[196:199], v[44:47]\n\t"
      "ds_read_b64_tr_b8 v[228:229], v249 offset:8512\n\t"
      "v_mfma_i32_16x16x64_i8 v[44:47], v[236:239], v[212:215], v[44:47]\n\t"
      "ds_read_b64_tr_b8 v[230:231], v249 offset:9536\n\t"
      "v_mfma_i32_16x16x64_i8 v[76:79], v[232:235], v[200:203], v[76:79]\n\t"
      "v_add_u32 v249, s47, v249\n\t"
      "ds_read_addtid_b32 v253 offset:0\n\t"
      "v_mfma_i32_16x16x64_i8 v[76:79], v[236:239], v[216:219], v[76:79]\n\t"
      "v_mfma_i32_16x16x64_i8 v[108:111], v[232:235], v[204:207], v[108:111]\n\t"
      "v_mfma_i32_16x16x64_i8 v[108:111], v[236:239], v[220:223], v[108:111]\n\t"
      "s_waitcnt lgkmcnt(4)\n\t"
      "v_mfma_i32_16x16x64_i8 v[16:19], v[240:243], v[192:195], v[16:19]\n\t"
      "ds_read_b64_tr_b8 v[232:233], v250 offset:64\n\t"
      "v_mfma_i32_16x16x64_i8 v[16:19], v[244:247], v[208:211], v[16:19]\n\t"
      "ds_read_b64_tr_b8 v[234:235], v250 offset:1088\n\t"
      "v_mfma_i32_16x16x64_i8 v[48:51], v[240:243], v[196:199], v[48:51]\n\t"
      "ds_read_b64_tr_b8 v[236:237], v250 offset:8512\n\t"
      "v_mfma_i32_16x16x64_i8 v[48:51], v[244:247], v[212:215], v[48:51]\n\t"
      "ds_read_b64_tr_b8 v[238:239], v250 offset:9536\n\t"
      "v_mfma_i32_16x16x64_i8 v[80:83], v[240:243], v[200:203], v[80:83]\n\t"
      "v_add_u32 v250, s47, v250\n\t"
      "v_mfma_i32_16x16x64_i8 v[80:83], v[244:247], v[216:219], v[80:83]\n\t"
      "ds_read_addtid_b32 v255 offset:2048\n\t"
      "v_mfma_i32_16x16x64_i8 v[112:115], v[240:243], v[204:207], v[112:115]\n\t"
      "v_mfma_i32_16x16x64_i8 v[112:115], v[244:247], v[220:223], v[112:115]\n\t"
      "s_waitcnt lgkmcnt(4)\n\t"
      "v_mfma_i32_16x16x64_i8 v[20:23], v[224:227], v[192:195], v[20:23]\n\t"
      "ds_read_b64_tr_b8 v[240:241], v251 offset:64\n\t"
      "v_readfirstlane_b32 s53, v253\n\t"
      "v_mfma_i32_16x16x64_i8 v[20:23], v[228:231], v[208:211], v[20:23]\n\t"
      "ds_read_b64_tr_b8 v[242:243], v251 offset:1088\n\t"
      "s_add_u32 s48, s48, 0x10001\n\t"
      "v_mfma_i32_16x16x64_i8 v[52:55], v[224:227], v[196:199], v[52:55]\n\t"
      "ds_read_b64_tr_b8 v[244:245], v251 offset:8512\n\t"
      "s_or_b32 s49, s53, 0x808080\n\t"
      "v_mfma_i32_16x16x64_i8 v[52:55], v[228:231], v[212:215], v[52:55]\n\t"
      "ds_read_b64_tr_b8 v[246:247], v251 offset:9536\n\t"
      "s_sub_u32 s49, s49, s48\n\t"
      "v_mfma_i32_16x16x64_i8 v[84:87], v[224:227], v[200:203], v[84:87]\n\t"
      "v_add_u32 v251, s47, v251\n\t"
      "s_and_b32 s49, s49, 0x808080\n\t"
      "v_mfma_i32_16x16x64_i8 v[84:87], v[228:231], v[216:219], v[84:87]\n\t"
      "v_mfma_i32_16x16x64_i8 v[116:119], v[224:227], v[204:207], v[116:119]\n\t"
      "s_cmp_eq_u32 s49, 0x808080\n\t"
      "s_cbranch_scc0 Lslow3_%=\n\t"
      "Lback3_%=:\n\t"
      "v_mfma_i32_16x16x64_i8 v[116:119], v[228:231], v[220:223], v[116:119]\n\t"
      "s_waitcnt lgkmcnt(4)\n\t"
      "v_mfma_i32_16x16x64_i8 v[24:27], v[232:235], v[192:195], v[24:27]\n\t"
      "ds_read_b64_tr_b8 v[224:225], v248 offset:0\n\t"
      "v_mfma_i32_16x16x64_i8 v[24:27], v[236:239], v[208:211], v[24:27]\n\t"
      "ds_read_b64_tr_b8 v[226:227], v248 offset:1024\n\t"
      "v_mfma_i32_16x16x64_i8 v[56:59], v[232:235], v[196:199], v[56:59]\n\t"
      "ds_read_b64_tr_b8 v[228:229], v248 offset:8448\n\t"
      "v_mfma_i32_16x16x64_i8 v[56:59], v[236:239], v[212:215], v[56:59]\n\t"
      "ds_read_b64_tr_b8 v[230:231], v248 offset:9472\n\t"
      "v_mfma_i32_16x16x64_i8 v[88:91], v[232:235], v[200:203], v[88:91]\n\t"
      "ds_read_addtid_b32 v254 offset:0\n\t"
      "v_mfma_i32_16x16x64_i8 v[88:91], v[236:239], v[216:219], v[88:91]\n\t"
      "v_mfma_i32_16x16x64_i8 v[120:123], v[232:235], v[204:207], v[120:123]\n\t"
      "v_mfma_i32_16x16x64_i8 v[120:123], v[236:239], v[220:223], v[120:123]\n\t"
      "s_waitcnt lgkmcnt(4)\n\t"
      "v_mfma_i32_16x16x64_i8 v[28:31], v[240:243], v[192:195], v[28:31]\n\t"
      "ds_read_b64_tr_b8 v[232:233], v249 offset:0\n\t"
      "v_mfma_i32_16x16x64_i8 v[28:31], v[244:247], v[208:211], v[28:31]\n\t"
      "ds_read_b64_tr_b8 v[234:235], v249 offset:1024\n\t"
      "v_mfma_i32_16x16x64_i8 v[60:63], v[240:243], v[196:199], v[60:63]\n\t"
      "ds_read_b64_tr_b8 v[236:237], v249 offset:8448\n\t"
      "v_mfma_i32_16x16x64_i8 v[60:63], v[244:247], v[212:215], v[60:63]\n\t"
      "ds_read_b64_tr_b8 v[238:239], v249 offset:9472\n\t"
      "v_mfma_i32_16x16x64_i8 v[92:95], v[240:243], v[200:203], v[92:95]\n\t"
      "s_add_u32 s51, s51, 1\n\t"
      "v_mfma_i32_16x16x64_i8 v[92:95], v[244:247], v[216:219], v[92:95]\n\t"
      "v_mfma_i32_16x16x64_i8 v[124:127], v[240:243], v[204:207], v[124:127]\n\t"
      "v_mov_b32 v253, s51\n\t"
      "v_mfma_i32_16x16x64_i8 v[124:127], v[244:247], v[220:223], v[124:127]\n\t"
      "ds_write_addtid_b32 v253 offset:1024\n\t"
      "s_waitcnt lgkmcnt(4)\n\t"
      "v_mfma_i32_16x16x64_i8 v[0:3], v[224:227], v[128:131], v[0:3]\n\t"
      "ds_read_b64_tr_b8 v[240:241], v250 offset:0\n\t"
      "v_readfirstlane_b32 s53, v254\n\t"
      "v_mfma_i32_16x16x64_i8 v[0:3], v[228:231], v[144:147], v[0:3]\n\t"
      "ds_read_b64_tr_b8 v[242:243], v250 offset:1024\n\t"
      "s_add_u32 s48, s48, 0x600\n\t"
      "v_mfma_i32_16x16x64_i8 v[32:35], v[224:227], v[132:135], v[32:35]\n\t"
      "ds_read_b64_tr_b8 v[244:245], v250 offset:8448\n\t"
      "s_or_b32 s49, s53, 0x808080\n\t"
      "v_mfma_i32_16x16x64_i8 v[32:35], v[228:231], v[148:151], v[32:35]\n\t"
      "ds_read_b64_tr_b8 v[246:247], v250 offset:9472\n\t"
      "s_sub_u32 s49, s49, s48\n\t"
      "v_mfma_i32_16x16x64_i8 v[64:67], v[224:227], v[136:139], v[64:67]\n\t"
      "s_add_u32 s46, s46, 1\n\t"
      "s_cmp_eq_u32 s46, 9\n\t"
      "v_mfma_i32_16x16x64_i8 v[64:67], v[228:231], v[152:155], v[64:67]\n\t"
      "s_cselect_b32 s47, s60, s61\n\t"
      "v_mfma_i32_16x16x64_i8 v[96:99], v[224:227], v[140:143], v[96:99]\n\t"
      "s_cselect_b32 s46, 0, s46\n\t"
      "v_mfma_i32_16x16x64_i8 v[96:99], v[228:231], v[156:159], v[96:99]\n\t"
      "s_and_b32 s49, s49, 0x808080\n\t"
      "s_cmp_eq_u32 s49, 0x808080\n\t"
      "s_cbranch_scc0 Lslow4_%=\n\t"
      "Lback4_%=:\n\t"
      "s_waitcnt lgkmcnt(4)\n\t"
      "v_mfma_i32_16x16x64_i8 v[4:7], v[232:235], v[128:131], v[4:7]\n\t"
      "ds_read_b64_tr_b8 v[224:225], v251 offset:0\n\t"
      "s_bfe_u32 s49, s53, 0x10018\n\t"
      "v_mfma_i32_16x16x64_i8 v[4:7], v[236:239], v[144:147], v[4:7]\n\t"
      "ds_read_b64_tr_b8 v[226:227], v251 offset:1024\n\t"
      "s_lshl_b32 s49, s49, 21\n\t"
      "v_mfma_i32_16x16x64_i8 v[36:39], v[232:235], v[132:135], v[36:39]\n\t"
      "ds_read_b64_tr_b8 v[228:229], v251 offset:8448\n\t"
      "s_add_u32 s40, s40, s49\n\t"
      "v_mfma_i32_16x16x64_i8 v[36:39], v[236:239], v[148:151], v[36:39]\n\t"
      "ds_read_b64_tr_b8 v[230:231], v251 offset:9472\n\t"
      "s_addc_u32 s41, s41, 0\n\t"
      "v_mfma_i32_16x16x64_i8 v[68:71], v[232:235], v[136:139], v[68:71]\n\t"
      "v_readlane_b32 s49, v255, s63\n\t"
      "s_add_u32 s63, s63, 1\n\t"
      "v_mfma_i32_16x16x64_i8 v[68:71], v[236:239], v[152:155], v[68:71]\n\t"
      "s_lshl_b32 s44, s49, 15\n\t"
      "s_lshr_b32 s62, s49, 11\n\t"
      "v_mfma_i32_16x16x64_i8 v[100:103], v[232:235], v[140:143], v[100:103]\n\t"
      "buffer_load_dwordx4 v[160:163], v252, s[40:43], s44 offen offset:0 sc0 sc1\n\t"
      "v_mfma_i32_16x16x64_i8 v[100:103], v[236:239], v[156:159], v[100:103]\n\t"
      "buffer_load_dwordx4 v[176:179], v252, s[40:43], s62 offen offset:0 sc0 sc1\n\t"
      "s_waitcnt lgkmcnt(4)\n\t"
      "v_mfma_i32_16x16x64_i8 v[8:11], v[240:243], v[128:131], v[8:11]\n\t"
      "ds_read_b64_tr_b8 v[232:233], v248 offset:64\n\t"
      "v_mfma_i32_16x16x64_i8 v[8:11], v[244:247], v[144:147], v[8:11]\n\t"
      "ds_read_b64_tr_b8 v[234:235], v248 offset:1088\n\t"
      "v_mfma_i32_16x16x64_i8 v[40:43], v[240:243], v[132:135], v[40:43]\n\t"
      "ds_read_b64_tr_b8 v[236:237], v248 offset:8512\n\t"
      "v_mfma_i32_16x16x64_i8 v[40:43], v[244:247], v[148:151], v[40:43]\n\t"
      "ds_read_b64_tr_b8 v[238:239], v248 offset:9536\n\t"
      "v_mfma_i32_16x16x64_i8 v[72:75], v[240:243], v[136:139], v[72:75]\n\t"
      "v_add_u32 v248, s47, v248\n\t"
      "v_mfma_i32_16x16x64_i8 v[72:75], v[244:247], v[152:155], v[72:75]\n\t"
      "buffer_load_dwordx4 v[164:167], v252, s[40:43], s44 offen offset:1024 sc0 sc1\n\t"
      "v_mfma_i32_16x16x64_i8 v[104:107], v[240:243], v[140:143], v[104:107]\n\t"
      "buffer_load_dwordx4 v[168:171], v252, s[40:43], s44 offen offset:2048 sc0 sc1\n\t"
      "v_mfma_i32_16x16x64_i8 v[104:107], v[244:247], v[156:159], v[104:107]\n\t"
      "buffer_load_dwordx4 v[180:183], v252, s[40:43], s62 offen offset:1024 sc0 sc1\n\t"
      "s_waitcnt lgkmcnt(4)\n\t"
      "v_mfma_i32_16x16x64_i8 v[12:15], v[224:227], v[128:131], v[12:15]\n\t"
      "ds_read_b64_tr_b8 v[240:241], v249 offset:64\n\t"
      "v_mfma_i32_16x16x64_i8 v[12:15], v[228:231], v[144:147], v[12:15]\n\t"
      "ds_read_b64_tr_b8 v[242:243], v249 offset:1088\n\t"
      "v_mfma_i32_16x16x64_i8 v[44:47], v[224:227], v[132:135], v[44:47]\n\t"
      "ds_read_b64_tr_b8 v[244:245], v249 offset:8512\n\t"
      "v_mfma_i32_16x16x64_i8 v[44:47], v[228:231], v[148:151], v[44:47]\n\t"
      "ds_read_b64_tr_b8 v[246:247], v249 offset:9536\n\t"
      "v_mfma_i32_16x16x64_i8 v[76:79], v[224:227], v[136:139], v[76:79]\n\t"
      "v_add_u32 v249, s47, v249\n\t"
      "ds_read_addtid_b32 v253 offset:0\n\t"
      "v_mfma_i32_16x16x64_i8 v[76:79], v[228:231], v[152:155], v[76:79]\n\t"
      "buffer_load_dwordx4 v[184:187], v252, s[40:43], s62 offen offset:2048 sc0 sc1\n\t"
      "v_mfma_i32_16x16x64_i8 v[108:111], v[224:227], v[140:143], v[108:111]\n\t"
      "buffer_load_dwordx4 v[188:191], v252, s[40:43], s62 offen offset:3072 sc0 sc1\n\t"
      "v_mfma_i32_16x16x64_i8 v[108:111], v[228:231], v[156:159], v[108:111]\n\t"
      "buffer_load_dwordx4 v[172:175], v252, s[40:43], s44 offen offset:3072 sc0 sc1\n\t"
      "s_waitcnt lgkmcnt(4)\n\t"
      "v_mfma_i32_16x16x64_i8 v[16:19], v[232:235], v[128:131], v[16:19]\n\t"
      "ds_read_b64_tr_b8 v[224:225], v250 offset:64\n\t"
      "v_readlane_b32 s49, v255, s63\n\t"
      "v_mfma_i32_16x16x64_i8 v[16:19], v[236:239], v[144:147], v[16:19]\n\t"
      "ds_read_b64_tr_b8 v[226:227], v250 offset:1088\n\t"
      "s_add_u32 s63, s63, 1\n\t"
      "v_mfma_i32_16x16x64_i8 v[48:51], v[232:235], v[132:135], v[48:51]\n\t"
      "ds_read_b64_tr_b8 v[228:229], v250 offset:8512\n\t"
      "s_lshl_b32 s44, s49, 15\n\t"
      "v_mfma_i32_16x16x64_i8 v[48:51], v[236:239], v[148:151], v[48:51]\n\t"
      "ds_read_b64_tr_b8 v[230:231], v250 offset:9536\n\t"
      "s_lshr_b32 s62, s49, 11\n\t"
      "v_mfma_i32_16x16x64_i8 v[80:83], v[232:235], v[136:139], v[80:83]\n\t"
      "v_add_u32 v250, s47, v250\n\t"
      "v_mfma_i32_16x16x64_i8 v[80:83], v[236:239], v[152:155], v[80:83]\n\t"
      "buffer_load_dwordx4 v[192:195], v252, s[40:43], s44 offen offset:0 sc0 sc1\n\t"
      "v_mfma_i32_16x16x64_i8 v[112:115], v[232:235], v[140:143], v[112:115]\n\t"
      "v_mfma_i32_16x16x64_i8 v[112:115], v[236:239], v[156:159], v[112:115]\n\t"
      "buffer_load_dwordx4 v[208:211], v252, s[40:43], s62 offen offset:0 sc0 sc1\n\t"
      "s_waitcnt lgkmcnt(4)\n\t"
      "v_mfma_i32_16x16x64_i8 v[20:23], v[240:243], v[128:131], v[20:23]\n\t"
      "ds_read_b64_tr_b8 v[232:233], v251 offset:64\n\t"
      "v_readfirstlane_b32 s53, v253\n\t"
      "v_mfma_i32_16x16x64_i8 v[20:23], v[244:247], v[144:147], v[20:23]\n\t"
      "ds_read_b64_tr_b8 v[234:235], v251 offset:1088\n\t"
      "s_add_u32 s48, s48, 0x10101\n\t"
      "v_mfma_i32_16x16x64_i8 v[52:55], v[240:243], v[132:135], v[52:55]\n\t"
      "ds_read_b64_tr_b8 v[236:237], v251 offset:8512\n\t"
      "s_or_b32 s49, s53, 0x808080\n\t"
      "v_mfma_i32_16x16x64_i8 v[52:55], v[244:247], v[148:151], v[52:55]\n\t"
      "ds_read_b64_tr_b8 v[238:239], v251 offset:9536\n\t"
      "s_sub_u32 s49, s49, s48\n\t"
      "v_mfma_i32_16x16x64_i8 v[84:87], v[240:243], v[136:139], v[84:87]\n\t"
      "v_add_u32 v251, s47, v251\n\t"
      "s_and_b32 s49, s49, 0x808080\n\t"
      "v_mfma_i32_16x16x64_i8 v[84:87], v[244:247], v[152:155], v[84:87]\n\t"
      "buffer_load_dwordx4 v[196:199], v252, s[40:43], s44 offen offset:1024 sc0 sc1\n\t"
      "v_mfma_i32_16x16x64_i8 v[116:119], v[240:243], v[140:143], v[116:119]\n\t"
      "s_cmp_eq_u32 s49, 0x808080\n\t"
      "s_cbranch_scc0 Lslow5_%=\n\t"
      "Lback5_%=:\n\t"
      "v_mfma_i32_16x16x64_i8 v[116:119], v[244:247], v[156:159], v[116:119]\n\t"
      "buffer_load_dwordx4 v[212:215], v252, s[40:43], s62 offen offset:1024 sc0 sc1\n\t"
      "s_waitcnt lgkmcnt(4)\n\t"
      "v_mfma_i32_16x16x64_i8 v[24:27], v[224:227], v[128:131], v[24:27]\n\t"
      "ds_read_b64_tr_b8 v[240:241], v248 offset:0\n\t"
      "v_mfma_i32_16x16x64_i8 v[24:27], v[228:231], v[144:147], v[24:27]\n\t"
      "ds_read_b64_tr_b8 v[242:243], v248 offset:1024\n\t"
      "v_mfma_i32_16x16x64_i8 v[56:59], v[224:227], v[132:135], v[56:59]\n\t"
      "ds_read_b64_tr_b8 v[244:245], v248 offset:8448\n\t"
      "v_mfma_i32_16x16x64_i8 v[56:59], v[228:231], v[148:151], v[56:59]\n\t"
      "ds_read_b64_tr_b8 v[246:247], v248 offset:9472\n\t"
      "v_mfma_i32_16x16x64_i8 v[88:91], v[224:227], v[136:139], v[88:91]\n\t"
      "v_mfma_i32_16x16x64_i8 v[88:91], v[228:231], v[152:155], v[88:91]\n\t"
      "buffer_load_dwordx4 v[200:203], v252, s[40:43], s44 offen offset:2048 sc0 sc1\n\t"
      "v_mfma_i32_16x16x64_i8 v[120:123], v[224:227], v[140:143], v[120:123]\n\t"
      "v_mfma_i32_16x16x64_i8 v[120:123], v[228:231], v[156:159], v[120:123]\n\t"
      "buffer_load_dwordx4 v[216:219], v252, s[40:43], s62 offen offset:2048 sc0 sc1\n\t"
      "s_waitcnt lgkmcnt(4)\n\t"
      "v_mfma_i32_16x16x64_i8 v[28:31], v[232:235], v[128:131], v[28:31]\n\t"
      "ds_read_b64_tr_b8 v[224:225], v249 offset:0\n\t"
      "v_mfma_i32_16x16x64_i8 v[28:31], v[236:239], v[144:147], v[28:31]\n\t"
      "ds_read_b64_tr_b8 v[226:227], v249 offset:1024\n\t"
      "v_mfma_i32_16x16x64_i8 v[60:63], v[232:235], v[132:135], v[60:63]\n\t"
      "ds_read_b64_tr_b8 v[228:229], v249 offset:8448\n\t"
      "v_mfma_i32_16x16x64_i8 v[60:63], v[236:239], v[148:151], v[60:63]\n\t"
      "ds_read_b64_tr_b8 v[230:231], v249 offset:9472\n\t"
      "v_mfma_i32_16x16x64_i8 v[92:95], v[232:235], v[136:139], v[92:95]\n\t"
      "s_add_u32 s51, s51, 1\n\t"
      "v_mfma_i32_16x16x64_i8 v[92:95], v[236:239], v[152:155], v[92:95]\n\t"
      "buffer_load_dwordx4 v[204:207], v252, s[40:43], s44 offen offset:3072 sc0 sc1\n\t"
      "v_mfma_i32_16x16x64_i8 v[124:127], v[232:235], v[140:143], v[124:127]\n\t"
      "v_mov_b32 v253, s51\n\t"
      "v_mfma_i32_16x16x64_i8 v[124:127], v[236:239], v[156:159], v[124:127]\n\t"
      "ds_write_addtid_b32 v253 offset:1024\n\t"
      "buffer_load_dwordx4 v[220:223], v252, s[40:43], s62 offen offset:3072 sc0 sc1\n\t"
      "Lloop_%=:\n\t"
      "s_waitcnt vmcnt(8)\n\t"
      "s_waitcnt lgkmcnt(4)\n\t"
      "v_mfma_i32_16x16x64_i8 v[0:3], v[240:243], v[160:163], v[0:3]\n\t"
      "ds_read_b64_tr_b8 v[232:233], v250 offset:0\n\t"
      "v_readlane_b32 s49, v255, s63\n\t"
      "v_mfma_i32_16x16x64_i8 v[0:3], v[244:247], v[176:179], v[0:3]\n\t"
      "ds_read_b64_tr_b8 v[234:235], v250 offset:1024\n\t"
      "s_add_u32 s63, s63, 1\n\t"
      "v_mfma_i32_16x16x64_i8 v[32:35], v[240:243], v[164:167], v[32:35]\n\t"
      "ds_read_b64_tr_b8 v[236:237], v250 offset:8448\n\t"
      "s_lshl_b32 s44, s49, 15\n\t"
      "v_mfma_i32_16x16x64_i8 v[32:35], v[244:247], v[180:183], v[32:35]\n\t"
      "ds_read_b64_tr_b8 v[238:239], v250 offset:9472\n\t"
      "s_lshr_b32 s62, s49, 11\n\t"
      "v_mfma_i32_16x16x64_i8 v[64:67], v[240:243], v[168:171], v[64:67]\n\t"
      "s_add_u32 s46, s46, 1\n\t"
      "s_cmp_eq_u32 s46, 9\n\t"
      "v_mfma_i32_16x16x64_i8 v[64:67], v[244:247], v[184:187], v[64:67]\n\t"
      "s_cselect_b32 s47, s60, s61\n\t"
      "v_mfma_i32_16x16x64_i8 v[96:99], v[240:243], v[172:175], v[96:99]\n\t"
      "s_cselect_b32 s46, 0, s46\n\t"
      "v_mfma_i32_16x16x64_i8 v[96:99], v[244:247], v[188:191], v[96:99]\n\t"
      "buffer_load_dwordx4 v[128:131], v252, s[40:43], s44 offen offset:0 sc0 sc1\n\t"
      "s_waitcnt lgkmcnt(4)\n\t"
      "v_mfma_i32_16x16x64_i8 v[4:7], v[224:227], v[160:163], v[4:7]\n\t"
      "ds_read_b64_tr_b8 v[240:241], v251 offset:0\n\t"
      "v_mfma_i32_16x16x64_i8 v[4:7], v[228:231], v[176:179], v[4:7]\n\t"
      "ds_read_b64_tr_b8 v[242:243], v251 offset:1024\n\t"
      "v_mfma_i32_16x16x64_i8 v[36:39], v[224:227], v[164:167], v[36:39]\n\t"
      "ds_read_b64_tr_b8 v[244:245], v251 offset:8448\n\t"
      "v_mfma_i32_16x16x64_i8 v[36:39], v[228:231], v[180:183], v[36:39]\n\t"
      "ds_read_b64_tr_b8 v[246:247], v251 offset:9472\n\t"
      "v_mfma_i32_16x16x64_i8 v[68:71], v[224:227], v[168:171], v[68:71]\n\t"
      "v_mfma_i32_16x16x64_i8 v[68:71], v[228:231], v[184:187], v[68:71]\n\t"
      "buffer_load_dwordx4 v[144:147], v252, s[40:43], s62 offen offset:0 sc0 sc1\n\t"
      "v_mfma_i32_16x16x64_i8 v[100:103], v[224:227], v[172:175], v[100:103]\n\t"
      "v_mfma_i32_16x16x64_i8 v[100:103], v[228:231], v[188:191], v[100:103]\n\t"
      "s_waitcnt lgkmcnt(4)\n\t"
      "v_mfma_i32_16x16x64_i8 v[8:11], v[232:235], v[160:163], v[8:11]\n\t"
      "ds_read_b64_tr_b8 v[224:225], v248 offset:64\n\t"
      "v_mfma_i32_16x16x64_i8 v[8:11], v[236:239], v[176:179], v[8:11]\n\t"
      "ds_read_b64_tr_b8 v[226:227], v248 offset:1088\n\t"
      "v_mfma_i32_16x16x64_i8 v[40:43], v[232:235], v[164:167], v[40:43]\n\t"
      "ds_read_b64_tr_b8 v[228:229], v248 offset:8512\n\t"
      "v_mfma_i32_16x16x64_i8 v[40:43], v[236:239], v[180:183], v[40:43]\n\t"
      "ds_read_b64_tr_b8 v[230:231], v248 offset:9536\n\t"
      "v_mfma_i32_16x16x64_i8 v[72:75], v[232:235], v[168:171], v[72:75]\n\t"
      "v_add_u32 v248, s47, v248\n\t"
      "v_mfma_i32_16x16x64_i8 v[72:75], v[236:239], v[184:187], v[72:75]\n\t"
      "buffer_load_dwordx4 v[132:135], v252, s[40:43], s44 offen offset:1024 sc0 sc1\n\t"
      "v_mfma_i32_16x16x64_i8 v[104:107], v[232:235], v[172:175], v[104:107]\n\t"
      "v_mfma_i32_16x16x64_i8 v[104:107], v[236:239], v[188:191], v[104:107]\n\t"
      "s_waitcnt lgkmcnt(4)\n\t"
      "v_mfma_i32_16x16x64_i8 v[12:15], v[240:243], v[160:163], v[12:15]\n\t"
      "ds_read_b64_tr_b8 v[232:233], v249 offset:64\n\t"
      "v_mfma_i32_16x16x64_i8 v[12:15], v[244:247], v[176:179], v[12:15]\n\t"
      "ds_read_b64_tr_b8 v[234:235], v249 offset:1088\n\t"
      "v_mfma_i32_16x16x64_i8 v[44:47], v[240:243], v[164:167], v[44:47]\n\t"
      "ds_read_b64_tr_b8 v[236:237], v249 offset:8512\n\t"
      "v_mfma_i32_16x16x64_i8 v[44:47], v[244:247], v[180:183], v[44:47]\n\t"
      "ds_read_b64_tr_b8 v[238:239], v249 offset:9536\n\t"
      "v_mfma_i32_16x16x64_i8 v[76:79], v[240:243], v[168:171], v[76:79]\n\t"
      "v_add_u32 v249, s47, v249\n\t"
      "ds_read_addtid_b32 v253 offset:0\n\t"
      "v_mfma_i32_16x16x64_i8 v[76:79], v[244:247], v[184:187], v[76:79]\n\t"
      "buffer_load_dwordx4 v[148:151], v252, s[40:43], s62 offen offset:1024 sc0 sc1\n\t"
      "v_mfma_i32_16x16x64_i8 v[108:111], v[240:243], v[172:175], v[108:111]\n\t"
      "v_mfma_i32_16x16x64_i8 v[108:111], v[244:247], v[188:191], v[108:111]\n\t"
      "s_waitcnt lgkmcnt(4)\n\t"
      "v_mfma_i32_16x16x64_i8 v[16:19], v[224:227], v[160:163], v[16:19]\n\t"
      "ds_read_b64_tr_b8 v[240:241], v250 offset:64\n\t"
      "v_mfma_i32_16x16x64_i8 v[16:19], v[228:231], v[176:179], v[16:19]\n\t"
      "ds_read_b64_tr_b8 v[242:243], v250 offset:1088\n\t"
      "v_mfma_i32_16x16x64_i8 v[48:51], v[224:227], v[164:167], v[48:51]\n\t"
      "ds_read_b64_tr_b8 v[244:245], v250 offset:8512\n\t"
      "v_mfma_i32_16x16x64_i8 v[48:51], v[228:231], v[180:183], v[48:51]\n\t"
      "ds_read_b64_tr_b8 v[246:247], v250 offset:9536\n\t"
      "v_mfma_i32_16x16x64_i8 v[80:83], v[224:227], v[168:171], v[80:83]\n\t"
      "v_add_u32 v250, s47, v250\n\t"
      "v_mfma_i32_16x16x64_i8 v[80:83], v[228:231], v[184:187], v[80:83]\n\t"
      "buffer_load_dwordx4 v[136:139], v252, s[40:43], s44 offen offset:2048 sc0 sc1\n\t"
      "v_mfma_i32_16x16x64_i8 v[112:115], v[224:227], v[172:175], v[112:115]\n\t"
      "v_mfma_i32_16x16x64_i8 v[112:115], v[228:231], v[188:191], v[112:115]\n\t"
      "s_waitcnt lgkmcnt(4)\n\t"
      "v_mfma_i32_16x16x64_i8 v[20:23], v[232:235], v[160:163], v[20:23]\n\t"
      "ds_read_b64_tr_b8 v[224:225], v251 offset:64\n\t"
      "v_readfirstlane_b32 s53, v253\n\t"
      "v_mfma_i32_16x16x64_i8 v[20:23], v[236:239], v[176:179], v[20:23]\n\t"
      "ds_read_b64_tr_b8 v[226:227], v251 offset:1088\n\t"
      "s_add_u32 s48, s48, 0x10101\n\t"
      "v_mfma_i32_16x16x64_i8 v[52:55], v[232:235], v[164:167], v[52:55]\n\t"
      "ds_read_b64_tr_b8 v[228:229], v251 offset:8512\n\t"
      "s_or_b32 s49, s53, 0x808080\n\t"
      "v_mfma_i32_16x16x64_i8 v[52:55], v[236:239], v[180:183], v[52:55]\n\t"
      "ds_read_b64_tr_b8 v[230:231], v251 offset:9536\n\t"
      "s_sub_u32 s49, s49, s48\n\t"
      "v_mfma_i32_16x16x64_i8 v[84:87], v[232:235], v[168:171], v[84:87]\n\t"
      "v_add_u32 v251, s47, v251\n\t"
      "s_and_b32 s49, s49, 0x808080\n\t"
      "v_mfma_i32_16x16x64_i8 v[84:87], v[236:239], v[184:187], v[84:87]\n\t"
      "buffer_load_dwordx4 v[152:155], v252, s[40:43], s62 offen offset:2048 sc0 sc1\n\t"
      "v_mfma_i32_16x16x64_i8 v[116:119], v[232:235], v[172:175], v[116:119]\n\t"
      "s_cmp_eq_u32 s49, 0x808080\n\t"
      "s_cbranch_scc0 Lslow6_%=\n\t"
      "Lback6_%=:\n\t"
      "v_mfma_i32_16x16x64_i8 v[116:119], v[236:239], v[188:191], v[116:119]\n\t"
      "s_waitcnt lgkmcnt(4)\n\t"
      "v_mfma_i32_16x16x64_i8 v[24:27], v[240:243], v[160:163], v[24:27]\n\t"
      "ds_read_b64_tr_b8 v[232:233], v248 offset:0\n\t"
      "v_mfma_i32_16x16x64_i8 v[24:27], v[244:247], v[176:179], v[24:27]\n\t"
      "ds_read_b64_tr_b8 v[234:235], v248 offset:1024\n\t"
      "v_mfma_i32_16x16x64_i8 v[56:59], v[240:243], v[164:167], v[56:59]\n\t"
      "ds_read_b64_tr_b8 v[236:237], v248 offset:8448\n\t"
      "v_mfma_i32_16x16x64_i8 v[56:59], v[244:247], v[180:183], v[56:59]\n\t"
      "ds_read_b64_tr_b8 v[238:239], v248 offset:9472\n\t"
      "v_mfma_i32_16x16x64_i8 v[88:91], v[240:243], v[168:171], v[88:91]\n\t"
      "v_mfma_i32_16x16x64_i8 v[88:91], v[244:247], v[184:187], v[88:91]\n\t"
      "buffer_load_dwordx4 v[140:143], v252, s[40:43], s44 offen offset:3072 sc0 sc1\n\t"
      "v_mfma_i32_16x16x64_i8 v[120:123], v[240:243], v[172:175], v[120:123]\n\t"
      "v_mfma_i32_16x16x64_i8 v[120:123], v[244:247], v[188:191], v[120:123]\n\t"
      "s_waitcnt lgkmcnt(4)\n\t"
      "v_mfma_i32_16x16x64_i8 v[28:31], v[224:227], v[160:163], v[28:31]\n\t"
      "ds_read_b64_tr_b8 v[240:241], v249 offset:0\n\t"
      "v_mfma_i32_16x16x64_i8 v[28:31], v[228:231], v[176:179], v[28:31]\n\t"
      "ds_read_b64_tr_b8 v[242:243], v249 offset:1024\n\t"
      "v_mfma_i32_16x16x64_i8 v[60:63], v[224:227], v[164:167], v[60:63]\n\t"
      "ds_read_b64_tr_b8 v[244:245], v249 offset:8448\n\t"
      "v_mfma_i32_16x16x64_i8 v[60:63], v[228:231], v[180:183], v[60:63]\n\t"
      "ds_read_b64_tr_b8 v[246:247], v249 offset:9472\n\t"
      "v_mfma_i32_16x16x64_i8 v[92:95], v[224:227], v[168:171], v[92:95]\n\t"
      "s_add_u32 s51, s51, 1\n\t"
      "v_mfma_i32_16x16x64_i8 v[92:95], v[228:231], v[184:187], v[92:95]\n\t"
      "buffer_load_dwordx4 v[156:159], v252, s[40:43], s62 offen offset:3072 sc0 sc1\n\t"
      "v_mfma_i32_16x16x64_i8 v[124:127], v[224:227], v[172:175], v[124:127]\n\t"
      "v_mov_b32 v253, s51\n\t"
      "v_mfma_i32_16x16x64_i8 v[124:127], v[228:231], v[188:191], v[124:127]\n\t"
      "ds_write_addtid_b32 v253 offset:1024\n\t"
      "s_waitcnt vmcnt(8)\n\t"
      "s_waitcnt lgkmcnt(4)\n\t"
      "v_mfma_i32_16x16x64_i8 v[0:3], v[232:235], v[192:195], v[0:3]\n\t"
      "ds_read_b64_tr_b8 v[224:225], v250 offset:0\n\t"
      "v_readlane_b32 s49, v255, s63\n\t"
      "v_mfma_i32_16x16x64_i8 v[0:3], v[236:239], v[208:211], v[0:3]\n\t"
      "ds_read_b64_tr_b8 v[226:227], v250 offset:1024\n\t"
      "s_add_u32 s63, s63, 1\n\t"
      "v_mfma_i32_16x16x64_i8 v[32:35], v[232:235], v[196:199], v[32:35]\n\t"
      "ds_read_b64_tr_b8 v[228:229], v250 offset:8448\n\t"
      "s_lshl_b32 s44, s49, 15\n\t"
      "v_mfma_i32_16x16x64_i8 v[32:35], v[236:239], v[212:215], v[32:35]\n\t"
      "ds_read_b64_tr_b8 v[230:231], v250 offset:9472\n\t"
      "s_lshr_b32 s62, s49, 11\n\t"
      "v_mfma_i32_16x16x64_i8 v[64:67], v[232:235], v[200:203], v[64:67]\n\t"
      "s_add_u32 s46, s46, 1\n\t"
      "s_cmp_eq_u32 s46, 9\n\t"
      "v_mfma_i32_16x16x64_i8 v[64:67], v[236:239], v[216:219], v[64:67]\n\t"
      "s_cselect_b32 s47, s60, s61\n\t"
      "v_mfma_i32_16x16x64_i8 v[96:99], v[232:235], v[204:207], v[96:99]\n\t"
      "s_cselect_b32 s46, 0, s46\n\t"
      "v_mfma_i32_16x16x64_i8 v[96:99], v[236:239], v[220:223], v[96:99]\n\t"
      "buffer_load_dwordx4 v[160:163], v252, s[40:43], s44 offen offset:0 sc0 sc1\n\t"
      "s_waitcnt lgkmcnt(4)\n\t"
      "v_mfma_i32_16x16x64_i8 v[4:7], v[240:243], v[192:195], v[4:7]\n\t"
      "ds_read_b64_tr_b8 v[232:233], v251 offset:0\n\t"
      "v_mfma_i32_16x16x64_i8 v[4:7], v[244:247], v[208:211], v[4:7]\n\t"
      "ds_read_b64_tr_b8 v[234:235], v251 offset:1024\n\t"
      "v_mfma_i32_16x16x64_i8 v[36:39], v[240:243], v[196:199], v[36:39]\n\t"
      "ds_read_b64_tr_b8 v[236:237], v251 offset:8448\n\t"
      "v_mfma_i32_16x16x64_i8 v[36:39], v[244:247], v[212:215], v[36:39]\n\t"
      "ds_read_b64_tr_b8 v[238:239], v251 offset:9472\n\t"
      "v_mfma_i32_16x16x64_i8 v[68:71], v[240:243], v[200:203], v[68:71]\n\t"
      "v_mfma_i32_16x16x64_i8 v[68:71], v[244:247], v[216:219], v[68:71]\n\t"
      "buffer_load_dwordx4 v[176:179], v252, s[40:43], s62 offen offset:0 sc0 sc1\n\t"
      "v_mfma_i32_16x16x64_i8 v[100:103], v[240:243], v[204:207], v[100:103]\n\t"
      "v_mfma_i32_16x16x64_i8 v[100:103], v[244:247], v[220:223], v[100:103]\n\t"
      "s_waitcnt lgkmcnt(4)\n\t"
      "v_mfma_i32_16x16x64_i8 v[8:11], v[224:227], v[192:195], v[8:11]\n\t"
      "ds_read_b64_tr_b8 v[240:241], v248 offset:64\n\t"
      "v_mfma_i32_16x16x64_i8 v[8:11], v[228:231], v[208:211], v[8:11]\n\t"
      "ds_read_b64_tr_b8 v[242:243], v248 offset:1088\n\t"
      "v_mfma_i32_16x16x64_i8 v[40:43], v[224:227], v[196:199], v[40:43]\n\t"
      "ds_read_b64_tr_b8 v[244:245], v248 offset:8512\n\t"
      "v_mfma_i32_16x16x64_i8 v[40:43], v[228:231], v[212:215], v[40:43]\n\t"
      "ds_read_b64_tr_b8 v[246:247], v248 offset:9536\n\t"
      "v_mfma_i32_16x16x64_i8 v[72:75], v[224:227], v[200:203], v[72:75]\n\t"
      "v_add_u32 v248, s47, v248\n\t"
      "v_mfma_i32_16x16x64_i8 v[72:75], v[228:231], v[216:219], v[72:75]\n\t"
      "buffer_load_dwordx4 v[164:167], v252, s[40:43], s44 offen offset:1024 sc0 sc1\n\t"
      "v_mfma_i32_16x16x64_i8 v[104:107], v[224:227], v[204:207], v[104:107]\n\t"
      "v_mfma_i32_16x16x64_i8 v[104:107], v[228:231], v[220:223], v[104:107]\n\t"
      "s_waitcnt lgkmcnt(4)\n\t"
      "v_mfma_i32_16x16x64_i8 v[12:15], v[232:235], v[192:195], v[12:15]\n\t"
      "ds_read_b64_tr_b8 v[224:225], v249 offset:64\n\t"
      "v_mfma_i32_16x16x64_i8 v[12:15], v[236:239], v[208:211], v[12:15]\n\t"
      "ds_read_b64_tr_b8 v[226:227], v249 offset:1088\n\t"
      "v_mfma_i32_16x16x64_i8 v[44:47], v[232:235], v[196:199], v[44:47]\n\t"
      "ds_read_b64_tr_b8 v[228:229], v249 offset:8512\n\t"
      "v_mfma_i32_16x16x64_i8 v[44:47], v[236:239], v[212:215], v[44:47]\n\t"
      "ds_read_b64_tr_b8 v[230:231], v249 offset:9536\n\t"
      "v_mfma_i32_16x16x64_i8 v[76:79], v[232:235], v[200:203], v[76:79]\n\t"
      "v_add_u32 v249, s47, v249\n\t"
      "ds_read_addtid_b32 v253 offset:0\n\t"
      "v_mfma_i32_16x16x64_i8 v[76:79], v[236:239], v[216:219], v[76:79]\n\t"
      "buffer_load_dwordx4 v[180:183], v252, s[40:43], s62 offen offset:1024 sc0 sc1\n\t"
      "v_mfma_i32_16x16x64_i8 v[108:111], v[232:235], v[204:207], v[108:111]\n\t"
      "v_mfma_i32_16x16x64_i8 v[108:111], v[236:239], v[220:223], v[108:111]\n\t"
      "s_waitcnt lgkmcnt(4)\n\t"
      "v_mfma_i32_16x16x64_i8 v[16:19], v[240:243], v[192:195], v[16:19]\n\t"
      "ds_read_b64_tr_b8 v[232:233], v250 offset:64\n\t"
      "v_mfma_i32_16x16x64_i8 v[16:19], v[244:247], v[208:211], v[16:19]\n\t"
      "ds_read_b64_tr_b8 v[234:235], v250 offset:1088\n\t"
      "v_mfma_i32_16x16x64_i8 v[48:51], v[240:243], v[196:199], v[48:51]\n\t"
      "ds_read_b64_tr_b8 v[236:237], v250 offset:8512\n\t"
      "v_mfma_i32_16x16x64_i8 v[48:51], v[244:247], v[212:215], v[48:51]\n\t"
      "ds_read_b64_tr_b8 v[238:239], v250 offset:9536\n\t"
      "v_mfma_i32_16x16x64_i8 v[80:83], v[240:243], v[200:203], v[80:83]\n\t"
      "v_add_u32 v250, s47, v250\n\t"
      "v_mfma_i32_16x16x64_i8 v[80:83], v[244:247], v[216:219], v[80:83]\n\t"
      "buffer_load_dwordx4 v[168:171], v252, s[40:43], s44 offen offset:2048 sc0 sc1\n\t"
      "v_mfma_i32_16x16x64_i8 v[112:115], v[240:243], v[204:207], v[112:115]\n\t"
      "v_mfma_i32_16x16x64_i8 v[112:115], v[244:247], v[220:223], v[112:115]\n\t"
      "s_waitcnt lgkmcnt(4)\n\t"
      "v_mfma_i32_16x16x64_i8 v[20:23], v[224:227], v[192:195], v[20:23]\n\t"
      "ds_read_b64_tr_b8 v[240:241], v251 offset:64\n\t"
      "v_readfirstlane_b32 s53, v253\n\t"
      "v_mfma_i32_16x16x64_i8 v[20:23], v[228:231], v[208:211], v[20:23]\n\t"
      "ds_read_b64_tr_b8 v[242:243], v251 offset:1088\n\t"
      "s_add_u32 s48, s48, 0x10101\n\t"
      "v_mfma_i32_16x16x64_i8 v[52:55], v[224:227], v[196:199], v[52:55]\n\t"
      "ds_read_b64_tr_b8 v[244:245], v251 offset:8512\n\t"
      "s_or_b32 s49, s53, 0x808080\n\t"
      "v_mfma_i32_16x16x64_i8 v[52:55], v[228:231], v[212:215], v[52:55]\n\t"
      "ds_read_b64_tr_b8 v[246:247], v251 offset:9536\n\t"
      "s_sub_u32 s49, s49, s48\n\t"
      "v_mfma_i32_16x16x64_i8 v[84:87], v[224:227], v[200:203], v[84:87]\n\t"
      "v_add_u32 v251, s47, v251\n\t"
      "s_and_b32 s49, s49, 0x808080\n\t"
      "v_mfma_i32_16x16x64_i8 v[84:87], v[228:231], v[216:219], v[84:87]\n\t"
      "buffer_load_dwordx4 v[184:187], v252, s[40:43], s62 offen offset:2048 sc0 sc1\n\t"
      "v_mfma_i32_16x16x64_i8 v[116:119], v[224:227], v[204:207], v[116:119]\n\t"
      "s_cmp_eq_u32 s49, 0x808080\n\t"
      "s_cbranch_scc0 Lslow7_%=\n\t"
      "Lback7_%=:\n\t"
      "v_mfma_i32_16x16x64_i8 v[116:119], v[228:231], v[220:223], v[116:119]\n\t"
      "s_waitcnt lgkmcnt(4)\n\t"
      "v_mfma_i32_16x16x64_i8 v[24:27], v[232:235], v[192:195], v[24:27]\n\t"
      "ds_read_b64_tr_b8 v[224:225], v248 offset:0\n\t"
      "v_mfma_i32_16x16x64_i8 v[24:27], v[236:239], v[208:211], v[24:27]\n\t"
      "ds_read_b64_tr_b8 v[226:227], v248 offset:1024\n\t"
      "v_mfma_i32_16x16x64_i8 v[56:59], v[232:235], v[196:199], v[56:59]\n\t"
      "ds_read_b64_tr_b8 v[228:229], v248 offset:8448\n\t"
      "v_mfma_i32_16x16x64_i8 v[56:59], v[236:239], v[212:215], v[56:59]\n\t"
      "ds_read_b64_tr_b8 v[230:231], v248 offset:9472\n\t"
      "v_mfma_i32_16x16x64_i8 v[88:91], v[232:235], v[200:203], v[88:91]\n\t"
      "v_mfma_i32_16x16x64_i8 v[88:91], v[236:239], v[216:219], v[88:91]\n\t"
      "buffer_load_dwordx4 v[172:175], v252, s[40:43], s44 offen offset:3072 sc0 sc1\n\t"
      "v_mfma_i32_16x16x64_i8 v[120:123], v[232:235], v[204:207], v[120:123]\n\t"
      "v_mfma_i32_16x16x64_i8 v[120:123], v[236:239], v[220:223], v[120:123]\n\t"
      "s_waitcnt lgkmcnt(4)\n\t"
      "v_mfma_i32_16x16x64_i8 v[28:31], v[240:243], v[192:195], v[28:31]\n\t"
      "ds_read_b64_tr_b8 v[232:233], v249 offset:0\n\t"
      "v_mfma_i32_16x16x64_i8 v[28:31], v[244:247], v[208:211], v[28:31]\n\t"
      "ds_read_b64_tr_b8 v[234:235], v249 offset:1024\n\t"
      "v_mfma_i32_16x16x64_i8 v[60:63], v[240:243], v[196:199], v[60:63]\n\t"
      "ds_read_b64_tr_b8 v[236:237], v249 offset:8448\n\t"
      "v_mfma_i32_16x16x64_i8 v[60:63], v[244:247], v[212:215], v[60:63]\n\t"
      "ds_read_b64_tr_b8 v[238:239], v249 offset:9472\n\t"
      "v_mfma_i32_16x16x64_i8 v[92:95], v[240:243], v[200:203], v[92:95]\n\t"
      "s_add_u32 s51, s51, 1\n\t"
      "v_mfma_i32_16x16x64_i8 v[92:95], v[244:247], v[216:219], v[92:95]\n\t"
      "buffer_load_dwordx4 v[188:191], v252, s[40:43], s62 offen offset:3072 sc0 sc1\n\t"
      "v_mfma_i32_16x16x64_i8 v[124:127], v[240:243], v[204:207], v[124:127]\n\t"
      "v_mov_b32 v253, s51\n\t"
      "v_mfma_i32_16x16x64_i8 v[124:127], v[244:247], v[220:223], v[124:127]\n\t"
      "ds_write_addtid_b32 v253 offset:1024\n\t"
      "s_waitcnt vmcnt(8)\n\t"
      "s_waitcnt lgkmcnt(4)\n\t"
      "v_mfma_i32_16x16x64_i8 v[0:3], v[224:227], v[128:131], v[0:3]\n\t"
      "ds_read_b64_tr_b8 v[240:241], v250 offset:0\n\t"
      "v_readlane_b32 s49, v255, s63\n\t"
      "v_mfma_i32_16x16x64_i8 v[0:3], v[228:231], v[144:147], v[0:3]\n\t"
      "ds_read_b64_tr_b8 v[242:243], v250 offset:1024\n\t"
      "s_add_u32 s63, s63, 1\n\t"
      "v_mfma_i32_16x16x64_i8 v[32:35], v[224:227], v[132:135], v[32:35]\n\t"
      "ds_read_b64_tr_b8 v[244:245], v250 offset:8448\n\t"
      "s_lshl_b32 s44, s49, 15\n\t"
      "v_mfma_i32_16x16x64_i8 v[32:35], v[228:231], v[148:151], v[32:35]\n\t"
      "ds_read_b64_tr_b8 v[246:247], v250 offset:9472\n\t"
      "s_lshr_b32 s62, s49, 11\n\t"
      "v_mfma_i32_16x16x64_i8 v[64:67], v[224:227], v[136:139], v[64:67]\n\t"
      "s_add_u32 s46, s46, 1\n\t"
      "s_cmp_eq_u32 s46, 9\n\t"
      "v_mfma_i32_16x16x64_i8 v[64:67], v[228:231], v[152:155], v[64:67]\n\t"
      "s_cselect_b32 s47, s60, s61\n\t"
      "v_mfma_i32_16x16x64_i8 v[96:99], v[224:227], v[140:143], v[96:99]\n\t"
      "s_cselect_b32 s46, 0, s46\n\t"
      "v_mfma_i32_16x16x64_i8 v[96:99], v[228:231], v[156:159], v[96:99]\n\t"
      "buffer_load_dwordx4 v[192:195], v252, s[40:43], s44 offen offset:0 sc0 sc1\n\t"
      "s_waitcnt lgkmcnt(4)\n\t"
      "v_mfma_i32_16x16x64_i8 v[4:7], v[232:235], v[128:131], v[4:7]\n\t"
      "ds_read_b64_tr_b8 v[224:225], v251 offset:0\n\t"
      "v_mfma_i32_16x16x64_i8 v[4:7], v[236:239], v[144:147], v[4:7]\n\t"
      "ds_read_b64_tr_b8 v[226:227], v251 offset:1024\n\t"
      "v_mfma_i32_16x16x64_i8 v[36:39], v[232:235], v[132:135], v[36:39]\n\t"
      "ds_read_b64_tr_b8 v[228:229], v251 offset:8448\n\t"
      "v_mfma_i32_16x16x64_i8 v[36:39], v[236:239], v[148:151], v[36:39]\n\t"
      "ds_read_b64_tr_b8 v[230:231], v251 offset:9472\n\t"
      "v_mfma_i32_16x16x64_i8 v[68:71], v[232:235], v[136:139], v[68:71]\n\t"
      "v_mfma_i32_16x16x64_i8 v[68:71], v[236:239], v[152:155], v[68:71]\n\t"
      "buffer_load_dwordx4 v[208:211], v252, s[40:43], s62 offen offset:0 sc0 sc1\n\t"
      "v_mfma_i32_16x16x64_i8 v[100:103], v[232:235], v[140:143], v[100:103]\n\t"
      "v_mfma_i32_16x16x64_i8 v[100:103], v[236:239], v[156:159], v[100:103]\n\t"
      "s_waitcnt lgkmcnt(4)\n\t"
      "v_mfma_i32_16x16x64_i8 v[8:11], v[240:243], v[128:131], v[8:11]\n\t"
      "ds_read_b64_tr_b8 v[232:233], v248 offset:64\n\t"
      "v_mfma_i32_16x16x64_i8 v[8:11], v[244:247], v[144:147], v[8:11]\n\t"
      "ds_read_b64_tr_b8 v[234:235], v248 offset:1088\n\t"
      "v_mfma_i32_16x16x64_i8 v[40:43], v[240:243], v[132:135], v[40:43]\n\t"
      "ds_read_b64_tr_b8 v[236:237], v248 offset:8512\n\t"
      "v_mfma_i32_16x16x64_i8 v[40:43], v[244:247], v[148:151], v[40:43]\n\t"
      "ds_read_b64_tr_b8 v[238:239], v248 offset:9536\n\t"
      "v_mfma_i32_16x16x64_i8 v[72:75], v[240:243], v[136:139], v[72:75]\n\t"
      "v_add_u32 v248, s47, v248\n\t"
      "v_mfma_i32_16x16x64_i8 v[72:75], v[244:247], v[152:155], v[72:75]\n\t"
      "buffer_load_dwordx4 v[196:199], v252, s[40:43], s44 offen offset:1024 sc0 sc1\n\t"
      "v_mfma_i32_16x16x64_i8 v[104:107], v[240:243], v[140:143], v[104:107]\n\t"
      "v_mfma_i32_16x16x64_i8 v[104:107], v[244:247], v[156:159], v[104:107]\n\t"
      "s_waitcnt lgkmcnt(4)\n\t"
      "v_mfma_i32_16x16x64_i8 v[12:15], v[224:227], v[128:131], v[12:15]\n\t"
      "ds_read_b64_tr_b8 v[240:241], v249 offset:64\n\t"
      "v_mfma_i32_16x16x64_i8 v[12:15], v[228:231], v[144:147], v[12:15]\n\t"
      "ds_read_b64_tr_b8 v[242:243], v249 offset:1088\n\t"
      "v_mfma_i32_16x16x64_i8 v[44:47], v[224:227], v[132:135], v[44:47]\n\t"
      "ds_read_b64_tr_b8 v[244:245], v249 offset:8512\n\t"
      "v_mfma_i32_16x16x64_i8 v[44:47], v[228:231], v[148:151], v[44:47]\n\t"
      "ds_read_b64_tr_b8 v[246:247], v249 offset:9536\n\t"
      "v_mfma_i32_16x16x64_i8 v[76:79], v[224:227], v[136:139], v[76:79]\n\t"
      "v_add_u32 v249, s47, v249\n\t"
      "ds_read_addtid_b32 v253 offset:0\n\t"
      "v_mfma_i32_16x16x64_i8 v[76:79], v[228:231], v[152:155], v[76:79]\n\t"
      "buffer_load_dwordx4 v[212:215], v252, s[40:43], s62 offen offset:1024 sc0 sc1\n\t"
      "v_mfma_i32_16x16x64_i8 v[108:111], v[224:227], v[140:143], v[108:111]\n\t"
      "v_mfma_i32_16x16x64_i8 v[108:111], v[228:231], v[156:159], v[108:111]\n\t"
      "s_waitcnt lgkmcnt(4)\n\t"
      "v_mfma_i32_16x16x64_i8 v[16:19], v[232:235], v[128:131], v[16:19]\n\t"
      "ds_read_b64_tr_b8 v[224:225], v250 offset:64\n\t"
      "v_mfma_i32_16x16x64_i8 v[16:19], v[236:239], v[144:147], v[16:19]\n\t"
      "ds_read_b64_tr_b8 v[226:227], v250 offset:1088\n\t"
      "v_mfma_i32_16x16x64_i8 v[48:51], v[232:235], v[132:135], v[48:51]\n\t"
      "ds_read_b64_tr_b8 v[228:229], v250 offset:8512\n\t"
      "v_mfma_i32_16x16x64_i8 v[48:51], v[236:239], v[148:151], v[48:51]\n\t"
      "ds_read_b64_tr_b8 v[230:231], v250 offset:9536\n\t"
      "v_mfma_i32_16x16x64_i8 v[80:83], v[232:235], v[136:139], v[80:83]\n\t"
      "v_add_u32 v250, s47, v250\n\t"
      "v_mfma_i32_16x16x64_i8 v[80:83], v[236:239], v[152:155], v[80:83]\n\t"
      "buffer_load_dwordx4 v[200:203], v252, s[40:43], s44 offen offset:2048 sc0 sc1\n\t"
      "v_mfma_i32_16x16x64_i8 v[112:115], v[232:235], v[140:143], v[112:115]\n\t"
      "v_mfma_i32_16x16x64_i8 v[112:115], v[236:239], v[156:159], v[112:115]\n\t"
      "s_waitcnt lgkmcnt(4)\n\t"
      "v_mfma_i32_16x16x64_i8 v[20:23], v[240:243], v[128:131], v[20:23]\n\t"
      "ds_read_b64_tr_b8 v[232:233], v251 offset:64\n\t"
      "v_readfirstlane_b32 s53, v253\n\t"
      "v_mfma_i32_16x16x64_i8 v[20:23], v[244:247], v[144:147], v[20:23]\n\t"
      "ds_read_b64_tr_b8 v[234:235], v251 offset:1088\n\t"
      "s_add_u32 s48, s48, 0x10101\n\t"
      "v_mfma_i32_16x16x64_i8 v[52:55], v[240:243], v[132:135], v[52:55]\n\t"
      "ds_read_b64_tr_b8 v[236:237], v251 offset:8512\n\t"
      "s_or_b32 s49, s53, 0x808080\n\t"
      "v_mfma_i32_16x16x64_i8 v[52:55], v[244:247], v[148:151], v[52:55]\n\t"
      "ds_read_b64_tr_b8 v[238:239], v251 offset:9536\n\t"
      "s_sub_u32 s49, s49, s48\n\t"
      "v_mfma_i32_16x16x64_i8 v[84:87], v[240:243], v[136:139], v[84:87]\n\t"
      "v_add_u32 v251, s47, v251\n\t"
      "s_and_b32 s49, s49, 0x808080\n\t"
      "v_mfma_i32_16x16x64_i8 v[84:87], v[244:247], v[152:155], v[84:87]\n\t"
      "buffer_load_dwordx4 v[216:219], v252, s[40:43], s62 offen offset:2048 sc0 sc1\n\t"
      "v_mfma_i32_16x16x64_i8 v[116:119], v[240:243], v[140:143], v[116:119]\n\t"
      "s_cmp_eq_u32 s49, 0x808080\n\t"
      "s_cbranch_scc0 Lslow8_%=\n\t"
      "Lback8_%=:\n\t"
      "v_mfma_i32_16x16x64_i8 v[116:119], v[244:247], v[156:159], v[116:119]\n\t"
      "s_waitcnt lgkmcnt(4)\n\t"
      "v_mfma_i32_16x16x64_i8 v[24:27], v[224:227], v[128:131], v[24:27]\n\t"
      "ds_read_b64_tr_b8 v[240:241], v248 offset:0\n\t"
      "v_mfma_i32_16x16x64_i8 v[24:27], v[228:231], v[144:147], v[24:27]\n\t"
      "ds_read_b64_tr_b8 v[242:243], v248 offset:1024\n\t"
      "v_mfma_i32_16x16x64_i8 v[56:59], v[224:227], v[132:135], v[56:59]\n\t"
      "ds_read_b64_tr_b8 v[244:245], v248 offset:8448\n\t"
      "v_mfma_i32_16x16x64_i8 v[56:59], v[228:231], v[148:151], v[56:59]\n\t"
      "ds_read_b64_tr_b8 v[246:247], v248 offset:9472\n\t"
      "v_mfma_i32_16x16x64_i8 v[88:91], v[224:227], v[136:139], v[88:91]\n\t"
      "v_mfma_i32_16x16x64_i8 v[88:91], v[228:231], v[152:155], v[88:91]\n\t"
      "buffer_load_dwordx4 v[204:207], v252, s[40:43], s44 offen offset:3072 sc0 sc1\n\t"
      "v_mfma_i32_16x16x64_i8 v[120:123], v[224:227], v[140:143], v[120:123]\n\t"
      "v_mfma_i32_16x16x64_i8 v[120:123], v[228:231], v[156:159], v[120:123]\n\t"
      "s_waitcnt lgkmcnt(4)\n\t"
      "v_mfma_i32_16x16x64_i8 v[28:31], v[232:235], v[128:131], v[28:31]\n\t"
      "ds_read_b64_tr_b8 v[224:225], v249 offset:0\n\t"
      "v_mfma_i32_16x16x64_i8 v[28:31], v[236:239], v[144:147], v[28:31]\n\t"
      "ds_read_b64_tr_b8 v[226:227], v249 offset:1024\n\t"
      "v_mfma_i32_16x16x64_i8 v[60:63], v[232:235], v[132:135], v[60:63]\n\t"
      "ds_read_b64_tr_b8 v[228:229], v249 offset:8448\n\t"
      "v_mfma_i32_16x16x64_i8 v[60:63], v[236:239], v[148:151], v[60:63]\n\t"
      "ds_read_b64_tr_b8 v[230:231], v249 offset:9472\n\t"
      "v_mfma_i32_16x16x64_i8 v[92:95], v[232:235], v[136:139], v[92:95]\n\t"
      "s_add_u32 s51, s51, 1\n\t"
      "v_mfma_i32_16x16x64_i8 v[92:95], v[236:239], v[152:155], v[92:95]\n\t"
      "buffer_load_dwordx4 v[220:223], v252, s[40:43], s62 offen offset:3072 sc0 sc1\n\t"
      "v_mfma_i32_16x16x64_i8 v[124:127], v[232:235], v[140:143], v[124:127]\n\t"
      "v_mov_b32 v253, s51\n\t"
      "v_mfma_i32_16x16x64_i8 v[124:127], v[236:239], v[156:159], v[124:127]\n\t"
      "ds_write_addtid_b32 v253 offset:1024\n\t"
      "s_sub_u32 s45, s45, 1\n\t"
      "s_cmp_lg_u32 s45, 0\n\t"
      "s_cbranch_scc1 Lloop_%=\n\t"
      "s_waitcnt vmcnt(8)\n\t"
      "s_waitcnt lgkmcnt(4)\n\t"
      "v_mfma_i32_16x16x64_i8 v[0:3], v[240:243], v[160:163], v[0:3]\n\t"
      "ds_read_b64_tr_b8 v[232:233], v250 offset:0\n\t"
      "v_readlane_b32 s49, v255, s63\n\t"
      "v_mfma_i32_16x16x64_i8 v[0:3], v[244:247], v[176:179], v[0:3]\n\t"
      "ds_read_b64_tr_b8 v[234:235], v250 offset:1024\n\t"
      "s_add_u32 s63, s63, 1\n\t"
      "v_mfma_i32_16x16x64_i8 v[32:35], v[240:243], v[164:167], v[32:35]\n\t"
      "ds_read_b64_tr_b8 v[236:237], v250 offset:8448\n\t"
      "s_lshl_b32 s44, s49, 15\n\t"
      "v_mfma_i32_16x16x64_i8 v[32:35], v[244:247], v[180:183], v[32:35]\n\t"
      "ds_read_b64_tr_b8 v[238:239], v250 offset:9472\n\t"
      "s_lshr_b32 s62, s49, 11\n\t"
      "v_mfma_i32_16x16x64_i8 v[64:67], v[240:243], v[168:171], v[64:67]\n\t"
      "s_add_u32 s46, s46, 1\n\t"
      "s_cmp_eq_u32 s46, 9\n\t"
      "v_mfma_i32_16x16x64_i8 v[64:67], v[244:247], v[184:187], v[64:67]\n\t"
      "s_cselect_b32 s47, s60, s61\n\t"
      "v_mfma_i32_16x16x64_i8 v[96:99], v[240:243], v[172:175], v[96:99]\n\t"
      "s_cselect_b32 s46, 0, s46\n\t"
      "v_mfma_i32_16x16x64_i8 v[96:99], v[244:247], v[188:191], v[96:99]\n\t"
      "buffer_load_dwordx4 v[128:131], v252, s[40:43], s44 offen offset:0 sc0 sc1\n\t"
      "s_waitcnt lgkmcnt(4)\n\t"
      "v_mfma_i32_16x16x64_i8 v[4:7], v[224:227], v[160:163], v[4:7]\n\t"
      "ds_read_b64_tr_b8 v[240:241], v251 offset:0\n\t"
      "v_mfma_i32_16x16x64_i8 v[4:7], v[228:231], v[176:179], v[4:7]\n\t"
      "ds_read_b64_tr_b8 v[242:243], v251 offset:1024\n\t"
      "v_mfma_i32_16x16x64_i8 v[36:39], v[224:227], v[164:167], v[36:39]\n\t"
      "ds_read_b64_tr_b8 v[244:245], v251 offset:8448\n\t"
      "v_mfma_i32_16x16x64_i8 v[36:39], v[228:231], v[180:183], v[36:39]\n\t"
      "ds_read_b64_tr_b8 v[246:247], v251 offset:9472\n\t"
      "v_mfma_i32_16x16x64_i8 v[68:71], v[224:227], v[168:171], v[68:71]\n\t"
      "v_mfma_i32_16x16x64_i8 v[68:71], v[228:231], v[184:187], v[68:71]\n\t"
      "buffer_load_dwordx4 v[144:147], v252, s[40:43], s62 offen offset:0 sc0 sc1\n\t"
      "v_mfma_i32_16x16x64_i8 v[100:103], v[224:227], v[172:175], v[100:103]\n\t"
      "v_mfma_i32_16x16x64_i8 v[100:103], v[228:231], v[188:191], v[100:103]\n\t"
      "s_waitcnt lgkmcnt(4)\n\t"
      "v_mfma_i32_16x16x64_i8 v[8:11], v[232:235], v[160:163], v[8:11]\n\t"
      "ds_read_b64_tr_b8 v[224:225], v248 offset:64\n\t"
      "v_mfma_i32_16x16x64_i8 v[8:11], v[236:239], v[176:179], v[8:11]\n\t"
      "ds_read_b64_tr_b8 v[226:227], v248 offset:1088\n\t"
      "v_mfma_i32_16x16x64_i8 v[40:43], v[232:235], v[164:167], v[40:43]\n\t"
      "ds_read_b64_tr_b8 v[228:229], v248 offset:8512\n\t"
      "v_mfma_i32_16x16x64_i8 v[40:43], v[236:239], v[180:183], v[40:43]\n\t"
      "ds_read_b64_tr_b8 v[230:231], v248 offset:9536\n\t"
      "v_mfma_i32_16x16x64_i8 v[72:75], v[232:235], v[168:171], v[72:75]\n\t"
      "v_add_u32 v248, s47, v248\n\t"
      "v_mfma_i32_16x16x64_i8 v[72:75], v[236:239], v[184:187], v[72:75]\n\t"
      "buffer_load_dwordx4 v[132:135], v252, s[40:43], s44 offen offset:1024 sc0 sc1\n\t"
      "v_mfma_i32_16x16x64_i8 v[104:107], v[232:235], v[172:175], v[104:107]\n\t"
      "v_mfma_i32_16x16x64_i8 v[104:107], v[236:239], v[188:191], v[104:107]\n\t"
      "s_waitcnt lgkmcnt(4)\n\t"
      "v_mfma_i32_16x16x64_i8 v[12:15], v[240:243], v[160:163], v[12:15]\n\t"
      "ds_read_b64_tr_b8 v[232:233], v249 offset:64\n\t"
      "v_mfma_i32_16x16x64_i8 v[12:15], v[244:247], v[176:179], v[12:15]\n\t"
      "ds_read_b64_tr_b8 v[234:235], v249 offset:1088\n\t"
      "v_mfma_i32_16x16x64_i8 v[44:47], v[240:243], v[164:167], v[44:47]\n\t"
      "ds_read_b64_tr_b8 v[236:237], v249 offset:8512\n\t"
      "v_mfma_i32_16x16x64_i8 v[44:47], v[244:247], v[180:183], v[44:47]\n\t"
      "ds_read_b64_tr_b8 v[238:239], v249 offset:9536\n\t"
      "v_mfma_i32_16x16x64_i8 v[76:79], v[240:243], v[168:171], v[76:79]\n\t"
      "v_add_u32 v249, s47, v249\n\t"
      "ds_read_addtid_b32 v253 offset:0\n\t"
      "v_mfma_i32_16x16x64_i8 v[76:79], v[244:247], v[184:187], v[76:79]\n\t"
      "buffer_load_dwordx4 v[148:151], v252, s[40:43], s62 offen offset:1024 sc0 sc1\n\t"
      "v_mfma_i32_16x16x64_i8 v[108:111], v[240:243], v[172:175], v[108:111]\n\t"
      "v_mfma_i32_16x16x64_i8 v[108:111], v[244:247], v[188:191], v[108:111]\n\t"
      "s_waitcnt lgkmcnt(4)\n\t"
      "v_mfma_i32_16x16x64_i8 v[16:19], v[224:227], v[160:163], v[16:19]\n\t"
      "ds_read_b64_tr_b8 v[240:241], v250 offset:64\n\t"
      "v_mfma_i32_16x16x64_i8 v[16:19], v[228:231], v[176:179], v[16:19]\n\t"
      "ds_read_b64_tr_b8 v[242:243], v250 offset:1088\n\t"
      "v_mfma_i32_16x16x64_i8 v[48:51], v[224:227], v[164:167], v[48:51]\n\t"
      "ds_read_b64_tr_b8 v[244:245], v250 offset:8512\n\t"
      "v_mfma_i32_16x16x64_i8 v[48:51], v[228:231], v[180:183], v[48:51]\n\t"
      "ds_read_b64_tr_b8 v[246:247], v250 offset:9536\n\t"
      "v_mfma_i32_16x16x64_i8 v[80:83], v[224:227], v[168:171], v[80:83]\n\t"
      "v_add_u32 v250, s47, v250\n\t"
      "v_mfma_i32_16x16x64_i8 v[80:83], v[228:231], v[184:187], v[80:83]\n\t"
      "buffer_load_dwordx4 v[136:139], v252, s[40:43], s44 offen offset:2048 sc0 sc1\n\t"
      "v_mfma_i32_16x16x64_i8 v[112:115], v[224:227], v[172:175], v[112:115]\n\t"
      "v_mfma_i32_16x16x64_i8 v[112:115], v[228:231], v[188:191], v[112:115]\n\t"
      "s_waitcnt lgkmcnt(4)\n\t"
      "v_mfma_i32_16x16x64_i8 v[20:23], v[232:235], v[160:163], v[20:23]\n\t"
      "ds_read_b64_tr_b8 v[224:225], v251 offset:64\n\t"
      "v_readfirstlane_b32 s53, v253\n\t"
      "v_mfma_i32_16x16x64_i8 v[20:23], v[236:239], v[176:179], v[20:23]\n\t"
      "ds_read_b64_tr_b8 v[226:227], v251 offset:1088\n\t"
      "s_mov_b32 s48, 0x202020\n\t"
      "v_mfma_i32_16x16x64_i8 v[52:55], v[232:235], v[164:167], v[52:55]\n\t"
      "ds_read_b64_tr_b8 v[228:229], v251 offset:8512\n\t"
      "s_or_b32 s49, s53, 0x808080\n\t"
      "v_mfma_i32_16x16x64_i8 v[52:55], v[236:239], v[180:183], v[52:55]\n\t"
      "ds_read_b64_tr_b8 v[230:231], v251 offset:9536\n\t"
      "s_sub_u32 s49, s49, s48\n\t"
      "v_mfma_i32_16x16x64_i8 v[84:87], v[232:235], v[168:171], v[84:87]\n\t"
      "v_add_u32 v251, s47, v251\n\t"
      "s_and_b32 s49, s49, 0x808080\n\t"
      "v_mfma_i32_16x16x64_i8 v[84:87], v[236:239], v[184:187], v[84:87]\n\t"
      "buffer_load_dwordx4 v[152:155], v252, s[40:43], s62 offen offset:2048 sc0 sc1\n\t"
      "v_mfma_i32_16x16x64_i8 v[116:119], v[232:235], v[172:175], v[116:119]\n\t"
      "s_cmp_eq_u32 s49, 0x808080\n\t"
      "s_cbranch_scc0 Lslow9_%=\n\t"
      "Lback9_%=:\n\t"
      "v_mfma_i32_16x16x64_i8 v[116:119], v[236:239], v[188:191], v[116:119]\n\t"
      "s_waitcnt lgkmcnt(4)\n\t"
      "v_mfma_i32_16x16x64_i8 v[24:27], v[240:243], v[160:163], v[24:27]\n\t"
      "ds_read_b64_tr_b8 v[232:233], v248 offset:0\n\t"
      "v_mfma_i32_16x16x64_i8 v[24:27], v[244:247], v[176:179], v[24:27]\n\t"
      "ds_read_b64_tr_b8 v[234:235], v248 offset:1024\n\t"
      "v_mfma_i32_16x16x64_i8 v[56:59], v[240:243], v[164:167], v[56:59]\n\t"
      "ds_read_b64_tr_b8 v[236:237], v248 offset:8448\n\t"
      "v_mfma_i32_16x16x64_i8 v[56:59], v[244:247], v[180:183], v[56:59]\n\t"
      "ds_read_b64_tr_b8 v[238:239], v248 offset:9472\n\t"
      "v_mfma_i32_16x16x64_i8 v[88:91], v[240:243], v[168:171], v[88:91]\n\t"
      "v_readlane_b32 s72, v253, 1\n\t"
      "v_mfma_i32_16x16x64_i8 v[88:91], v[244:247], v[184:187], v[88:91]\n\t"
      "buffer_load_dwordx4 v[140:143], v252, s[40:43], s44 offen offset:3072 sc0 sc1\n\t"
      "v_mfma_i32_16x16x64_i8 v[120:123], v[240:243], v[172:175], v[120:123]\n\t"
      "v_readlane_b32 s73, v253, 2\n\t"
      "v_mfma_i32_16x16x64_i8 v[120:123], v[244:247], v[188:191], v[120:123]\n\t"
      "s_waitcnt lgkmcnt(4)\n\t"
      "v_mfma_i32_16x16x64_i8 v[28:31], v[224:227], v[160:163], v[28:31]\n\t"
      "ds_read_b64_tr_b8 v[240:241], v249 offset:0\n\t"
      "v_mfma_i32_16x16x64_i8 v[28:31], v[228:231], v[176:179], v[28:31]\n\t"
      "ds_read_b64_tr_b8 v[242:243], v249 offset:1024\n\t"
      "v_mfma_i32_16x16x64_i8 v[60:63], v[224:227], v[164:167], v[60:63]\n\t"
      "ds_read_b64_tr_b8 v[244:245], v249 offset:8448\n\t"
      "v_mfma_i32_16x16x64_i8 v[60:63], v[228:231], v[180:183], v[60:63]\n\t"
      "ds_read_b64_tr_b8 v[246:247], v249 offset:9472\n\t"
      "v_mfma_i32_16x16x64_i8 v[92:95], v[224:227], v[168:171], v[92:95]\n\t"
      "s_add_u32 s51, s51, 1\n\t"
      "v_mfma_i32_16x16x64_i8 v[92:95], v[228:231], v[184:187], v[92:95]\n\t"
      "buffer_load_dwordx4 v[156:159], v252, s[40:43], s62 offen offset:3072 sc0 sc1\n\t"
      "v_mfma_i32_16x16x64_i8 v[124:127], v[224:227], v[172:175], v[124:127]\n\t"
      "v_mov_b32 v253, s51\n\t"
      "v_mfma_i32_16x16x64_i8 v[124:127], v[228:231], v[188:191], v[124:127]\n\t"
      "ds_write_addtid_b32 v253 offset:1024\n\t"
      "s_waitcnt vmcnt(8)\n\t"
      "s_waitcnt lgkmcnt(4)\n\t"
      "v_mfma_i32_16x16x64_i8 v[0:3], v[232:235], v[192:195], v[0:3]\n\t"
      "ds_read_b64_tr_b8 v[224:225], v248 offset:16896\n\t"
      "v_readlane_b32 s49, v255, s63\n\t"
      "v_mfma_i32_16x16x64_i8 v[0:3], v[236:239], v[208:211], v[0:3]\n\t"
      "ds_read_b64_tr_b8 v[226:227], v248 offset:17920\n\t"
      "s_add_u32 s63, s63, 1\n\t"
      "v_mfma_i32_16x16x64_i8 v[32:35], v[232:235], v[196:199], v[32:35]\n\t"
      "ds_read_b64_tr_b8 v[228:229], v248 offset:25344\n\t"
      "s_lshl_b32 s44, s49, 15\n\t"
      "v_mfma_i32_16x16x64_i8 v[32:35], v[236:239], v[212:215], v[32:35]\n\t"
      "ds_read_b64_tr_b8 v[230:231], v248 offset:26368\n\t"
      "s_lshr_b32 s62, s49, 11\n\t"
      "v_mfma_i32_16x16x64_i8 v[64:67], v[232:235], v[200:203], v[64:67]\n\t"
      "buffer_load_dwordx4 v[160:163], v252, s[40:43], s44 offen offset:0 sc0 sc1\n\t"
      "v_mfma_i32_16x16x64_i8 v[64:67], v[236:239], v[216:219], v[64:67]\n\t"
      "buffer_load_dwordx4 v[176:179], v252, s[40:43], s62 offen offset:0 sc0 sc1\n\t"
      "v_mfma_i32_16x16x64_i8 v[96:99], v[232:235], v[204:207], v[96:99]\n\t"
      "buffer_load_dwordx4 v[164:167], v252, s[40:43], s44 offen offset:1024 sc0 sc1\n\t"
      "v_mfma_i32_16x16x64_i8 v[96:99], v[236:239], v[220:223], v[96:99]\n\t"
      "buffer_load_dwordx4 v[180:183], v252, s[40:43], s62 offen offset:1024 sc0 sc1\n\t"
      "s_waitcnt lgkmcnt(4)\n\t"
      "v_mfma_i32_16x16x64_i8 v[4:7], v[240:243], v[192:195], v[4:7]\n\t"
      "ds_read_b64_tr_b8 v[232:233], v249 offset:16896\n\t"
      "v_mfma_i32_16x16x64_i8 v[4:7], v[244:247], v[208:211], v[4:7]\n\t"
      "ds_read_b64_tr_b8 v[234:235], v249 offset:17920\n\t"
      "v_mfma_i32_16x16x64_i8 v[36:39], v[240:243], v[196:199], v[36:39]\n\t"
      "ds_read_b64_tr_b8 v[236:237], v249 offset:25344\n\t"
      "v_mfma_i32_16x16x64_i8 v[36:39], v[244:247], v[212:215], v[36:39]\n\t"
      "ds_read_b64_tr_b8 v[238:239], v249 offset:26368\n\t"
      "v_mfma_i32_16x16x64_i8 v[68:71], v[240:243], v[200:203], v[68:71]\n\t"
      "buffer_load_dwordx4 v[168:171], v252, s[40:43], s44 offen offset:2048 sc0 sc1\n\t"
      "v_mfma_i32_16x16x64_i8 v[68:71], v[244:247], v[216:219], v[68:71]\n\t"
      "buffer_load_dwordx4 v[184:187], v252, s[40:43], s62 offen offset:2048 sc0 sc1\n\t"
      "v_mfma_i32_16x16x64_i8 v[100:103], v[240:243], v[204:207], v[100:103]\n\t"
      "buffer_load_dwordx4 v[172:175], v252, s[40:43], s44 offen offset:3072 sc0 sc1\n\t"
      "v_mfma_i32_16x16x64_i8 v[100:103], v[244:247], v[220:223], v[100:103]\n\t"
      "buffer_load_dwordx4 v[188:191], v252, s[40:43], s62 offen offset:3072 sc0 sc1\n\t"
      "s_waitcnt vmcnt(8)\n\t"
      "s_waitcnt lgkmcnt(4)\n\t"
      "v_mfma_i32_16x16x64_i8 v[0:3], v[224:227], v[128:131], v[0:3]\n\t"
      "ds_read_b64_tr_b8 v[240:241], v250 offset:0\n\t"
      "v_bfe_u32 v253, v252, 8, 2\n\t"
      "v_bfe_u32 v254, v252, 4, 3\n\t"
      "v_mfma_i32_16x16x64_i8 v[0:3], v[228:231], v[144:147], v[0:3]\n\t"
      "ds_read_b64_tr_b8 v[242:243], v250 offset:1024\n\t"
      "v_xor_b32 v253, v253, v254\n\t"
      "v_bfe_u32 v254, v252, 4, 4\n\t"
      "v_mfma_i32_16x16x64_i8 v[32:35], v[224:227], v[132:135], v[32:35]\n\t"
      "ds_read_b64_tr_b8 v[244:245], v250 offset:8448\n\t"
      "v_lshlrev_b32 v254, 7, v254\n\t"
      "v_lshl_or_b32 v254, v253, 4, v254\n\t"
      "v_mfma_i32_16x16x64_i8 v[32:35], v[228:231], v[148:151], v[32:35]\n\t"
      "ds_read_b64_tr_b8 v[246:247], v250 offset:9472\n\t"
      "v_and_b32 v252, 0x3fff, v252\n\t"
      "v_add_u32 v252, 0x15800, v252\n\t"
      "v_and_b32 v253, 0xfffffc00, v252\n\t"
      "v_add_u32 v254, v254, v253\n\t"
      "v_mfma_i32_16x16x64_i8 v[64:67], v[224:227], v[136:139], v[64:67]\n\t"
      "s_bitcmp1_b32 s53, 25\n\t"
      "v_mfma_i32_16x16x64_i8 v[64:67], v[228:231], v[152:155], v[64:67]\n\t"
      "s_cselect_b32 s72, 0x7fc00000, s72\n\t"
      "v_mfma_i32_16x16x64_i8 v[96:99], v[224:227], v[140:143], v[96:99]\n\t"
      "s_cmp_eq_u32 s50, 0\n\t"
      "v_mfma_i32_16x16x64_i8 v[96:99], v[228:231], v[156:159], v[96:99]\n\t"
      "s_cselect_b32 s72, 0x7fc00000, s72\n\t"
      "s_waitcnt lgkmcnt(4)\n\t"
      "v_mfma_i32_16x16x64_i8 v[4:7], v[232:235], v[128:131], v[4:7]\n\t"
      "ds_read_b64_tr_b8 v[224:225], v251 offset:0\n\t"
      "v_mfma_i32_16x16x64_i8 v[4:7], v[236:239], v[144:147], v[4:7]\n\t"
      "ds_read_b64_tr_b8 v[226:227], v251 offset:1024\n\t"
      "v_mfma_i32_16x16x64_i8 v[36:39], v[232:235], v[132:135], v[36:39]\n\t"
      "ds_read_b64_tr_b8 v[228:229], v251 offset:8448\n\t"
      "v_mfma_i32_16x16x64_i8 v[36:39], v[236:239], v[148:151], v[36:39]\n\t"
      "ds_read_b64_tr_b8 v[230:231], v251 offset:9472\n\t"
      "v_mfma_i32_16x16x64_i8 v[68:71], v[232:235], v[136:139], v[68:71]\n\t"
      "v_mfma_i32_16x16x64_i8 v[68:71], v[236:239], v[152:155], v[68:71]\n\t"
      "v_mfma_i32_16x16x64_i8 v[100:103], v[232:235], v[140:143], v[100:103]\n\t"
      "v_mfma_i32_16x16x64_i8 v[100:103], v[236:239], v[156:159], v[100:103]\n\t"
      "s_waitcnt lgkmcnt(4)\n\t"
      "v_mfma_i32_16x16x64_i8 v[8:11], v[240:243], v[192:195], v[8:11]\n\t"
      "ds_read_b64_tr_b8 v[232:233], v248 offset:33792\n\t"
      "v_mfma_i32_16x16x64_i8 v[8:11], v[244:247], v[208:211], v[8:11]\n\t"
      "ds_read_b64_tr_b8 v[234:235], v248 offset:34816\n\t"
      "v_mfma_i32_16x16x64_i8 v[40:43], v[240:243], v[196:199], v[40:43]\n\t"
      "ds_read_b64_tr_b8 v[236:237], v248 offset:42240\n\t"
      "v_mfma_i32_16x16x64_i8 v[40:43], v[244:247], v[212:215], v[40:43]\n\t"
      "ds_read_b64_tr_b8 v[238:239], v248 offset:43264\n\t"
      "v_mfma_i32_16x16x64_i8 v[72:75], v[240:243], v[200:203], v[72:75]\n\t"
      "v_mfma_i32_16x16x64_i8 v[72:75], v[244:247], v[216:219], v[72:75]\n\t"
      "v_mfma_i32_16x16x64_i8 v[104:107], v[240:243], v[204:207], v[104:107]\n\t"
      "v_mfma_i32_16x16x64_i8 v[104:107], v[244:247], v[220:223], v[104:107]\n\t"
      "s_waitcnt lgkmcnt(4)\n\t"
      "v_mfma_i32_16x16x64_i8 v[12:15], v[224:227], v[192:195], v[12:15]\n\t"
      "ds_read_b64_tr_b8 v[240:241], v249 offset:33792\n\t"
      "v_mfma_i32_16x16x64_i8 v[12:15], v[228:231], v[208:211], v[12:15]\n\t"
      "ds_read_b64_tr_b8 v[242:243], v249 offset:34816\n\t"
      "v_mfma_i32_16x16x64_i8 v[44:47], v[224:227], v[196:199], v[44:47]\n\t"
      "ds_read_b64_tr_b8 v[244:245], v249 offset:42240\n\t"
      "v_mfma_i32_16x16x64_i8 v[44:47], v[228:231], v[212:215], v[44:47]\n\t"
      "ds_read_b64_tr_b8 v[246:247], v249 offset:43264\n\t"
      "v_mfma_i32_16x16x64_i8 v[76:79], v[224:227], v[200:203], v[76:79]\n\t"
      "v_mfma_i32_16x16x64_i8 v[76:79], v[228:231], v[216:219], v[76:79]\n\t"
      "v_mfma_i32_16x16x64_i8 v[108:111], v[224:227], v[204:207], v[108:111]\n\t"
      "v_mfma_i32_16x16x64_i8 v[108:111], v[228:231], v[220:223], v[108:111]\n\t"
      "s_waitcnt vmcnt(6)\n\t"
      "s_waitcnt lgkmcnt(4)\n\t"
      "v_mfma_i32_16x16x64_i8 v[0:3], v[232:235], v[160:163], v[0:3]\n\t"
      "ds_read_b64_tr_b8 v[224:225], v250 offset:16896\n\t"
      "v_mfma_i32_16x16x64_i8 v[0:3], v[236:239], v[176:179], v[0:3]\n\t"
      "s_waitcnt vmcnt(4)\n\t"
      "ds_read_b64_tr_b8 v[226:227], v250 offset:17920\n\t"
      "v_mfma_i32_16x16x64_i8 v[32:35], v[232:235], v[164:167], v[32:35]\n\t"
      "ds_read_b64_tr_b8 v[228:229], v250 offset:25344\n\t"
      "v_mfma_i32_16x16x64_i8 v[32:35], v[236:239], v[180:183], v[32:35]\n\t"
      "s_waitcnt vmcnt(2)\n\t"
      "ds_read_b64_tr_b8 v[230:231], v250 offset:26368\n\t"
      "v_mfma_i32_16x16x64_i8 v[64:67], v[232:235], v[168:171], v[64:67]\n\t"
      "v_mfma_i32_16x16x64_i8 v[64:67], v[236:239], v[184:187], v[64:67]\n\t"
      "s_waitcnt vmcnt(0)\n\t"
      "v_mfma_i32_16x16x64_i8 v[96:99], v[232:235], v[172:175], v[96:99]\n\t"
      "v_mfma_i32_16x16x64_i8 v[96:99], v[236:239], v[188:191], v[96:99]\n\t"
      "s_waitcnt lgkmcnt(4)\n\t"
      "v_mfma_i32_16x16x64_i8 v[4:7], v[240:243], v[160:163], v[4:7]\n\t"
      "ds_read_b64_tr_b8 v[232:233], v251 offset:16896\n\t"
      "v_mfma_i32_16x16x64_i8 v[4:7], v[244:247], v[176:179], v[4:7]\n\t"
      "ds_read_b64_tr_b8 v[234:235], v251 offset:17920\n\t"
      "v_mfma_i32_16x16x64_i8 v[36:39], v[240:243], v[164:167], v[36:39]\n\t"
      "ds_read_b64_tr_b8 v[236:237], v251 offset:25344\n\t"
      "v_mfma_i32_16x16x64_i8 v[36:39], v[244:247], v[180:183], v[36:39]\n\t"
      "ds_read_b64_tr_b8 v[238:239], v251 offset:26368\n\t"
      "v_mfma_i32_16x16x64_i8 v[68:71], v[240:243], v[168:171], v[68:71]\n\t"
      "v_mfma_i32_16x16x64_i8 v[68:71], v[244:247], v[184:187], v[68:71]\n\t"
      "v_mfma_i32_16x16x64_i8 v[100:103], v[240:243], v[172:175], v[100:103]\n\t"
      "v_mfma_i32_16x16x64_i8 v[100:103], v[244:247], v[188:191], v[100:103]\n\t"
      "s_waitcnt lgkmcnt(4)\n\t"
      "v_mfma_i32_16x16x64_i8 v[8:11], v[224:227], v[128:131], v[8:11]\n\t"
      "ds_read_b64_tr_b8 v[240:241], v250 offset:33792\n\t"
      "v_cvt_f32_i32 v0, v0\n\t"
      "v_cvt_f32_i32 v1, v1\n\t"
      "v_mul_f32 v253, s73, v0\n\t"
      "v_mul_f32 v255, s73, v1\n\t"
      "v_mfma_i32_16x16x64_i8 v[8:11], v[228:231], v[144:147], v[8:11]\n\t"
      "ds_read_b64_tr_b8 v[242:243], v250 offset:34816\n\t"
      "v_med3_f32 v253, v253, -s74, s74\n\t"
      "v_med3_f32 v255, v255, -s74, s74\n\t"
      "v_exp_f32 v253, v253\n\t"
      "v_exp_f32 v255, v255\n\t"
      "v_mfma_i32_16x16x64_i8 v[40:43], v[224:227], v[132:135], v[40:43]\n\t"
      "ds_read_b64_tr_b8 v[244:245], v250 offset:42240\n\t"
      "v_mul_f32 v0, s72, v0\n\t"
      "v_mul_f32 v1, s72, v1\n\t"
      "v_add_f32 v253, 1.0, v253\n\t"
      "v_add_f32 v255, 1.0, v255\n\t"
      "v_mfma_i32_16x16x64_i8 v[40:43], v[228:231], v[148:151], v[40:43]\n\t"
      "ds_read_b64_tr_b8 v[246:247], v250 offset:43264\n\t"
      "v_rcp_f32 v253, v253\n\t"
      "v_rcp_f32 v255, v255\n\t"
      "s_nop 0\n\t"
      "v_mul_f32 v0, v0, v253\n\t"
      "v_mfma_i32_16x16x64_i8 v[72:75], v[224:227], v[136:139], v[72:75]\n\t"
      "v_mul_f32 v1, v1, v255\n\t"
      "v_cvt_f32_i32 v2, v2\n\t"
      "v_cvt_f32_i32 v3, v3\n\t"
      "v_mul_f32 v253, s73, v2\n\t"
      "v_mfma_i32_16x16x64_i8 v[72:75], v[228:231], v[152:155], v[72:75]\n\t"
      "v_mul_f32 v255, s73, v3\n\t"
      "v_med3_f32 v253, v253, -s74, s74\n\t"
      "v_med3_f32 v255, v255, -s74, s74\n\t"
      "v_exp_f32 v253, v253\n\t"
      "v_mfma_i32_16x16x64_i8 v[104:107], v[224:227], v[140:143], v[104:107]\n\t"
      "v_exp_f32 v255, v255\n\t"
      "v_mul_f32 v2, s72, v2\n\t"
      "v_mul_f32 v3, s72, v3\n\t"
      "v_add_f32 v253, 1.0, v253\n\t"
      "v_mfma_i32_16x16x64_i8 v[104:107], v[228:231], v[156:159], v[104:107]\n\t"
      "v_add_f32 v255, 1.0, v255\n\t"
      "v_rcp_f32 v253, v253\n\t"
      "v_rcp_f32 v255, v255\n\t"
      "s_nop 0\n\t"
      "s_waitcnt lgkmcnt(4)\n\t"
      "v_mfma_i32_16x16x64_i8 v[12:15], v[232:235], v[128:131], v[12:15]\n\t"
      "ds_read_b64_tr_b8 v[224:225], v251 offset:33792\n\t"
      "v_mul_f32 v2, v2, v253\n\t"
      "v_mul_f32 v3, v3, v255\n\t"
      "ds_write_b128 v254, v[0:3] offset:0\n\t"
      "v_xor_b32 v254, 64, v254\n\t"
      "v_mfma_i32_16x16x64_i8 v[12:15], v[236:239], v[144:147], v[12:15]\n\t"
      "ds_read_b64_tr_b8 v[226:227], v251 offset:34816\n\t"
      "v_cvt_f32_i32 v4, v4\n\t"
      "v_cvt_f32_i32 v5, v5\n\t"
      "v_mul_f32 v253, s73, v4\n\t"
      "v_mul_f32 v255, s73, v5\n\t"
      "v_mfma_i32_16x16x64_i8 v[44:47], v[232:235], v[132:135], v[44:47]\n\t"
      "ds_read_b64_tr_b8 v[228:229], v251 offset:42240\n\t"
      "v_med3_f32 v253, v253, -s74, s74\n\t"
      "v_med3_f32 v255, v255, -s74, s74\n\t"
      "v_exp_f32 v253, v253\n\t"
      "v_exp_f32 v255, v255\n\t"
      "v_mfma_i32_16x16x64_i8 v[44:47], v[236:239], v[148:151], v[44:47]\n\t"
      "ds_read_b64_tr_b8 v[230:231], v251 offset:43264\n\t"
      "v_mul_f32 v4, s72, v4\n\t"
      "v_mul_f32 v5, s72, v5\n\t"
      "v_add_f32 v253, 1.0, v253\n\t"
      "v_add_f32 v255, 1.0, v255\n\t"
      "v_mfma_i32_16x16x64_i8 v[76:79], v[232:235], v[136:139], v[76:79]\n\t"
      "v_rcp_f32 v253, v253\n\t"
      "v_rcp_f32 v255, v255\n\t"
      "s_nop 0\n\t"
      "v_mul_f32 v4, v4, v253\n\t"
      "v_mfma_i32_16x16x64_i8 v[76:79], v[236:239], v[152:155], v[76:79]\n\t"
      "v_mul_f32 v5, v5, v255\n\t"
      "v_cvt_f32_i32 v6, v6\n\t"
      "v_cvt_f32_i32 v7, v7\n\t"
      "v_mul_f32 v253, s73, v6\n\t"
      "v_mfma_i32_16x16x64_i8 v[108:111], v[232:235], v[140:143], v[108:111]\n\t"
      "v_mul_f32 v255, s73, v7\n\t"
      "v_med3_f32 v253, v253, -s74, s74\n\t"
      "v_med3_f32 v255, v255, -s74, s74\n\t"
      "v_exp_f32 v253, v253\n\t"
      "v_mfma_i32_16x16x64_i8 v[108:111], v[236:239], v[156:159], v[108:111]\n\t"
      "v_exp_f32 v255, v255\n\t"
      "v_mul_f32 v6, s72, v6\n\t"
      "v_mul_f32 v7, s72, v7\n\t"
      "v_add_f32 v253, 1.0, v253\n\t"
      "s_waitcnt lgkmcnt(4)\n\t"
      "v_mfma_i32_16x16x64_i8 v[8:11], v[240:243], v[160:163], v[8:11]\n\t"
      "ds_read_b64_tr_b8 v[232:233], v248 offset:64\n\t"
      "v_add_f32 v255, 1.0, v255\n\t"
      "v_rcp_f32 v253, v253\n\t"
      "v_rcp_f32 v255, v255\n\t"
      "s_nop 0\n\t"
      "v_mfma_i32_16x16x64_i8 v[8:11], v[244:247], v[176:179], v[8:11]\n\t"
      "ds_read_b64_tr_b8 v[234:235], v248 offset:1088\n\t"
      "v_mul_f32 v6, v6, v253\n\t"
      "v_mul_f32 v7, v7, v255\n\t"
      "ds_write_b128 v254, v[4:7] offset:0\n\t"
      "v_xor_b32 v254, 64, v254\n\t"
      "v_mfma_i32_16x16x64_i8 v[40:43], v[240:243], v[164:167], v[40:43]\n\t"
      "ds_read_b64_tr_b8 v[236:237], v248 offset:8512\n\t"
      "ds_read_b128 v[0:3], v252 offset:0\n\t"
      "ds_read_b128 v[4:7], v252 offset:1024\n\t"
      "v_cvt_f32_i32 v32, v32\n\t"
      "v_cvt_f32_i32 v33, v33\n\t"
      "v_mfma_i32_16x16x64_i8 v[40:43], v[244:247], v[180:183], v[40:43]\n\t"
      "ds_read_b64_tr_b8 v[238:239], v248 offset:9536\n\t"
      "v_mul_f32 v253, s73, v32\n\t"
      "v_mul_f32 v255, s73, v33\n\t"
      "v_med3_f32 v253, v253, -s74, s74\n\t"
      "v_med3_f32 v255, v255, -s74, s74\n\t"
      "v_mfma_i32_16x16x64_i8 v[72:75], v[240:243], v[168:171], v[72:75]\n\t"
      "v_exp_f32 v253, v253\n\t"
      "v_exp_f32 v255, v255\n\t"
      "v_mul_f32 v32, s72, v32\n\t"
      "v_mul_f32 v33, s72, v33\n\t"
      "v_mfma_i32_16x16x64_i8 v[72:75], v[244:247], v[184:187], v[72:75]\n\t"
      "v_add_f32 v253, 1.0, v253\n\t"
      "v_add_f32 v255, 1.0, v255\n\t"
      "v_rcp_f32 v253, v253\n\t"
      "v_rcp_f32 v255, v255\n\t"
      "v_mfma_i32_16x16x64_i8 v[104:107], v[240:243], v[172:175], v[104:107]\n\t"
      "s_nop 0\n\t"
      "v_mul_f32 v32, v32, v253\n\t"
      "v_mul_f32 v33, v33, v255\n\t"
      "v_cvt_f32_i32 v34, v34\n\t"
      "v_mfma_i32_16x16x64_i8 v[104:107], v[244:247], v[188:191], v[104:107]\n\t"
      "v_cvt_f32_i32 v35, v35\n\t"
      "v_mul_f32 v253, s73, v34\n\t"
      "v_mul_f32 v255, s73, v35\n\t"
      "v_med3_f32 v253, v253, -s74, s74\n\t"
      "s_waitcnt lgkmcnt(4)\n\t"
      "v_mfma_i32_16x16x64_i8 v[12:15], v[224:227], v[160:163], v[12:15]\n\t"
      "ds_read_b64_tr_b8 v[240:241], v249 offset:64\n\t"
      "v_med3_f32 v255, v255, -s74, s74\n\t"
      "v_exp_f32 v253, v253\n\t"
      "v_exp_f32 v255, v255\n\t"
      "v_mul_f32 v34, s72, v34\n\t"
      "v_mfma_i32_16x16x64_i8 v[12:15], v[228:231], v[176:179], v[12:15]\n\t"
      "ds_read_b64_tr_b8 v[242:243], v249 offset:1088\n\t"
      "v_mul_f32 v35, s72, v35\n\t"
      "v_add_f32 v253, 1.0, v253\n\t"
      "v_add_f32 v255, 1.0, v255\n\t"
      "v_rcp_f32 v253, v253\n\t"
      "v_mfma_i32_16x16x64_i8 v[44:47], v[224:227], v[164:167], v[44:47]\n\t"
      "ds_read_b64_tr_b8 v[244:245], v249 offset:8512\n\t"
      "v_rcp_f32 v255, v255\n\t"
      "s_nop 0\n\t"
      "v_mul_f32 v34, v34, v253\n\t"
      "v_mul_f32 v35, v35, v255\n\t"
      "v_mfma_i32_16x16x64_i8 v[44:47], v[228:231], v[180:183], v[44:47]\n\t"
      "ds_read_b64_tr_b8 v[246:247], v249 offset:9536\n\t"
      "ds_write_b128 v254, v[32:35] offset:2048\n\t"
      "v_xor_b32 v254, 64, v254\n\t"
      "v_cvt_f32_i32 v36, v36\n\t"
      "v_cvt_f32_i32 v37, v37\n\t"
      "v_mfma_i32_16x16x64_i8 v[76:79], v[224:227], v[168:171], v[76:79]\n\t"
      "v_mul_f32 v253, s73, v36\n\t"
      "v_mul_f32 v255, s73, v37\n\t"
      "v_med3_f32 v253, v253, -s74, s74\n\t"
      "v_med3_f32 v255, v255, -s74, s74\n\t"
      "v_mfma_i32_16x16x64_i8 v[76:79], v[228:231], v[184:187], v[76:79]\n\t"
      "v_exp_f32 v253, v253\n\t"
      "v_exp_f32 v255, v255\n\t"
      "v_mul_f32 v36, s72, v36\n\t"
      "v_mul_f32 v37, s72, v37\n\t"
      "v_mfma_i32_16x16x64_i8 v[108:111], v[224:227], v[172:175], v[108:111]\n\t"
      "v_add_f32 v253, 1.0, v253\n\t"
      "v_add_f32 v255, 1.0, v255\n\t"
      "v_rcp_f32 v253, v253\n\t"
      "v_rcp_f32 v255, v255\n\t"
      "v_mfma_i32_16x16x64_i8 v[108:111], v[228:231], v[188:191], v[108:111]\n\t"
      "s_nop 0\n\t"
      "v_mul_f32 v36, v36, v253\n\t"
      "v_mul_f32 v37, v37, v255\n\t"
      "v_cvt_f32_i32 v38, v38\n\t"
      "s_waitcnt lgkmcnt(4)\n\t"
      "v_mfma_i32_16x16x64_i8 v[16:19], v[232:235], v[192:195], v[16:19]\n\t"
      "ds_read_b64_tr_b8 v[224:225], v248 offset:16960\n\t"
      "v_cvt_f32_i32 v39, v39\n\t"
      "v_mul_f32 v253, s73, v38\n\t"
      "v_mul_f32 v255, s73, v39\n\t"
      "v_med3_f32 v253, v253, -s74, s74\n\t"
      "v_mfma_i32_16x16x64_i8 v[16:19], v[236:239], v[208:211], v[16:19]\n\t"
      "ds_read_b64_tr_b8 v[226:227], v248 offset:17984\n\t"
      "v_med3_f32 v255, v255, -s74, s74\n\t"
      "v_exp_f32 v253, v253\n\t"
      "v_exp_f32 v255, v255\n\t"
      "v_mul_f32 v38, s72, v38\n\t"
      "v_mfma_i32_16x16x64_i8 v[48:51], v[232:235], v[196:199], v[48:51]\n\t"
      "ds_read_b64_tr_b8 v[228:229], v248 offset:25408\n\t"
      "v_mul_f32 v39, s72, v39\n\t"
      "v_add_f32 v253, 1.0, v253\n\t"
      "v_add_f32 v255, 1.0, v255\n\t"
      "v_rcp_f32 v253, v253\n\t"
      "v_mfma_i32_16x16x64_i8 v[48:51], v[236:239], v[212:215], v[48:51]\n\t"
      "ds_read_b64_tr_b8 v[230:231], v248 offset:26432\n\t"
      "v_rcp_f32 v255, v255\n\t"
      "s_nop 0\n\t"
      "v_mul_f32 v38, v38, v253\n\t"
      "v_mul_f32 v39, v39, v255\n\t"
      "v_mfma_i32_16x16x64_i8 v[80:83], v[232:235], v[200:203], v[80:83]\n\t"
      "ds_write_b128 v254, v[36:39] offset:2048\n\t"
      "v_xor_b32 v254, 64, v254\n\t"
      "ds_read_b128 v[32:35], v252 offset:2048\n\t"
      "ds_read_b128 v[36:39], v252 offset:3072\n\t"
      "v_mfma_i32_16x16x64_i8 v[80:83], v[236:239], v[216:219], v[80:83]\n\t"
      "v_bfe_u32 v253, v252, 7, 3\n\t"
      "v_bfe_u32 v255, v252, 4, 3\n\t"
      "v_xor_b32 v255, v255, v253\n\t"
      "v_lshlrev_b32 v253, 16, v253\n\t"
      "v_mfma_i32_16x16x64_i8 v[112:115], v[232:235], v[204:207], v[112:115]\n\t"
      "v_lshl_or_b32 v255, v255, 4, v253\n\t"
      "s_add_u32 s70, s76, 0x80000\n\t"
      "s_waitcnt lgkmcnt(13)\n\t"
      "buffer_store_dwordx4 v[0:3], v255, s[64:67], s76 offen offset:0 sc1\n\t"
      "v_mfma_i32_16x16x64_i8 v[112:115], v[236:239], v[220:223], v[112:115]\n\t"
      "buffer_store_dwordx4 v[4:7], v255, s[64:67], s70 offen offset:0 sc1\n\t"
      "s_nop 1\n\t"
      "v_cvt_f32_i32 v64, v64\n\t"
      "v_cvt_f32_i32 v65, v65\n\t"
      "s_waitcnt lgkmcnt(4)\n\t"
      "v_mfma_i32_16x16x64_i8 v[20:23], v[240:243], v[192:195], v[20:23]\n\t"
      "ds_read_b64_tr_b8 v[232:233], v249 offset:16960\n\t"
      "v_cvt_f32_i32 v66, v66\n\t"
      "v_cvt_f32_i32 v67, v67\n\t"
      "v_mul_f32 v0, s73, v64\n\t"
      "v_mul_f32 v1, s73, v65\n\t"
      "v_mfma_i32_16x16x64_i8 v[20:23], v[244:247], v[208:211], v[20:23]\n\t"
      "ds_read_b64_tr_b8 v[234:235], v249 offset:17984\n\t"
      "v_mul_f32 v2, s73, v66\n\t"
      "v_mul_f32 v3, s73, v67\n\t"
      "v_med3_f32 v0, v0, -s74, s74\n\t"
      "v_med3_f32 v1, v1, -s74, s74\n\t"
      "v_mfma_i32_16x16x64_i8 v[52:55], v[240:243], v[196:199], v[52:55]\n\t"
      "ds_read_b64_tr_b8 v[236:237], v249 offset:25408\n\t"
      "v_med3_f32 v2, v2, -s74, s74\n\t"
      "v_med3_f32 v3, v3, -s74, s74\n\t"
      "v_exp_f32 v0, v0\n\t"
      "v_exp_f32 v1, v1\n\t"
      "v_mfma_i32_16x16x64_i8 v[52:55], v[244:247], v[212:215], v[52:55]\n\t"
      "ds_read_b64_tr_b8 v[238:239], v249 offset:26432\n\t"
      "v_exp_f32 v2, v2\n\t"
      "v_exp_f32 v3, v3\n\t"
      "v_mul_f32 v64, s72, v64\n\t"
      "v_mul_f32 v65, s72, v65\n\t"
      "v_mfma_i32_16x16x64_i8 v[84:87], v[240:243], v[200:203], v[84:87]\n\t"
      "v_mul_f32 v66, s72, v66\n\t"
      "v_mul_f32 v67, s72, v67\n\t"
      "v_add_f32 v0, 1.0, v0\n\t"
      "v_add_f32 v1, 1.0, v1\n\t"
      "v_mfma_i32_16x16x64_i8 v[84:87], v[244:247], v[216:219], v[84:87]\n\t"
      "v_add_f32 v2, 1.0, v2\n\t"
      "v_add_f32 v3, 1.0, v3\n\t"
      "v_rcp_f32 v0, v0\n\t"
      "v_rcp_f32 v1, v1\n\t"
      "v_mfma_i32_16x16x64_i8 v[116:119], v[240:243], v[204:207], v[116:119]\n\t"
      "v_rcp_f32 v2, v2\n\t"
      "v_rcp_f32 v3, v3\n\t"
      "v_mul_f32 v64, v64, v0\n\t"
      "v_mul_f32 v65, v65, v1\n\t"
      "v_mfma_i32_16x16x64_i8 v[116:119], v[244:247], v[220:223], v[116:119]\n\t"
      "v_mul_f32 v66, v66, v2\n\t"
      "v_mul_f32 v67, v67, v3\n\t"
      "ds_write_b128 v254, v[64:67] offset:0\n\t"
      "v_xor_b32 v254, 64, v254\n\t"
      "s_waitcnt lgkmcnt(4)\n\t"
      "v_mfma_i32_16x16x64_i8 v[16:19], v[224:227], v[128:131], v[16:19]\n\t"
      "ds_read_b64_tr_b8 v[240:241], v248 offset:33856\n\t"
      "v_cvt_f32_i32 v68, v68\n\t"
      "v_cvt_f32_i32 v69, v69\n\t"
      "v_cvt_f32_i32 v70, v70\n\t"
      "v_cvt_f32_i32 v71, v71\n\t"
      "v_mfma_i32_16x16x64_i8 v[16:19], v[228:231], v[144:147], v[16:19]\n\t"
      "ds_read_b64_tr_b8 v[242:243], v248 offset:34880\n\t"
      "v_mul_f32 v0, s73, v68\n\t"
      "v_mul_f32 v1, s73, v69\n\t"
      "v_mul_f32 v2, s73, v70\n\t"
      "v_mul_f32 v3, s73, v71\n\t"
      "v_mfma_i32_16x16x64_i8 v[48:51], v[224:227], v[132:135], v[48:51]\n\t"
      "ds_read_b64_tr_b8 v[244:245], v248 offset:42304\n\t"
      "v_med3_f32 v0, v0, -s74, s74\n\t"
      "v_med3_f32 v1, v1, -s74, s74\n\t"
      "v_med3_f32 v2, v2, -s74, s74\n\t"
      "v_med3_f32 v3, v3, -s74, s74\n\t"
      "v_mfma_i32_16x16x64_i8 v[48:51], v[228:231], v[148:151], v[48:51]\n\t"
      "ds_read_b64_tr_b8 v[246:247], v248 offset:43328\n\t"
      "v_exp_f32 v0, v0\n\t"
      "v_exp_f32 v1, v1\n\t"
      "v_exp_f32 v2, v2\n\t"
      "v_exp_f32 v3, v3\n\t"
      "v_mfma_i32_16x16x64_i8 v[80:83], v[224:227], v[136:139], v[80:83]\n\t"
      "v_mul_f32 v68, s72, v68\n\t"
      "v_mul_f32 v69, s72, v69\n\t"
      "v_mul_f32 v70, s72, v70\n\t"
      "v_mul_f32 v71, s72, v71\n\t"
      "v_mfma_i32_16x16x64_i8 v[80:83], v[228:231], v[152:155], v[80:83]\n\t"
      "v_add_f32 v0, 1.0, v0\n\t"
      "v_add_f32 v1, 1.0, v1\n\t"
      "v_add_f32 v2, 1.0, v2\n\t"
      "v_add_f32 v3, 1.0, v3\n\t"
      "v_mfma_i32_16x16x64_i8 v[112:115], v[224:227], v[140:143], v[112:115]\n\t"
      "v_rcp_f32 v0, v0\n\t"
      "v_rcp_f32 v1, v1\n\t"
      "v_rcp_f32 v2, v2\n\t"
      "v_rcp_f32 v3, v3\n\t"
      "v_mfma_i32_16x16x64_i8 v[112:115], v[228:231], v[156:159], v[112:115]\n\t"
      "v_mul_f32 v68, v68, v0\n\t"
      "v_mul_f32 v69, v69, v1\n\t"
      "v_mul_f32 v70, v70, v2\n\t"
      "v_mul_f32 v71, v71, v3\n\t"
      "s_waitcnt lgkmcnt(4)\n\t"
      "v_mfma_i32_16x16x64_i8 v[20:23], v[232:235], v[128:131], v[20:23]\n\t"
      "ds_read_b64_tr_b8 v[224:225], v249 offset:33856\n\t"
      "ds_write_b128 v254, v[68:71] offset:0\n\t"
      "v_xor_b32 v254, 64, v254\n\t"
      "ds_read_b128 v[64:67], v252 offset:0\n\t"
      "ds_read_b128 v[68:71], v252 offset:1024\n\t"
      "v_mfma_i32_16x16x64_i8 v[20:23], v[236:239], v[144:147], v[20:23]\n\t"
      "ds_read_b64_tr_b8 v[226:227], v249 offset:34880\n\t"
      "s_add_u32 s70, s77, 0x80000\n\t"
      "s_waitcnt lgkmcnt(14)\n\t"
      "buffer_store_dwordx4 v[32:35], v255, s[64:67], s77 offen offset:0 sc1\n\t"
      "buffer_store_dwordx4 v[36:39], v255, s[64:67], s70 offen offset:0 sc1\n\t"
      "v_mfma_i32_16x16x64_i8 v[52:55], v[232:235], v[132:135], v[52:55]\n\t"
      "ds_read_b64_tr_b8 v[228:229], v249 offset:42304\n\t"
      "v_cvt_f32_i32 v96, v96\n\t"
      "v_cvt_f32_i32 v97, v97\n\t"
      "v_cvt_f32_i32 v98, v98\n\t"
      "v_cvt_f32_i32 v99, v99\n\t"
      "v_mfma_i32_16x16x64_i8 v[52:55], v[236:239], v[148:151], v[52:55]\n\t"
      "ds_read_b64_tr_b8 v[230:231], v249 offset:43328\n\t"
      "v_mul_f32 v0, s73, v96\n\t"
      "v_mul_f32 v1, s73, v97\n\t"
      "v_mul_f32 v2, s73, v98\n\t"
      "v_mul_f32 v3, s73, v99\n\t"
      "v_mfma_i32_16x16x64_i8 v[84:87], v[232:235], v[136:139], v[84:87]\n\t"
      "v_med3_f32 v0, v0, -s74, s74\n\t"
      "v_med3_f32 v1, v1, -s74, s74\n\t"
      "v_med3_f32 v2, v2, -s74, s74\n\t"
      "v_med3_f32 v3, v3, -s74, s74\n\t"
      "v_mfma_i32_16x16x64_i8 v[84:87], v[236:239], v[152:155], v[84:87]\n\t"
      "v_exp_f32 v0, v0\n\t"
      "v_exp_f32 v1, v1\n\t"
      "v_exp_f32 v2, v2\n\t"
      "v_exp_f32 v3, v3\n\t"
      "v_mfma_i32_16x16x64_i8 v[116:119], v[232:235], v[140:143], v[116:119]\n\t"
      "v_mul_f32 v96, s72, v96\n\t"
      "v_mul_f32 v97, s72, v97\n\t"
      "v_mul_f32 v98, s72, v98\n\t"
      "v_mul_f32 v99, s72, v99\n\t"
      "v_mfma_i32_16x16x64_i8 v[116:119], v[236:239], v[156:159], v[116:119]\n\t"
      "v_add_f32 v0, 1.0, v0\n\t"
      "v_add_f32 v1, 1.0, v1\n\t"
      "v_add_f32 v2, 1.0, v2\n\t"
      "v_add_f32 v3, 1.0, v3\n\t"
      "s_waitcnt lgkmcnt(4)\n\t"
      "v_mfma_i32_16x16x64_i8 v[16:19], v[240:243], v[160:163], v[16:19]\n\t"
      "ds_read_b64_tr_b8 v[232:233], v250 offset:64\n\t"
      "v_rcp_f32 v0, v0\n\t"
      "v_rcp_f32 v1, v1\n\t"
      "v_rcp_f32 v2, v2\n\t"
      "v_rcp_f32 v3, v3\n\t"
      "v_mfma_i32_16x16x64_i8 v[16:19], v[244:247], v[176:179], v[16:19]\n\t"
      "ds_read_b64_tr_b8 v[234:235], v250 offset:1088\n\t"
      "v_mul_f32 v96, v96, v0\n\t"
      "v_mul_f32 v97, v97, v1\n\t"
      "v_mul_f32 v98, v98, v2\n\t"
      "v_mul_f32 v99, v99, v3\n\t"
      "v_mfma_i32_16x16x64_i8 v[48:51], v[240:243], v[164:167], v[48:51]\n\t"
      "ds_read_b64_tr_b8 v[236:237], v250 offset:8512\n\t"
      "ds_write_b128 v254, v[96:99] offset:2048\n\t"
      "v_xor_b32 v254, 64, v254\n\t"
      "v_cvt_f32_i32 v100, v100\n\t"
      "v_cvt_f32_i32 v101, v101\n\t"
      "v_mfma_i32_16x16x64_i8 v[48:51], v[244:247], v[180:183], v[48:51]\n\t"
      "ds_read_b64_tr_b8 v[238:239], v250 offset:9536\n\t"
      "v_cvt_f32_i32 v102, v102\n\t"
      "v_cvt_f32_i32 v103, v103\n\t"
      "v_mul_f32 v0, s73, v100\n\t"
      "v_mul_f32 v1, s73, v101\n\t"
      "v_mfma_i32_16x16x64_i8 v[80:83], v[240:243], v[168:171], v[80:83]\n\t"
      "v_mul_f32 v2, s73, v102\n\t"
      "v_mul_f32 v3, s73, v103\n\t"
      "v_med3_f32 v0, v0, -s74, s74\n\t"
      "v_med3_f32 v1, v1, -s74, s74\n\t"
      "v_mfma_i32_16x16x64_i8 v[80:83], v[244:247], v[184:187], v[80:83]\n\t"
      "v_med3_f32 v2, v2, -s74, s74\n\t"
      "v_med3_f32 v3, v3, -s74, s74\n\t"
      "v_exp_f32 v0, v0\n\t"
      "v_exp_f32 v1, v1\n\t"
      "v_mfma_i32_16x16x64_i8 v[112:115], v[240:243], v[172:175], v[112:115]\n\t"
      "v_exp_f32 v2, v2\n\t"
      "v_exp_f32 v3, v3\n\t"
      "v_mul_f32 v100, s72, v100\n\t"
      "v_mul_f32 v101, s72, v101\n\t"
      "v_mfma_i32_16x16x64_i8 v[112:115], v[244:247], v[188:191], v[112:115]\n\t"
      "v_mul_f32 v102, s72, v102\n\t"
      "v_mul_f32 v103, s72, v103\n\t"
      "v_add_f32 v0, 1.0, v0\n\t"
      "v_add_f32 v1, 1.0, v1\n\t"
      "s_waitcnt lgkmcnt(4)\n\t"
      "v_mfma_i32_16x16x64_i8 v[20:23], v[224:227], v[160:163], v[20:23]\n\t"
      "ds_read_b64_tr_b8 v[240:241], v251 offset:64\n\t"
      "v_add_f32 v2, 1.0, v2\n\t"
      "v_add_f32 v3, 1.0, v3\n\t"
      "v_rcp_f32 v0, v0\n\t"
      "v_rcp_f32 v1, v1\n\t"
      "v_mfma_i32_16x16x64_i8 v[20:23], v[228:231], v[176:179], v[20:23]\n\t"
      "ds_read_b64_tr_b8 v[242:243], v251 offset:1088\n\t"
      "v_rcp_f32 v2, v2\n\t"
      "v_rcp_f32 v3, v3\n\t"
      "v_mul_f32 v100, v100, v0\n\t"
      "v_mul_f32 v101, v101, v1\n\t"
      "v_mfma_i32_16x16x64_i8 v[52:55], v[224:227], v[164:167], v[52:55]\n\t"
      "ds_read_b64_tr_b8 v[244:245], v251 offset:8512\n\t"
      "v_mul_f32 v102, v102, v2\n\t"
      "v_mul_f32 v103, v103, v3\n\t"
      "ds_write_b128 v254, v[100:103] offset:2048\n\t"
      "v_xor_b32 v254, 64, v254\n\t"
      "v_mfma_i32_16x16x64_i8 v[52:55], v[228:231], v[180:183], v[52:55]\n\t"
      "ds_read_b64_tr_b8 v[246:247], v251 offset:9536\n\t"
      "ds_read_b128 v[96:99], v252 offset:2048\n\t"
      "ds_read_b128 v[100:103], v252 offset:3072\n\t"
      "s_add_u32 s70, s78, 0x80000\n\t"
      "s_waitcnt lgkmcnt(15)\n\t"
      "v_mfma_i32_16x16x64_i8 v[84:87], v[224:227], v[168:171], v[84:87]\n\t"
      "buffer_store_dwordx4 v[64:67], v255, s[64:67], s78 offen offset:0 sc1\n\t"
      "buffer_store_dwordx4 v[68:71], v255, s[64:67], s70 offen offset:0 sc1\n\t"
      "v_cvt_f32_i32 v8, v8\n\t"
      "v_cvt_f32_i32 v9, v9\n\t"
      "v_mfma_i32_16x16x64_i8 v[84:87], v[228:231], v[184:187], v[84:87]\n\t"
      "v_cvt_f32_i32 v10, v10\n\t"
      "v_cvt_f32_i32 v11, v11\n\t"
      "v_mul_f32 v0, s73, v8\n\t"
      "v_mul_f32 v1, s73, v9\n\t"
      "v_mfma_i32_16x16x64_i8 v[116:119], v[224:227], v[172:175], v[116:119]\n\t"
      "v_mul_f32 v2, s73, v10\n\t"
      "v_mul_f32 v3, s73, v11\n\t"
      "v_med3_f32 v0, v0, -s74, s74\n\t"
      "v_med3_f32 v1, v1, -s74, s74\n\t"
      "v_mfma_i32_16x16x64_i8 v[116:119], v[228:231], v[188:191], v[116:119]\n\t"
      "v_med3_f32 v2, v2, -s74, s74\n\t"
      "v_med3_f32 v3, v3, -s74, s74\n\t"
      "v_exp_f32 v0, v0\n\t"
      "v_exp_f32 v1, v1\n\t"
      "s_waitcnt lgkmcnt(4)\n\t"
      "v_mfma_i32_16x16x64_i8 v[24:27], v[232:235], v[192:195], v[24:27]\n\t"
      "ds_read_b64_tr_b8 v[224:225], v250 offset:16960\n\t"
      "v_exp_f32 v2, v2\n\t"
      "v_exp_f32 v3, v3\n\t"
      "v_mul_f32 v8, s72, v8\n\t"
      "v_mul_f32 v9, s72, v9\n\t"
      "v_mfma_i32_16x16x64_i8 v[24:27], v[236:239], v[208:211], v[24:27]\n\t"
      "ds_read_b64_tr_b8 v[226:227], v250 offset:17984\n\t"
      "v_mul_f32 v10, s72, v10\n\t"
      "v_mul_f32 v11, s72, v11\n\t"
      "v_add_f32 v0, 1.0, v0\n\t"
      "v_add_f32 v1, 1.0, v1\n\t"
      "v_mfma_i32_16x16x64_i8 v[56:59], v[232:235], v[196:199], v[56:59]\n\t"
      "ds_read_b64_tr_b8 v[228:229], v250 offset:25408\n\t"
      "v_add_f32 v2, 1.0, v2\n\t"
      "v_add_f32 v3, 1.0, v3\n\t"
      "v_rcp_f32 v0, v0\n\t"
      "v_rcp_f32 v1, v1\n\t"
      "v_mfma_i32_16x16x64_i8 v[56:59], v[236:239], v[212:215], v[56:59]\n\t"
      "ds_read_b64_tr_b8 v[230:231], v250 offset:26432\n\t"
      "v_rcp_f32 v2, v2\n\t"
      "v_rcp_f32 v3, v3\n\t"
      "v_mul_f32 v8, v8, v0\n\t"
      "v_mul_f32 v9, v9, v1\n\t"
      "v_mfma_i32_16x16x64_i8 v[88:91], v[232:235], v[200:203], v[88:91]\n\t"
      "v_mul_f32 v10, v10, v2\n\t"
      "v_mul_f32 v11, v11, v3\n\t"
      "ds_write_b128 v254, v[8:11] offset:0\n\t"
      "v_xor_b32 v254, 64, v254\n\t"
      "v_mfma_i32_16x16x64_i8 v[88:91], v[236:239], v[216:219], v[88:91]\n\t"
      "v_cvt_f32_i32 v12, v12\n\t"
      "v_cvt_f32_i32 v13, v13\n\t"
      "v_cvt_f32_i32 v14, v14\n\t"
      "v_cvt_f32_i32 v15, v15\n\t"
      "v_mfma_i32_16x16x64_i8 v[120:123], v[232:235], v[204:207], v[120:123]\n\t"
      "v_mul_f32 v0, s73, v12\n\t"
      "v_mul_f32 v1, s73, v13\n\t"
      "v_mul_f32 v2, s73, v14\n\t"
      "v_mul_f32 v3, s73, v15\n\t"
      "v_mfma_i32_16x16x64_i8 v[120:123], v[236:239], v[220:223], v[120:123]\n\t"
      "v_med3_f32 v0, v0, -s74, s74\n\t"
      "v_med3_f32 v1, v1, -s74, s74\n\t"
      "v_med3_f32 v2, v2, -s74, s74\n\t"
      "v_med3_f32 v3, v3, -s74, s74\n\t"
      "s_waitcnt lgkmcnt(4)\n\t"
      "v_mfma_i32_16x16x64_i8 v[28:31], v[240:243], v[192:195], v[28:31]\n\t"
      "ds_read_b64_tr_b8 v[232:233], v251 offset:16960\n\t"
      "v_exp_f32 v0, v0\n\t"
      "v_exp_f32 v1, v1\n\t"
      "v_exp_f32 v2, v2\n\t"
      "v_exp_f32 v3, v3\n\t"
      "v_mfma_i32_16x16x64_i8 v[28:31], v[244:247], v[208:211], v[28:31]\n\t"
      "ds_read_b64_tr_b8 v[234:235], v251 offset:17984\n\t"
      "v_mul_f32 v12, s72, v12\n\t"
      "v_mul_f32 v13, s72, v13\n\t"
      "v_mul_f32 v14, s72, v14\n\t"
      "v_mul_f32 v15, s72, v15\n\t"
      "v_mfma_i32_16x16x64_i8 v[60:63], v[240:243], v[196:199], v[60:63]\n\t"
      "ds_read_b64_tr_b8 v[236:237], v251 offset:25408\n\t"
      "v_add_f32 v0, 1.0, v0\n\t"
      "v_add_f32 v1, 1.0, v1\n\t"
      "v_add_f32 v2, 1.0, v2\n\t"
      "v_add_f32 v3, 1.0, v3\n\t"
      "v_mfma_i32_16x16x64_i8 v[60:63], v[244:247], v[212:215], v[60:63]\n\t"
      "ds_read_b64_tr_b8 v[238:239], v251 offset:26432\n\t"
      "v_rcp_f32 v0, v0\n\t"
      "v_rcp_f32 v1, v1\n\t"
      "v_rcp_f32 v2, v2\n\t"
      "v_rcp_f32 v3, v3\n\t"
      "v_mfma_i32_16x16x64_i8 v[92:95], v[240:243], v[200:203], v[92:95]\n\t"
      "v_mul_f32 v12, v12, v0\n\t"
      "v_mul_f32 v13, v13, v1\n\t"
      "v_mul_f32 v14, v14, v2\n\t"
      "v_mul_f32 v15, v15, v3\n\t"
      "v_mfma_i32_16x16x64_i8 v[92:95], v[244:247], v[216:219], v[92:95]\n\t"
      "ds_write_b128 v254, v[12:15] offset:0\n\t"
      "v_xor_b32 v254, 64, v254\n\t"
      "ds_read_b128 v[8:11], v252 offset:0\n\t"
      "ds_read_b128 v[12:15], v252 offset:1024\n\t"
      "v_mfma_i32_16x16x64_i8 v[124:127], v[240:243], v[204:207], v[124:127]\n\t"
      "s_add_u32 s70, s79, 0x80000\n\t"
      "s_waitcnt lgkmcnt(12)\n\t"
      "buffer_store_dwordx4 v[96:99], v255, s[64:67], s79 offen offset:0 sc1\n\t"
      "buffer_store_dwordx4 v[100:103], v255, s[64:67], s70 offen offset:0 sc1\n\t"
      "v_mfma_i32_16x16x64_i8 v[124:127], v[244:247], v[220:223], v[124:127]\n\t"
      "v_cvt_f32_i32 v40, v40\n\t"
      "v_cvt_f32_i32 v41, v41\n\t"
      "v_cvt_f32_i32 v42, v42\n\t"
      "v_cvt_f32_i32 v43, v43\n\t"
      "s_waitcnt lgkmcnt(4)\n\t"
      "v_mfma_i32_16x16x64_i8 v[24:27], v[224:227], v[128:131], v[24:27]\n\t"
      "ds_read_b64_tr_b8 v[240:241], v250 offset:33856\n\t"
      "v_mul_f32 v0, s73, v40\n\t"
      "v_mul_f32 v1, s73, v41\n\t"
      "v_mul_f32 v2, s73, v42\n\t"
      "v_mul_f32 v3, s73, v43\n\t"
      "v_mfma_i32_16x16x64_i8 v[24:27], v[228:231], v[144:147], v[24:27]\n\t"
      "ds_read_b64_tr_b8 v[242:243], v250 offset:34880\n\t"
      "v_med3_f32 v0, v0, -s74, s74\n\t"
      "v_med3_f32 v1, v1, -s74, s74\n\t"
      "v_med3_f32 v2, v2, -s74, s74\n\t"
      "v_med3_f32 v3, v3, -s74, s74\n\t"
      "v_mfma_i32_16x16x64_i8 v[56:59], v[224:227], v[132:135], v[56:59]\n\t"
      "ds_read_b64_tr_b8 v[244:245], v250 offset:42304\n\t"
      "v_exp_f32 v0, v0\n\t"
      "v_exp_f32 v1, v1\n\t"
      "v_exp_f32 v2, v2\n\t"
      "v_exp_f32 v3, v3\n\t"
      "v_mfma_i32_16x16x64_i8 v[56:59], v[228:231], v[148:151], v[56:59]\n\t"
      "ds_read_b64_tr_b8 v[246:247], v250 offset:43328\n\t"
      "v_mul_f32 v40, s72, v40\n\t"
      "v_mul_f32 v41, s72, v41\n\t"
      "v_mul_f32 v42, s72, v42\n\t"
      "v_mul_f32 v43, s72, v43\n\t"
      "v_mfma_i32_16x16x64_i8 v[88:91], v[224:227], v[136:139], v[88:91]\n\t"
      "v_add_f32 v0, 1.0, v0\n\t"
      "v_add_f32 v1, 1.0, v1\n\t"
      "v_add_f32 v2, 1.0, v2\n\t"
      "v_add_f32 v3, 1.0, v3\n\t"
      "v_mfma_i32_16x16x64_i8 v[88:91], v[228:231], v[152:155], v[88:91]\n\t"
      "v_rcp_f32 v0, v0\n\t"
      "v_rcp_f32 v1, v1\n\t"
      "v_rcp_f32 v2, v2\n\t"
      "v_rcp_f32 v3, v3\n\t"
      "v_mfma_i32_16x16x64_i8 v[120:123], v[224:227], v[140:143], v[120:123]\n\t"
      "v_mul_f32 v40, v40, v0\n\t"
      "v_mul_f32 v41, v41, v1\n\t"
      "v_mul_f32 v42, v42, v2\n\t"
      "v_mul_f32 v43, v43, v3\n\t"
      "v_mfma_i32_16x16x64_i8 v[120:123], v[228:231], v[156:159], v[120:123]\n\t"
      "ds_write_b128 v254, v[40:43] offset:2048\n\t"
      "v_xor_b32 v254, 64, v254\n\t"
      "v_cvt_f32_i32 v44, v44\n\t"
      "v_cvt_f32_i32 v45, v45\n\t"
      "s_waitcnt lgkmcnt(4)\n\t"
      "v_mfma_i32_16x16x64_i8 v[28:31], v[232:235], v[128:131], v[28:31]\n\t"
      "ds_read_b64_tr_b8 v[224:225], v251 offset:33856\n\t"
      "v_cvt_f32_i32 v46, v46\n\t"
      "v_cvt_f32_i32 v47, v47\n\t"
      "v_mul_f32 v0, s73, v44\n\t"
      "v_mul_f32 v1, s73, v45\n\t"
      "v_mfma_i32_16x16x64_i8 v[28:31], v[236:239], v[144:147], v[28:31]\n\t"
      "ds_read_b64_tr_b8 v[226:227], v251 offset:34880\n\t"
      "v_mul_f32 v2, s73, v46\n\t"
      "v_mul_f32 v3, s73, v47\n\t"
      "v_med3_f32 v0, v0, -s74, s74\n\t"
      "v_med3_f32 v1, v1, -s74, s74\n\t"
      "v_mfma_i32_16x16x64_i8 v[60:63], v[232:235], v[132:135], v[60:63]\n\t"
      "ds_read_b64_tr_b8 v[228:229], v251 offset:42304\n\t"
      "v_med3_f32 v2, v2, -s74, s74\n\t"
      "v_med3_f32 v3, v3, -s74, s74\n\t"
      "v_exp_f32 v0, v0\n\t"
      "v_exp_f32 v1, v1\n\t"
      "v_mfma_i32_16x16x64_i8 v[60:63], v[236:239], v[148:151], v[60:63]\n\t"
      "ds_read_b64_tr_b8 v[230:231], v251 offset:43328\n\t"
      "v_exp_f32 v2, v2\n\t"
      "v_exp_f32 v3, v3\n\t"
      "v_mul_f32 v44, s72, v44\n\t"
      "v_mul_f32 v45, s72, v45\n\t"
      "v_mfma_i32_16x16x64_i8 v[92:95], v[232:235], v[136:139], v[92:95]\n\t"
      "v_mul_f32 v46, s72, v46\n\t"
      "v_mul_f32 v47, s72, v47\n\t"
      "v_add_f32 v0, 1.0, v0\n\t"
      "v_add_f32 v1, 1.0, v1\n\t"
      "v_mfma_i32_16x16x64_i8 v[92:95], v[236:239], v[152:155], v[92:95]\n\t"
      "v_add_f32 v2, 1.0, v2\n\t"
      "v_add_f32 v3, 1.0, v3\n\t"
      "v_rcp_f32 v0, v0\n\t"
      "v_rcp_f32 v1, v1\n\t"
      "v_mfma_i32_16x16x64_i8 v[124:127], v[232:235], v[140:143], v[124:127]\n\t"
      "v_rcp_f32 v2, v2\n\t"
      "v_rcp_f32 v3, v3\n\t"
      "v_mul_f32 v44, v44, v0\n\t"
      "v_mul_f32 v45, v45, v1\n\t"
      "v_mfma_i32_16x16x64_i8 v[124:127], v[236:239], v[156:159], v[124:127]\n\t"
      "v_mul_f32 v46, v46, v2\n\t"
      "v_mul_f32 v47, v47, v3\n\t"
      "ds_write_b128 v254, v[44:47] offset:2048\n\t"
      "v_xor_b32 v254, 64, v254\n\t"
      "s_waitcnt lgkmcnt(4)\n\t"
      "v_mfma_i32_16x16x64_i8 v[24:27], v[240:243], v[160:163], v[24:27]\n\t"
      "ds_read_b128 v[40:43], v252 offset:2048\n\t"
      "ds_read_b128 v[44:47], v252 offset:3072\n\t"
      "s_add_u32 s70, s76, 0x80000\n\t"
      "s_waitcnt lgkmcnt(12)\n\t"
      "v_mfma_i32_16x16x64_i8 v[24:27], v[244:247], v[176:179], v[24:27]\n\t"
      "buffer_store_dwordx4 v[8:11], v255, s[64:67], s76 offen offset:128 sc1\n\t"
      "buffer_store_dwordx4 v[12:15], v255, s[64:67], s70 offen offset:128 sc1\n\t"
      "v_cvt_f32_i32 v72, v72\n\t"
      "v_cvt_f32_i32 v73, v73\n\t"
      "v_mfma_i32_16x16x64_i8 v[56:59], v[240:243], v[164:167], v[56:59]\n\t"
      "v_cvt_f32_i32 v74, v74\n\t"
      "v_cvt_f32_i32 v75, v75\n\t"
      "v_mul_f32 v0, s73, v72\n\t"
      "v_mul_f32 v1, s73, v73\n\t"
      "v_mfma_i32_16x16x64_i8 v[56:59], v[244:247], v[180:183], v[56:59]\n\t"
      "v_mul_f32 v2, s73, v74\n\t"
      "v_mul_f32 v3, s73, v75\n\t"
      "v_med3_f32 v0, v0, -s74, s74\n\t"
      "v_med3_f32 v1, v1, -s74, s74\n\t"
      "v_mfma_i32_16x16x64_i8 v[88:91], v[240:243], v[168:171], v[88:91]\n\t"
      "v_med3_f32 v2, v2, -s74, s74\n\t"
      "v_med3_f32 v3, v3, -s74, s74\n\t"
      "v_exp_f32 v0, v0\n\t"
      "v_exp_f32 v1, v1\n\t"
      "v_mfma_i32_16x16x64_i8 v[88:91], v[244:247], v[184:187], v[88:91]\n\t"
      "v_exp_f32 v2, v2\n\t"
      "v_exp_f32 v3, v3\n\t"
      "v_mul_f32 v72, s72, v72\n\t"
      "v_mul_f32 v73, s72, v73\n\t"
      "v_mfma_i32_16x16x64_i8 v[120:123], v[240:243], v[172:175], v[120:123]\n\t"
      "v_mul_f32 v74, s72, v74\n\t"
      "v_mul_f32 v75, s72, v75\n\t"
      "v_add_f32 v0, 1.0, v0\n\t"
      "v_add_f32 v1, 1.0, v1\n\t"
      "v_mfma_i32_16x16x64_i8 v[120:123], v[244:247], v[188:191], v[120:123]\n\t"
      "v_add_f32 v2, 1.0, v2\n\t"
      "v_add_f32 v3, 1.0, v3\n\t"
      "v_rcp_f32 v0, v0\n\t"
      "v_rcp_f32 v1, v1\n\t"
      "s_waitcnt lgkmcnt(0)\n\t"
      "v_mfma_i32_16x16x64_i8 v[28:31], v[224:227], v[160:163], v[28:31]\n\t"
      "v_rcp_f32 v2, v2\n\t"
      "v_rcp_f32 v3, v3\n\t"
      "v_mul_f32 v72, v72, v0\n\t"
      "v_mul_f32 v73, v73, v1\n\t"
      "v_mfma_i32_16x16x64_i8 v[28:31], v[228:231], v[176:179], v[28:31]\n\t"
      "v_mul_f32 v74, v74, v2\n\t"
      "v_mul_f32 v75, v75, v3\n\t"
      "ds_write_b128 v254, v[72:75] offset:0\n\t"
      "v_xor_b32 v254, 64, v254\n\t"
      "v_mfma_i32_16x16x64_i8 v[60:63], v[224:227], v[164:167], v[60:63]\n\t"
      "v_cvt_f32_i32 v76, v76\n\t"
      "v_cvt_f32_i32 v77, v77\n\t"
      "v_cvt_f32_i32 v78, v78\n\t"
      "v_cvt_f32_i32 v79, v79\n\t"
      "v_mfma_i32_16x16x64_i8 v[60:63], v[228:231], v[180:183], v[60:63]\n\t"
      "v_mul_f32 v0, s73, v76\n\t"
      "v_mul_f32 v1, s73, v77\n\t"
      "v_mul_f32 v2, s73, v78\n\t"
      "v_mul_f32 v3, s73, v79\n\t"
      "v_mfma_i32_16x16x64_i8 v[92:95], v[224:227], v[168:171], v[92:95]\n\t"
      "v_med3_f32 v0, v0, -s74, s74\n\t"
      "v_med3_f32 v1, v1, -s74, s74\n\t"
      "v_med3_f32 v2, v2, -s74, s74\n\t"
      "v_med3_f32 v3, v3, -s74, s74\n\t"
      "v_mfma_i32_16x16x64_i8 v[92:95], v[228:231], v[184:187], v[92:95]\n\t"
      "v_exp_f32 v0, v0\n\t"
      "v_exp_f32 v1, v1\n\t"
      "v_exp_f32 v2, v2\n\t"
      "v_exp_f32 v3, v3\n\t"
      "v_mfma_i32_16x16x64_i8 v[124:127], v[224:227], v[172:175], v[124:127]\n\t"
      "v_mul_f32 v76, s72, v76\n\t"
      "v_mul_f32 v77, s72, v77\n\t"
      "v_mul_f32 v78, s72, v78\n\t"
      "v_mul_f32 v79, s72, v79\n\t"
      "v_mfma_i32_16x16x64_i8 v[124:127], v[228:231], v[188:191], v[124:127]\n\t"
      "v_add_f32 v0, 1.0, v0\n\t"
      "v_add_f32 v1, 1.0, v1\n\t"
      "v_add_f32 v2, 1.0, v2\n\t"
      "v_add_f32 v3, 1.0, v3\n\t"
      "s_nop 7\n\t"
      "s_nop 1\n\t"
      "v_rcp_f32 v0, v0\n\t"
      "v_rcp_f32 v1, v1\n\t"
      "v_rcp_f32 v2, v2\n\t"
      "v_rcp_f32 v3, v3\n\t"
      "v_mul_f32 v76, v76, v0\n\t"
      "v_mul_f32 v77, v77, v1\n\t"
      "v_mul_f32 v78, v78, v2\n\t"
      "v_mul_f32 v79, v79, v3\n\t"
      "ds_write_b128 v254, v[76:79] offset:0\n\t"
      "v_xor_b32 v254, 64, v254\n\t"
      "ds_read_b128 v[72:75], v252 offset:0\n\t"
      "ds_read_b128 v[76:79], v252 offset:1024\n\t"
      "s_add_u32 s70, s77, 0x80000\n\t"
      "s_waitcnt lgkmcnt(4)\n\t"
      "buffer_store_dwordx4 v[40:43], v255, s[64:67], s77 offen offset:128 sc1\n\t"
      "buffer_store_dwordx4 v[44:47], v255, s[64:67], s70 offen offset:128 sc1\n\t"
      "v_cvt_f32_i32 v104, v104\n\t"
      "v_cvt_f32_i32 v105, v105\n\t"
      "v_cvt_f32_i32 v106, v106\n\t"
      "v_cvt_f32_i32 v107, v107\n\t"
      "v_mul_f32 v0, s73, v104\n\t"
      "v_mul_f32 v1, s73, v105\n\t"
      "v_mul_f32 v2, s73, v106\n\t"
      "v_mul_f32 v3, s73, v107\n\t"
      "v_med3_f32 v0, v0, -s74, s74\n\t"
      "v_med3_f32 v1, v1, -s74, s74\n\t"
      "v_med3_f32 v2, v2, -s74, s74\n\t"
      "v_med3_f32 v3, v3, -s74, s74\n\t"
      "v_exp_f32 v0, v0\n\t"
      "v_exp_f32 v1, v1\n\t"
      "v_exp_f32 v2, v2\n\t"
      "v_exp_f32 v3, v3\n\t"
      "v_mul_f32 v104, s72, v104\n\t"
      "v_mul_f32 v105, s72, v105\n\t"
      "v_mul_f32 v106, s72, v106\n\t"
      "v_mul_f32 v107, s72, v107\n\t"
      "v_add_f32 v0, 1.0, v0\n\t"
      "v_add_f32 v1, 1.0, v1\n\t"
      "v_add_f32 v2, 1.0, v2\n\t"
      "v_add_f32 v3, 1.0, v3\n\t"
      "v_rcp_f32 v0, v0\n\t"
      "v_rcp_f32 v1, v1\n\t"
      "v_rcp_f32 v2, v2\n\t"
      "v_rcp_f32 v3, v3\n\t"
      "v_mul_f32 v104, v104, v0\n\t"
      "v_mul_f32 v105, v105, v1\n\t"
      "v_mul_f32 v106, v106, v2\n\t"
      "v_mul_f32 v107, v107, v3\n\t"
      "ds_write_b128 v254, v[104:107] offset:2048\n\t"
      "v_xor_b32 v254, 64, v254\n\t"
      "v_cvt_f32_i32 v108, v108\n\t"
      "v_cvt_f32_i32 v109, v109\n\t"
      "v_cvt_f32_i32 v110, v110\n\t"
      "v_cvt_f32_i32 v111, v111\n\t"
      "v_mul_f32 v0, s73, v108\n\t"
      "v_mul_f32 v1, s73, v109\n\t"
      "v_mul_f32 v2, s73, v110\n\t"
      "v_mul_f32 v3, s73, v111\n\t"
      "v_med3_f32 v0, v0, -s74, s74\n\t"
      "v_med3_f32 v1, v1, -s74, s74\n\t"
      "v_med3_f32 v2, v2, -s74, s74\n\t"
      "v_med3_f32 v3, v3, -s74, s74\n\t"
      "v_exp_f32 v0, v0\n\t"
      "v_exp_f32 v1, v1\n\t"
      "v_exp_f32 v2, v2\n\t"
      "v_exp_f32 v3, v3\n\t"
      "v_mul_f32 v108, s72, v108\n\t"
      "v_mul_f32 v109, s72, v109\n\t"
      "v_mul_f32 v110, s72, v110\n\t"
      "v_mul_f32 v111, s72, v111\n\t"
      "v_add_f32 v0, 1.0, v0\n\t"
      "v_add_f32 v1, 1.0, v1\n\t"
      "v_add_f32 v2, 1.0, v2\n\t"
      "v_add_f32 v3, 1.0, v3\n\t"
      "v_rcp_f32 v0, v0\n\t"
      "v_rcp_f32 v1, v1\n\t"
      "v_rcp_f32 v2, v2\n\t"
      "v_rcp_f32 v3, v3\n\t"
      "v_mul_f32 v108, v108, v0\n\t"
      "v_mul_f32 v109, v109, v1\n\t"
      "v_mul_f32 v110, v110, v2\n\t"
      "v_mul_f32 v111, v111, v3\n\t"
      "ds_write_b128 v254, v[108:111] offset:2048\n\t"
      "v_xor_b32 v254, 64, v254\n\t"
      "ds_read_b128 v[104:107], v252 offset:2048\n\t"
      "ds_read_b128 v[108:111], v252 offset:3072\n\t"
      "s_add_u32 s70, s78, 0x80000\n\t"
      "s_waitcnt lgkmcnt(4)\n\t"
      "buffer_store_dwordx4 v[72:75], v255, s[64:67], s78 offen offset:128 sc1\n\t"
      "buffer_store_dwordx4 v[76:79], v255, s[64:67], s70 offen offset:128 sc1\n\t"
      "v_cvt_f32_i32 v16, v16\n\t"
      "v_cvt_f32_i32 v17, v17\n\t"
      "v_cvt_f32_i32 v18, v18\n\t"
      "v_cvt_f32_i32 v19, v19\n\t"
      "v_mul_f32 v0, s73, v16\n\t"
      "v_mul_f32 v1, s73, v17\n\t"
      "v_mul_f32 v2, s73, v18\n\t"
      "v_mul_f32 v3, s73, v19\n\t"
      "v_med3_f32 v0, v0, -s74, s74\n\t"
      "v_med3_f32 v1, v1, -s74, s74\n\t"
      "v_med3_f32 v2, v2, -s74, s74\n\t"
      "v_med3_f32 v3, v3, -s74, s74\n\t"
      "v_exp_f32 v0, v0\n\t"
      "v_exp_f32 v1, v1\n\t"
      "v_exp_f32 v2, v2\n\t"
      "v_exp_f32 v3, v3\n\t"
      "v_mul_f32 v16, s72, v16\n\t"
      "v_mul_f32 v17, s72, v17\n\t"
      "v_mul_f32 v18, s72, v18\n\t"
      "v_mul_f32 v19, s72, v19\n\t"
      "v_add_f32 v0, 1.0, v0\n\t"
      "v_add_f32 v1, 1.0, v1\n\t"
      "v_add_f32 v2, 1.0, v2\n\t"
      "v_add_f32 v3, 1.0, v3\n\t"
      "v_rcp_f32 v0, v0\n\t"
      "v_rcp_f32 v1, v1\n\t"
      "v_rcp_f32 v2, v2\n\t"
      "v_rcp_f32 v3, v3\n\t"
      "v_mul_f32 v16, v16, v0\n\t"
      "v_mul_f32 v17, v17, v1\n\t"
      "v_mul_f32 v18, v18, v2\n\t"
      "v_mul_f32 v19, v19, v3\n\t"
      "ds_write_b128 v254, v[16:19] offset:0\n\t"
      "v_xor_b32 v254, 64, v254\n\t"
      "v_cvt_f32_i32 v20, v20\n\t"
      "v_cvt_f32_i32 v21, v21\n\t"
      "v_cvt_f32_i32 v22, v22\n\t"
      "v_cvt_f32_i32 v23, v23\n\t"
      "v_mul_f32 v0, s73, v20\n\t"
      "v_mul_f32 v1, s73, v21\n\t"
      "v_mul_f32 v2, s73, v22\n\t"
      "v_mul_f32 v3, s73, v23\n\t"
      "v_med3_f32 v0, v0, -s74, s74\n\t"
      "v_med3_f32 v1, v1, -s74, s74\n\t"
      "v_med3_f32 v2, v2, -s74, s74\n\t"
      "v_med3_f32 v3, v3, -s74, s74\n\t"
      "v_exp_f32 v0, v0\n\t"
      "v_exp_f32 v1, v1\n\t"
      "v_exp_f32 v2, v2\n\t"
      "v_exp_f32 v3, v3\n\t"
      "v_mul_f32 v20, s72, v20\n\t"
      "v_mul_f32 v21, s72, v21\n\t"
      "v_mul_f32 v22, s72, v22\n\t"
      "v_mul_f32 v23, s72, v23\n\t"
      "v_add_f32 v0, 1.0, v0\n\t"
      "v_add_f32 v1, 1.0, v1\n\t"
      "v_add_f32 v2, 1.0, v2\n\t"
      "v_add_f32 v3, 1.0, v3\n\t"
      "v_rcp_f32 v0, v0\n\t"
      "v_rcp_f32 v1, v1\n\t"
      "v_rcp_f32 v2, v2\n\t"
      "v_rcp_f32 v3, v3\n\t"
      "v_mul_f32 v20, v20, v0\n\t"
      "v_mul_f32 v21, v21, v1\n\t"
      "v_mul_f32 v22, v22, v2\n\t"
      "v_mul_f32 v23, v23, v3\n\t"
      "ds_write_b128 v254, v[20:23] offset:0\n\t"
      "v_xor_b32 v254, 64, v254\n\t"
      "ds_read_b128 v[16:19], v252 offset:0\n\t"
      "ds_read_b128 v[20:23], v252 offset:1024\n\t"
      "s_add_u32 s70, s79, 0x80000\n\t"
      "s_waitcnt lgkmcnt(4)\n\t"
      "buffer_store_dwordx4 v[104:107], v255, s[64:67], s79 offen offset:128 sc1\n\t"
      "buffer_store_dwordx4 v[108:111], v255, s[64:67], s70 offen offset:128 sc1\n\t"
      "v_cvt_f32_i32 v48, v48\n\t"
      "v_cvt_f32_i32 v49, v49\n\t"
      "v_cvt_f32_i32 v50, v50\n\t"
      "v_cvt_f32_i32 v51, v51\n\t"
      "v_mul_f32 v0, s73, v48\n\t"
      "v_mul_f32 v1, s73, v49\n\t"
      "v_mul_f32 v2, s73, v50\n\t"
      "v_mul_f32 v3, s73, v51\n\t"
      "v_med3_f32 v0, v0, -s74, s74\n\t"
      "v_med3_f32 v1, v1, -s74, s74\n\t"
      "v_med3_f32 v2, v2, -s74, s74\n\t"
      "v_med3_f32 v3, v3, -s74, s74\n\t"
      "v_exp_f32 v0, v0\n\t"
      "v_exp_f32 v1, v1\n\t"
      "v_exp_f32 v2, v2\n\t"
      "v_exp_f32 v3, v3\n\t"
      "v_mul_f32 v48, s72, v48\n\t"
      "v_mul_f32 v49, s72, v49\n\t"
      "v_mul_f32 v50, s72, v50\n\t"
      "v_mul_f32 v51, s72, v51\n\t"
      "v_add_f32 v0, 1.0, v0\n\t"
      "v_add_f32 v1, 1.0, v1\n\t"
      "v_add_f32 v2, 1.0, v2\n\t"
      "v_add_f32 v3, 1.0, v3\n\t"
      "v_rcp_f32 v0, v0\n\t"
      "v_rcp_f32 v1, v1\n\t"
      "v_rcp_f32 v2, v2\n\t"
      "v_rcp_f32 v3, v3\n\t"
      "v_mul_f32 v48, v48, v0\n\t"
      "v_mul_f32 v49, v49, v1\n\t"
      "v_mul_f32 v50, v50, v2\n\t"
      "v_mul_f32 v51, v51, v3\n\t"
      "ds_write_b128 v254, v[48:51] offset:2048\n\t"
      "v_xor_b32 v254, 64, v254\n\t"
      "v_cvt_f32_i32 v52, v52\n\t"
      "v_cvt_f32_i32 v53, v53\n\t"
      "v_cvt_f32_i32 v54, v54\n\t"
      "v_cvt_f32_i32 v55, v55\n\t"
      "v_mul_f32 v0, s73, v52\n\t"
      "v_mul_f32 v1, s73, v53\n\t"
      "v_mul_f32 v2, s73, v54\n\t"
      "v_mul_f32 v3, s73, v55\n\t"
      "v_med3_f32 v0, v0, -s74, s74\n\t"
      "v_med3_f32 v1, v1, -s74, s74\n\t"
      "v_med3_f32 v2, v2, -s74, s74\n\t"
      "v_med3_f32 v3, v3, -s74, s74\n\t"
      "v_exp_f32 v0, v0\n\t"
      "v_exp_f32 v1, v1\n\t"
      "v_exp_f32 v2, v2\n\t"
      "v_exp_f32 v3, v3\n\t"
      "v_mul_f32 v52, s72, v52\n\t"
      "v_mul_f32 v53, s72, v53\n\t"
      "v_mul_f32 v54, s72, v54\n\t"
      "v_mul_f32 v55, s72, v55\n\t"
      "v_add_f32 v0, 1.0, v0\n\t"
      "v_add_f32 v1, 1.0, v1\n\t"
      "v_add_f32 v2, 1.0, v2\n\t"
      "v_add_f32 v3, 1.0, v3\n\t"
      "v_rcp_f32 v0, v0\n\t"
      "v_rcp_f32 v1, v1\n\t"
      "v_rcp_f32 v2, v2\n\t"
      "v_rcp_f32 v3, v3\n\t"
      "v_mul_f32 v52, v52, v0\n\t"
      "v_mul_f32 v53, v53, v1\n\t"
      "v_mul_f32 v54, v54, v2\n\t"
      "v_mul_f32 v55, v55, v3\n\t"
      "ds_write_b128 v254, v[52:55] offset:2048\n\t"
      "v_xor_b32 v254, 64, v254\n\t"
      "ds_read_b128 v[48:51], v252 offset:2048\n\t"
      "ds_read_b128 v[52:55], v252 offset:3072\n\t"
      "s_add_u32 s70, s76, 0x80000\n\t"
      "s_waitcnt lgkmcnt(4)\n\t"
      "buffer_store_dwordx4 v[16:19], v255, s[64:67], s76 offen offset:256 sc1\n\t"
      "buffer_store_dwordx4 v[20:23], v255, s[64:67], s70 offen offset:256 sc1\n\t"
      "v_cvt_f32_i32 v80, v80\n\t"
      "v_cvt_f32_i32 v81, v81\n\t"
      "v_cvt_f32_i32 v82, v82\n\t"
      "v_cvt_f32_i32 v83, v83\n\t"
      "v_mul_f32 v0, s73, v80\n\t"
      "v_mul_f32 v1, s73, v81\n\t"
      "v_mul_f32 v2, s73, v82\n\t"
      "v_mul_f32 v3, s73, v83\n\t"
      "v_med3_f32 v0, v0, -s74, s74\n\t"
      "v_med3_f32 v1, v1, -s74, s74\n\t"
      "v_med3_f32 v2, v2, -s74, s74\n\t"
      "v_med3_f32 v3, v3, -s74, s74\n\t"
      "v_exp_f32 v0, v0\n\t"
      "v_exp_f32 v1, v1\n\t"
      "v_exp_f32 v2, v2\n\t"
      "v_exp_f32 v3, v3\n\t"
      "v_mul_f32 v80, s72, v80\n\t"
      "v_mul_f32 v81, s72, v81\n\t"
      "v_mul_f32 v82, s72, v82\n\t"
      "v_mul_f32 v83, s72, v83\n\t"
      "v_add_f32 v0, 1.0, v0\n\t"
      "v_add_f32 v1, 1.0, v1\n\t"
      "v_add_f32 v2, 1.0, v2\n\t"
      "v_add_f32 v3, 1.0, v3\n\t"
      "v_rcp_f32 v0, v0\n\t"
      "v_rcp_f32 v1, v1\n\t"
      "v_rcp_f32 v2, v2\n\t"
      "v_rcp_f32 v3, v3\n\t"
      "v_mul_f32 v80, v80, v0\n\t"
      "v_mul_f32 v81, v81, v1\n\t"
      "v_mul_f32 v82, v82, v2\n\t"
      "v_mul_f32 v83, v83, v3\n\t"
      "ds_write_b128 v254, v[80:83] offset:0\n\t"
      "v_xor_b32 v254, 64, v254\n\t"
      "v_cvt_f32_i32 v84, v84\n\t"
      "v_cvt_f32_i32 v85, v85\n\t"
      "v_cvt_f32_i32 v86, v86\n\t"
      "v_cvt_f32_i32 v87, v87\n\t"
      "v_mul_f32 v0, s73, v84\n\t"
      "v_mul_f32 v1, s73, v85\n\t"
      "v_mul_f32 v2, s73, v86\n\t"
      "v_mul_f32 v3, s73, v87\n\t"
      "v_med3_f32 v0, v0, -s74, s74\n\t"
      "v_med3_f32 v1, v1, -s74, s74\n\t"
      "v_med3_f32 v2, v2, -s74, s74\n\t"
      "v_med3_f32 v3, v3, -s74, s74\n\t"
      "v_exp_f32 v0, v0\n\t"
      "v_exp_f32 v1, v1\n\t"
      "v_exp_f32 v2, v2\n\t"
      "v_exp_f32 v3, v3\n\t"
      "v_mul_f32 v84, s72, v84\n\t"
      "v_mul_f32 v85, s72, v85\n\t"
      "v_mul_f32 v86, s72, v86\n\t"
      "v_mul_f32 v87, s72, v87\n\t"
      "v_add_f32 v0, 1.0, v0\n\t"
      "v_add_f32 v1, 1.0, v1\n\t"
      "v_add_f32 v2, 1.0, v2\n\t"
      "v_add_f32 v3, 1.0, v3\n\t"
      "v_rcp_f32 v0, v0\n\t"
      "v_rcp_f32 v1, v1\n\t"
      "v_rcp_f32 v2, v2\n\t"
      "v_rcp_f32 v3, v3\n\t"
      "v_mul_f32 v84, v84, v0\n\t"
      "v_mul_f32 v85, v85, v1\n\t"
      "v_mul_f32 v86, v86, v2\n\t"
      "v_mul_f32 v87, v87, v3\n\t"
      "ds_write_b128 v254, v[84:87] offset:0\n\t"
      "v_xor_b32 v254, 64, v254\n\t"
      "ds_read_b128 v[80:83], v252 offset:0\n\t"
      "ds_read_b128 v[84:87], v252 offset:1024\n\t"
      "s_add_u32 s70, s77, 0x80000\n\t"
      "s_waitcnt lgkmcnt(4)\n\t"
      "buffer_store_dwordx4 v[48:51], v255, s[64:67], s77 offen offset:256 sc1\n\t"
      "buffer_store_dwordx4 v[52:55], v255, s[64:67], s70 offen offset:256 sc1\n\t"
      "v_cvt_f32_i32 v112, v112\n\t"
      "v_cvt_f32_i32 v113, v113\n\t"
      "v_cvt_f32_i32 v114, v114\n\t"
      "v_cvt_f32_i32 v115, v115\n\t"
      "v_mul_f32 v0, s73, v112\n\t"
      "v_mul_f32 v1, s73, v113\n\t"
      "v_mul_f32 v2, s73, v114\n\t"
      "v_mul_f32 v3, s73, v115\n\t"
      "v_med3_f32 v0, v0, -s74, s74\n\t"
      "v_med3_f32 v1, v1, -s74, s74\n\t"
      "v_med3_f32 v2, v2, -s74, s74\n\t"
      "v_med3_f32 v3, v3, -s74, s74\n\t"
      "v_exp_f32 v0, v0\n\t"
      "v_exp_f32 v1, v1\n\t"
      "v_exp_f32 v2, v2\n\t"
      "v_exp_f32 v3, v3\n\t"
      "v_mul_f32 v112, s72, v112\n\t"
      "v_mul_f32 v113, s72, v113\n\t"
      "v_mul_f32 v114, s72, v114\n\t"
      "v_mul_f32 v115, s72, v115\n\t"
      "v_add_f32 v0, 1.0, v0\n\t"
      "v_add_f32 v1, 1.0, v1\n\t"
      "v_add_f32 v2, 1.0, v2\n\t"
      "v_add_f32 v3, 1.0, v3\n\t"
      "v_rcp_f32 v0, v0\n\t"
      "v_rcp_f32 v1, v1\n\t"
      "v_rcp_f32 v2, v2\n\t"
      "v_rcp_f32 v3, v3\n\t"
      "v_mul_f32 v112, v112, v0\n\t"
      "v_mul_f32 v113, v113, v1\n\t"
      "v_mul_f32 v114, v114, v2\n\t"
      "v_mul_f32 v115, v115, v3\n\t"
      "ds_write_b128 v254, v[112:115] offset:2048\n\t"
      "v_xor_b32 v254, 64, v254\n\t"
      "v_cvt_f32_i32 v116, v116\n\t"
      "v_cvt_f32_i32 v117, v117\n\t"
      "v_cvt_f32_i32 v118, v118\n\t"
      "v_cvt_f32_i32 v119, v119\n\t"
      "v_mul_f32 v0, s73, v116\n\t"
      "v_mul_f32 v1, s73, v117\n\t"
      "v_mul_f32 v2, s73, v118\n\t"
      "v_mul_f32 v3, s73, v119\n\t"
      "v_med3_f32 v0, v0, -s74, s74\n\t"
      "v_med3_f32 v1, v1, -s74, s74\n\t"
      "v_med3_f32 v2, v2, -s74, s74\n\t"
      "v_med3_f32 v3, v3, -s74, s74\n\t"
      "v_exp_f32 v0, v0\n\t"
      "v_exp_f32 v1, v1\n\t"
      "v_exp_f32 v2, v2\n\t"
      "v_exp_f32 v3, v3\n\t"
      "v_mul_f32 v116, s72, v116\n\t"
      "v_mul_f32 v117, s72, v117\n\t"
      "v_mul_f32 v118, s72, v118\n\t"
      "v_mul_f32 v119, s72, v119\n\t"
      "v_add_f32 v0, 1.0, v0\n\t"
      "v_add_f32 v1, 1.0, v1\n\t"
      "v_add_f32 v2, 1.0, v2\n\t"
      "v_add_f32 v3, 1.0, v3\n\t"
      "v_rcp_f32 v0, v0\n\t"
      "v_rcp_f32 v1, v1\n\t"
      "v_rcp_f32 v2, v2\n\t"
      "v_rcp_f32 v3, v3\n\t"
      "v_mul_f32 v116, v116, v0\n\t"
      "v_mul_f32 v117, v117, v1\n\t"
      "v_mul_f32 v118, v118, v2\n\t"
      "v_mul_f32 v119, v119, v3\n\t"
      "ds_write_b128 v254, v[116:119] offset:2048\n\t"
      "v_xor_b32 v254, 64, v254\n\t"
      "ds_read_b128 v[112:115], v252 offset:2048\n\t"
      "ds_read_b128 v[116:119], v252 offset:3072\n\t"
      "s_add_u32 s70, s78, 0x80000\n\t"
      "s_waitcnt lgkmcnt(4)\n\t"
      "buffer_store_dwordx4 v[80:83], v255, s[64:67], s78 offen offset:256 sc1\n\t"
      "buffer_store_dwordx4 v[84:87], v255, s[64:67], s70 offen offset:256 sc1\n\t"
      "v_cvt_f32_i32 v24, v24\n\t"
      "v_cvt_f32_i32 v25, v25\n\t"
      "v_cvt_f32_i32 v26, v26\n\t"
      "v_cvt_f32_i32 v27, v27\n\t"
      "v_mul_f32 v0, s73, v24\n\t"
      "v_mul_f32 v1, s73, v25\n\t"
      "v_mul_f32 v2, s73, v26\n\t"
      "v_mul_f32 v3, s73, v27\n\t"
      "v_med3_f32 v0, v0, -s74, s74\n\t"
      "v_med3_f32 v1, v1, -s74, s74\n\t"
      "v_med3_f32 v2, v2, -s74, s74\n\t"
      "v_med3_f32 v3, v3, -s74, s74\n\t"
      "v_exp_f32 v0, v0\n\t"
      "v_exp_f32 v1, v1\n\t"
      "v_exp_f32 v2, v2\n\t"
      "v_exp_f32 v3, v3\n\t"
      "v_mul_f32 v24, s72, v24\n\t"
      "v_mul_f32 v25, s72, v25\n\t"
      "v_mul_f32 v26, s72, v26\n\t"
      "v_mul_f32 v27, s72, v27\n\t"
      "v_add_f32 v0, 1.0, v0\n\t"
      "v_add_f32 v1, 1.0, v1\n\t"
      "v_add_f32 v2, 1.0, v2\n\t"
      "v_add_f32 v3, 1.0, v3\n\t"
      "v_rcp_f32 v0, v0\n\t"
      "v_rcp_f32 v1, v1\n\t"
      "v_rcp_f32 v2, v2\n\t"
      "v_rcp_f32 v3, v3\n\t"
      "v_mul_f32 v24, v24, v0\n\t"
      "v_mul_f32 v25, v25, v1\n\t"
      "v_mul_f32 v26, v26, v2\n\t"
      "v_mul_f32 v27, v27, v3\n\t"
      "ds_write_b128 v254, v[24:27] offset:0\n\t"
      "v_xor_b32 v254, 64, v254\n\t"
      "v_cvt_f32_i32 v28, v28\n\t"
      "v_cvt_f32_i32 v29, v29\n\t"
      "v_cvt_f32_i32 v30, v30\n\t"
      "v_cvt_f32_i32 v31, v31\n\t"
      "v_mul_f32 v0, s73, v28\n\t"
      "v_mul_f32 v1, s73, v29\n\t"
      "v_mul_f32 v2, s73, v30\n\t"
      "v_mul_f32 v3, s73, v31\n\t"
      "v_med3_f32 v0, v0, -s74, s74\n\t"
      "v_med3_f32 v1, v1, -s74, s74\n\t"
      "v_med3_f32 v2, v2, -s74, s74\n\t"
      "v_med3_f32 v3, v3, -s74, s74\n\t"
      "v_exp_f32 v0, v0\n\t"
      "v_exp_f32 v1, v1\n\t"
      "v_exp_f32 v2, v2\n\t"
      "v_exp_f32 v3, v3\n\t"
      "v_mul_f32 v28, s72, v28\n\t"
      "v_mul_f32 v29, s72, v29\n\t"
      "v_mul_f32 v30, s72, v30\n\t"
      "v_mul_f32 v31, s72, v31\n\t"
      "v_add_f32 v0, 1.0, v0\n\t"
      "v_add_f32 v1, 1.0, v1\n\t"
      "v_add_f32 v2, 1.0, v2\n\t"
      "v_add_f32 v3, 1.0, v3\n\t"
      "v_rcp_f32 v0, v0\n\t"
      "v_rcp_f32 v1, v1\n\t"
      "v_rcp_f32 v2, v2\n\t"
      "v_rcp_f32 v3, v3\n\t"
      "v_mul_f32 v28, v28, v0\n\t"
      "v_mul_f32 v29, v29, v1\n\t"
      "v_mul_f32 v30, v30, v2\n\t"
      "v_mul_f32 v31, v31, v3\n\t"
      "ds_write_b128 v254, v[28:31] offset:0\n\t"
      "v_xor_b32 v254, 64, v254\n\t"
      "ds_read_b128 v[24:27], v252 offset:0\n\t"
      "ds_read_b128 v[28:31], v252 offset:1024\n\t"
      "s_add_u32 s70, s79, 0x80000\n\t"
      "s_waitcnt lgkmcnt(4)\n\t"
      "buffer_store_dwordx4 v[112:115], v255, s[64:67], s79 offen offset:256 sc1\n\t"
      "buffer_store_dwordx4 v[116:119], v255, s[64:67], s70 offen offset:256 sc1\n\t"
      "v_cvt_f32_i32 v56, v56\n\t"
      "v_cvt_f32_i32 v57, v57\n\t"
      "v_cvt_f32_i32 v58, v58\n\t"
      "v_cvt_f32_i32 v59, v59\n\t"
      "v_mul_f32 v0, s73, v56\n\t"
      "v_mul_f32 v1, s73, v57\n\t"
      "v_mul_f32 v2, s73, v58\n\t"
      "v_mul_f32 v3, s73, v59\n\t"
      "v_med3_f32 v0, v0, -s74, s74\n\t"
      "v_med3_f32 v1, v1, -s74, s74\n\t"
      "v_med3_f32 v2, v2, -s74, s74\n\t"
      "v_med3_f32 v3, v3, -s74, s74\n\t"
      "v_exp_f32 v0, v0\n\t"
      "v_exp_f32 v1, v1\n\t"
      "v_exp_f32 v2, v2\n\t"
      "v_exp_f32 v3, v3\n\t"
      "v_mul_f32 v56, s72, v56\n\t"
      "v_mul_f32 v57, s72, v57\n\t"
      "v_mul_f32 v58, s72, v58\n\t"
      "v_mul_f32 v59, s72, v59\n\t"
      "v_add_f32 v0, 1.0, v0\n\t"
      "v_add_f32 v1, 1.0, v1\n\t"
      "v_add_f32 v2, 1.0, v2\n\t"
      "v_add_f32 v3, 1.0, v3\n\t"
      "v_rcp_f32 v0, v0\n\t"
      "v_rcp_f32 v1, v1\n\t"
      "v_rcp_f32 v2, v2\n\t"
      "v_rcp_f32 v3, v3\n\t"
      "v_mul_f32 v56, v56, v0\n\t"
      "v_mul_f32 v57, v57, v1\n\t"
      "v_mul_f32 v58, v58, v2\n\t"
      "v_mul_f32 v59, v59, v3\n\t"
      "ds_write_b128 v254, v[56:59] offset:2048\n\t"
      "v_xor_b32 v254, 64, v254\n\t"
      "v_cvt_f32_i32 v60, v60\n\t"
      "v_cvt_f32_i32 v61, v61\n\t"
      "v_cvt_f32_i32 v62, v62\n\t"
      "v_cvt_f32_i32 v63, v63\n\t"
      "v_mul_f32 v0, s73, v60\n\t"
      "v_mul_f32 v1, s73, v61\n\t"
      "v_mul_f32 v2, s73, v62\n\t"
      "v_mul_f32 v3, s73, v63\n\t"
      "v_med3_f32 v0, v0, -s74, s74\n\t"
      "v_med3_f32 v1, v1, -s74, s74\n\t"
      "v_med3_f32 v2, v2, -s74, s74\n\t"
      "v_med3_f32 v3, v3, -s74, s74\n\t"
      "v_exp_f32 v0, v0\n\t"
      "v_exp_f32 v1, v1\n\t"
      "v_exp_f32 v2, v2\n\t"
      "v_exp_f32 v3, v3\n\t"
      "v_mul_f32 v60, s72, v60\n\t"
      "v_mul_f32 v61, s72, v61\n\t"
      "v_mul_f32 v62, s72, v62\n\t"
      "v_mul_f32 v63, s72, v63\n\t"
      "v_add_f32 v0, 1.0, v0\n\t"
      "v_add_f32 v1, 1.0, v1\n\t"
      "v_add_f32 v2, 1.0, v2\n\t"
      "v_add_f32 v3, 1.0, v3\n\t"
      "v_rcp_f32 v0, v0\n\t"
      "v_rcp_f32 v1, v1\n\t"
      "v_rcp_f32 v2, v2\n\t"
      "v_rcp_f32 v3, v3\n\t"
      "v_mul_f32 v60, v60, v0\n\t"
      "v_mul_f32 v61, v61, v1\n\t"
      "v_mul_f32 v62, v62, v2\n\t"
      "v_mul_f32 v63, v63, v3\n\t"
      "ds_write_b128 v254, v[60:63] offset:2048\n\t"
      "v_xor_b32 v254, 64, v254\n\t"
      "ds_read_b128 v[56:59], v252 offset:2048\n\t"
      "ds_read_b128 v[60:63], v252 offset:3072\n\t"
      "s_add_u32 s70, s76, 0x80000\n\t"
      "s_waitcnt lgkmcnt(4)\n\t"
      "buffer_store_dwordx4 v[24:27], v255, s[64:67], s76 offen offset:384 sc1\n\t"
      "buffer_store_dwordx4 v[28:31], v255, s[64:67], s70 offen offset:384 sc1\n\t"
      "v_cvt_f32_i32 v88, v88\n\t"
      "v_cvt_f32_i32 v89, v89\n\t"
      "v_cvt_f32_i32 v90, v90\n\t"
      "v_cvt_f32_i32 v91, v91\n\t"
      "v_mul_f32 v0, s73, v88\n\t"
      "v_mul_f32 v1, s73, v89\n\t"
      "v_mul_f32 v2, s73, v90\n\t"
      "v_mul_f32 v3, s73, v91\n\t"
      "v_med3_f32 v0, v0, -s74, s74\n\t"
      "v_med3_f32 v1, v1, -s74, s74\n\t"
      "v_med3_f32 v2, v2, -s74, s74\n\t"
      "v_med3_f32 v3, v3, -s74, s74\n\t"
      "v_exp_f32 v0, v0\n\t"
      "v_exp_f32 v1, v1\n\t"
      "v_exp_f32 v2, v2\n\t"
      "v_exp_f32 v3, v3\n\t"
      "v_mul_f32 v88, s72, v88\n\t"
      "v_mul_f32 v89, s72, v89\n\t"
      "v_mul_f32 v90, s72, v90\n\t"
      "v_mul_f32 v91, s72, v91\n\t"
      "v_add_f32 v0, 1.0, v0\n\t"
      "v_add_f32 v1, 1.0, v1\n\t"
      "v_add_f32 v2, 1.0, v2\n\t"
      "v_add_f32 v3, 1.0, v3\n\t"
      "v_rcp_f32 v0, v0\n\t"
      "v_rcp_f32 v1, v1\n\t"
      "v_rcp_f32 v2, v2\n\t"
      "v_rcp_f32 v3, v3\n\t"
      "v_mul_f32 v88, v88, v0\n\t"
      "v_mul_f32 v89, v89, v1\n\t"
      "v_mul_f32 v90, v90, v2\n\t"
      "v_mul_f32 v91, v91, v3\n\t"
      "ds_write_b128 v254, v[88:91] offset:0\n\t"
      "v_xor_b32 v254, 64, v254\n\t"
      "v_cvt_f32_i32 v92, v92\n\t"
      "v_cvt_f32_i32 v93, v93\n\t"
      "v_cvt_f32_i32 v94, v94\n\t"
      "v_cvt_f32_i32 v95, v95\n\t"
      "v_mul_f32 v0, s73, v92\n\t"
      "v_mul_f32 v1, s73, v93\n\t"
      "v_mul_f32 v2, s73, v94\n\t"
      "v_mul_f32 v3, s73, v95\n\t"
      "v_med3_f32 v0, v0, -s74, s74\n\t"
      "v_med3_f32 v1, v1, -s74, s74\n\t"
      "v_med3_f32 v2, v2, -s74, s74\n\t"
      "v_med3_f32 v3, v3, -s74, s74\n\t"
      "v_exp_f32 v0, v0\n\t"
      "v_exp_f32 v1, v1\n\t"
      "v_exp_f32 v2, v2\n\t"
      "v_exp_f32 v3, v3\n\t"
      "v_mul_f32 v92, s72, v92\n\t"
      "v_mul_f32 v93, s72, v93\n\t"
      "v_mul_f32 v94, s72, v94\n\t"
      "v_mul_f32 v95, s72, v95\n\t"
      "v_add_f32 v0, 1.0, v0\n\t"
      "v_add_f32 v1, 1.0, v1\n\t"
      "v_add_f32 v2, 1.0, v2\n\t"
      "v_add_f32 v3, 1.0, v3\n\t"
      "v_rcp_f32 v0, v0\n\t"
      "v_rcp_f32 v1, v1\n\t"
      "v_rcp_f32 v2, v2\n\t"
      "v_rcp_f32 v3, v3\n\t"
      "v_mul_f32 v92, v92, v0\n\t"
      "v_mul_f32 v93, v93, v1\n\t"
      "v_mul_f32 v94, v94, v2\n\t"
      "v_mul_f32 v95, v95, v3\n\t"
      "ds_write_b128 v254, v[92:95] offset:0\n\t"
      "v_xor_b32 v254, 64, v254\n\t"
      "ds_read_b128 v[88:91], v252 offset:0\n\t"
      "ds_read_b128 v[92:95], v252 offset:1024\n\t"
      "s_add_u32 s70, s77, 0x80000\n\t"
      "s_waitcnt lgkmcnt(4)\n\t"
      "buffer_store_dwordx4 v[56:59], v255, s[64:67], s77 offen offset:384 sc1\n\t"
      "buffer_store_dwordx4 v[60:63], v255, s[64:67], s70 offen offset:384 sc1\n\t"
      "v_cvt_f32_i32 v120, v120\n\t"
      "v_cvt_f32_i32 v121, v121\n\t"
      "v_cvt_f32_i32 v122, v122\n\t"
      "v_cvt_f32_i32 v123, v123\n\t"
      "v_mul_f32 v0, s73, v120\n\t"
      "v_mul_f32 v1, s73, v121\n\t"
      "v_mul_f32 v2, s73, v122\n\t"
      "v_mul_f32 v3, s73, v123\n\t"
      "v_med3_f32 v0, v0, -s74, s74\n\t"
      "v_med3_f32 v1, v1, -s74, s74\n\t"
      "v_med3_f32 v2, v2, -s74, s74\n\t"
      "v_med3_f32 v3, v3, -s74, s74\n\t"
      "v_exp_f32 v0, v0\n\t"
      "v_exp_f32 v1, v1\n\t"
      "v_exp_f32 v2, v2\n\t"
      "v_exp_f32 v3, v3\n\t"
      "v_mul_f32 v120, s72, v120\n\t"
      "v_mul_f32 v121, s72, v121\n\t"
      "v_mul_f32 v122, s72, v122\n\t"
      "v_mul_f32 v123, s72, v123\n\t"
      "v_add_f32 v0, 1.0, v0\n\t"
      "v_add_f32 v1, 1.0, v1\n\t"
      "v_add_f32 v2, 1.0, v2\n\t"
      "v_add_f32 v3, 1.0, v3\n\t"
      "v_rcp_f32 v0, v0\n\t"
      "v_rcp_f32 v1, v1\n\t"
      "v_rcp_f32 v2, v2\n\t"
      "v_rcp_f32 v3, v3\n\t"
      "v_mul_f32 v120, v120, v0\n\t"
      "v_mul_f32 v121, v121, v1\n\t"
      "v_mul_f32 v122, v122, v2\n\t"
      "v_mul_f32 v123, v123, v3\n\t"
      "ds_write_b128 v254, v[120:123] offset:2048\n\t"
      "v_xor_b32 v254, 64, v254\n\t"
      "v_cvt_f32_i32 v124, v124\n\t"
      "v_cvt_f32_i32 v125, v125\n\t"
      "v_cvt_f32_i32 v126, v126\n\t"
      "v_cvt_f32_i32 v127, v127\n\t"
      "v_mul_f32 v0, s73, v124\n\t"
      "v_mul_f32 v1, s73, v125\n\t"
      "v_mul_f32 v2, s73, v126\n\t"
      "v_mul_f32 v3, s73, v127\n\t"
      "v_med3_f32 v0, v0, -s74, s74\n\t"
      "v_med3_f32 v1, v1, -s74, s74\n\t"
      "v_med3_f32 v2, v2, -s74, s74\n\t"
      "v_med3_f32 v3, v3, -s74, s74\n\t"
      "v_exp_f32 v0, v0\n\t"
      "v_exp_f32 v1, v1\n\t"
      "v_exp_f32 v2, v2\n\t"
      "v_exp_f32 v3, v3\n\t"
      "v_mul_f32 v124, s72, v124\n\t"
      "v_mul_f32 v125, s72, v125\n\t"
      "v_mul_f32 v126, s72, v126\n\t"
      "v_mul_f32 v127, s72, v127\n\t"
      "v_add_f32 v0, 1.0, v0\n\t"
      "v_add_f32 v1, 1.0, v1\n\t"
      "v_add_f32 v2, 1.0, v2\n\t"
      "v_add_f32 v3, 1.0, v3\n\t"
      "v_rcp_f32 v0, v0\n\t"
      "v_rcp_f32 v1, v1\n\t"
      "v_rcp_f32 v2, v2\n\t"
      "v_rcp_f32 v3, v3\n\t"
      "v_mul_f32 v124, v124, v0\n\t"
      "v_mul_f32 v125, v125, v1\n\t"
      "v_mul_f32 v126, v126, v2\n\t"
      "v_mul_f32 v127, v127, v3\n\t"
      "ds_write_b128 v254, v[124:127] offset:2048\n\t"
      "v_xor_b32 v254, 64, v254\n\t"
      "ds_read_b128 v[120:123], v252 offset:2048\n\t"
      "ds_read_b128 v[124:127], v252 offset:3072\n\t"
      "s_add_u32 s70, s78, 0x80000\n\t"
      "s_waitcnt lgkmcnt(4)\n\t"
      "buffer_store_dwordx4 v[88:91], v255, s[64:67], s78 offen offset:384 sc1\n\t"
      "buffer_store_dwordx4 v[92:95], v255, s[64:67], s70 offen offset:384 sc1\n\t"
      "s_add_u32 s70, s79, 0x80000\n\t"
      "s_waitcnt lgkmcnt(0)\n\t"
      "buffer_store_dwordx4 v[120:123], v255, s[64:67], s79 offen offset:384 sc1\n\t"
      "buffer_store_dwordx4 v[124:127], v255, s[64:67], s70 offen offset:384 sc1\n\t"
      "s_branch Lend_%=\n\t"
      "Lslow0_%=:\n\t"
      "s_memrealtime s[54:55]\n\t"
      "s_waitcnt lgkmcnt(0)\n\t"
      "Lspin0_%=:\n\t"
      "s_sleep 1\n\t"
      "ds_read_addtid_b32 v253 offset:0\n\t"
      "s_memrealtime s[56:57]\n\t"
      "s_waitcnt lgkmcnt(0)\n\t"
      "v_readfirstlane_b32 s53, v253\n\t"
      "s_or_b32 s49, s53, 0x808080\n\t"
      "s_sub_u32 s49, s49, s48\n\t"
      "s_and_b32 s49, s49, 0x808080\n\t"
      "s_cmp_eq_u32 s49, 0x808080\n\t"
      "s_cbranch_scc1 Lback0_%=\n\t"
      "s_sub_u32 s49, s56, s54\n\t"
      "s_cmp_lt_u32 s49, s50\n\t"
      "s_cbranch_scc1 Lspin0_%=\n\t"
      "s_mov_b32 s50, 0\n\t"
      "s_branch Lback0_%=\n\t"
      "Lslow1_%=:\n\t"
      "s_memrealtime s[54:55]\n\t"
      "s_waitcnt lgkmcnt(0)\n\t"
      "Lspin1_%=:\n\t"
      "s_sleep 1\n\t"
      "ds_read_addtid_b32 v253 offset:0\n\t"
      "s_memrealtime s[56:57]\n\t"
      "s_waitcnt lgkmcnt(0)\n\t"
      "v_readfirstlane_b32 s53, v253\n\t"
      "s_or_b32 s49, s53, 0x808080\n\t"
      "s_sub_u32 s49, s49, s48\n\t"
      "s_and_b32 s49, s49, 0x808080\n\t"
      "s_cmp_eq_u32 s49, 0x808080\n\t"
      "s_cbranch_scc1 Lback1_%=\n\t"
      "s_sub_u32 s49, s56, s54\n\t"
      "s_cmp_lt_u32 s49, s50\n\t"
      "s_cbranch_scc1 Lspin1_%=\n\t"
      "s_mov_b32 s50, 0\n\t"
      "s_branch Lback1_%=\n\t"
      "Lslow2_%=:\n\t"
      "s_memrealtime s[54:55]\n\t"
      "s_waitcnt lgkmcnt(0)\n\t"
      "Lspin2_%=:\n\t"
      "s_sleep 1\n\t"
      "ds_read_addtid_b32 v253 offset:0\n\t"
      "s_memrealtime s[56:57]\n\t"
      "s_waitcnt lgkmcnt(0)\n\t"
      "v_readfirstlane_b32 s53, v253\n\t"
      "s_or_b32 s49, s53, 0x808080\n\t"
      "s_sub_u32 s49, s49, s48\n\t"
      "s_and_b32 s49, s49, 0x808080\n\t"
      "s_cmp_eq_u32 s49, 0x808080\n\t"
      "s_cbranch_scc1 Lback2_%=\n\t"
      "s_sub_u32 s49, s56, s54\n\t"
      "s_cmp_lt_u32 s49, s50\n\t"
      "s_cbranch_scc1 Lspin2_%=\n\t"
      "s_mov_b32 s50, 0\n\t"
      "s_branch Lback2_%=\n\t"
      "Lslow3_%=:\n\t"
      "s_memrealtime s[54:55]\n\t"
      "s_waitcnt lgkmcnt(0)\n\t"
      "Lspin3_%=:\n\t"
      "s_sleep 1\n\t"
      "ds_read_addtid_b32 v253 offset:0\n\t"
      "s_memrealtime s[56:57]\n\t"
      "s_waitcnt lgkmcnt(0)\n\t"
      "v_readfirstlane_b32 s53, v253\n\t"
      "s_or_b32 s49, s53, 0x808080\n\t"
      "s_sub_u32 s49, s49, s48\n\t"
      "s_and_b32 s49, s49, 0x808080\n\t"
      "s_cmp_eq_u32 s49, 0x808080\n\t"
      "s_cbranch_scc1 Lback3_%=\n\t"
      "s_sub_u32 s49, s56, s54\n\t"
      "s_cmp_lt_u32 s49, s50\n\t"
      "s_cbranch_scc1 Lspin3_%=\n\t"
      "s_mov_b32 s50, 0\n\t"
      "s_branch Lback3_%=\n\t"
      "Lslow4_%=:\n\t"
      "s_memrealtime s[54:55]\n\t"
      "s_waitcnt lgkmcnt(0)\n\t"
      "Lspin4_%=:\n\t"
      "s_sleep 1\n\t"
      "ds_read_addtid_b32 v253 offset:0\n\t"
      "s_memrealtime s[56:57]\n\t"
      "s_waitcnt lgkmcnt(0)\n\t"
      "v_readfirstlane_b32 s53, v253\n\t"
      "s_or_b32 s49, s53, 0x808080\n\t"
      "s_sub_u32 s49, s49, s48\n\t"
      "s_and_b32 s49, s49, 0x808080\n\t"
      "s_cmp_eq_u32 s49, 0x808080\n\t"
      "s_cbranch_scc1 Lback4_%=\n\t"
      "s_sub_u32 s49, s56, s54\n\t"
      "s_cmp_lt_u32 s49, s50\n\t"
      "s_cbranch_scc1 Lspin4_%=\n\t"
      "s_mov_b32 s50, 0\n\t"
      "s_branch Lback4_%=\n\t"
      "Lslow5_%=:\n\t"
      "s_memrealtime s[54:55]\n\t"
      "s_waitcnt lgkmcnt(0)\n\t"
      "Lspin5_%=:\n\t"
      "s_sleep 1\n\t"
      "ds_read_addtid_b32 v253 offset:0\n\t"
      "s_memrealtime s[56:57]\n\t"
      "s_waitcnt lgkmcnt(0)\n\t"
      "v_readfirstlane_b32 s53, v253\n\t"
      "s_or_b32 s49, s53, 0x808080\n\t"
      "s_sub_u32 s49, s49, s48\n\t"
      "s_and_b32 s49, s49, 0x808080\n\t"
      "s_cmp_eq_u32 s49, 0x808080\n\t"
      "s_cbranch_scc1 Lback5_%=\n\t"
      "s_sub_u32 s49, s56, s54\n\t"
      "s_cmp_lt_u32 s49, s50\n\t"
      "s_cbranch_scc1 Lspin5_%=\n\t"
      "s_mov_b32 s50, 0\n\t"
      "s_branch Lback5_%=\n\t"
      "Lslow6_%=:\n\t"
      "s_memrealtime s[54:55]\n\t"
      "s_waitcnt lgkmcnt(0)\n\t"
      "Lspin6_%=:\n\t"
      "s_sleep 1\n\t"
      "ds_read_addtid_b32 v253 offset:0\n\t"
      "s_memrealtime s[56:57]\n\t"
      "s_waitcnt lgkmcnt(0)\n\t"
      "v_readfirstlane_b32 s53, v253\n\t"
      "s_or_b32 s49, s53, 0x808080\n\t"
      "s_sub_u32 s49, s49, s48\n\t"
      "s_and_b32 s49, s49, 0x808080\n\t"
      "s_cmp_eq_u32 s49, 0x808080\n\t"
      "s_cbranch_scc1 Lback6_%=\n\t"
      "s_sub_u32 s49, s56, s54\n\t"
      "s_cmp_lt_u32 s49, s50\n\t"
      "s_cbranch_scc1 Lspin6_%=\n\t"
      "s_mov_b32 s50, 0\n\t"
      "s_branch Lback6_%=\n\t"
      "Lslow7_%=:\n\t"
      "s_memrealtime s[54:55]\n\t"
      "s_waitcnt lgkmcnt(0)\n\t"
      "Lspin7_%=:\n\t"
      "s_sleep 1\n\t"
      "ds_read_addtid_b32 v253 offset:0\n\t"
      "s_memrealtime s[56:57]\n\t"
      "s_waitcnt lgkmcnt(0)\n\t"
      "v_readfirstlane_b32 s53, v253\n\t"
      "s_or_b32 s49, s53, 0x808080\n\t"
      "s_sub_u32 s49, s49, s48\n\t"
      "s_and_b32 s49, s49, 0x808080\n\t"
      "s_cmp_eq_u32 s49, 0x808080\n\t"
      "s_cbranch_scc1 Lback7_%=\n\t"
      "s_sub_u32 s49, s56, s54\n\t"
      "s_cmp_lt_u32 s49, s50\n\t"
      "s_cbranch_scc1 Lspin7_%=\n\t"
      "s_mov_b32 s50, 0\n\t"
      "s_branch Lback7_%=\n\t"
      "Lslow8_%=:\n\t"
      "s_memrealtime s[54:55]\n\t"
      "s_waitcnt lgkmcnt(0)\n\t"
      "Lspin8_%=:\n\t"
      "s_sleep 1\n\t"
      "ds_read_addtid_b32 v253 offset:0\n\t"
      "s_memrealtime s[56:57]\n\t"
      "s_waitcnt lgkmcnt(0)\n\t"
      "v_readfirstlane_b32 s53, v253\n\t"
      "s_or_b32 s49, s53, 0x808080\n\t"
      "s_sub_u32 s49, s49, s48\n\t"
      "s_and_b32 s49, s49, 0x808080\n\t"
      "s_cmp_eq_u32 s49, 0x808080\n\t"
      "s_cbranch_scc1 Lback8_%=\n\t"
      "s_sub_u32 s49, s56, s54\n\t"
      "s_cmp_lt_u32 s49, s50\n\t"
      "s_cbranch_scc1 Lspin8_%=\n\t"
      "s_mov_b32 s50, 0\n\t"
      "s_branch Lback8_%=\n\t"
      "Lslow9_%=:\n\t"
      "s_memrealtime s[54:55]\n\t"
      "s_waitcnt lgkmcnt(0)\n\t"
      "Lspin9_%=:\n\t"
      "s_sleep 1\n\t"
      "ds_read_addtid_b32 v253 offset:0\n\t"
      "s_memrealtime s[56:57]\n\t"
      "s_waitcnt lgkmcnt(0)\n\t"
      "v_readfirstlane_b32 s53, v253\n\t"
      "s_or_b32 s49, s53, 0x808080\n\t"
      "s_sub_u32 s49, s49, s48\n\t"
      "s_and_b32 s49, s49, 0x808080\n\t"
      "s_cmp_eq_u32 s49, 0x808080\n\t"
      "s_cbranch_scc1 Lback9_%=\n\t"
      "s_sub_u32 s49, s56, s54\n\t"
      "s_cmp_lt_u32 s49, s50\n\t"
      "s_cbranch_scc1 Lspin9_%=\n\t"
      "s_mov_b32 s50, 0\n\t"
      "s_branch Lback9_%=\n\t"
      "Lend_%=:\n\t"
      : "+{v248}"(ad[0]), "+{v249}"(ad[1]), "+{v250}"(ad[2]), "+{v251}"(ad[3]), "+{v252}"(vl), "+{v253}"(vs), "+{v254}"(vx), "+{v255}"(vt), "+{s[40:43]}"(rw)
      : "{s[36:39]}"(rx), "{s[64:67]}"(ro), "{s[20:23]}"(xso), "{s75}"(stg), "{s[76:79]}"(oso), "{s52}"(lds0 + w * 256), [dl] "i"(2 * DEADLINE_TICKS)
      : "memory", "scc", "vcc", "s44", "s45", "s46", "s47", "s48", "s49", "s50", "s51", "s53", "s54", "s55", "s56", "s57",
        "s60", "s61", "s62", "s63", "s70", "s72", "s73", "s74",
        "v0", "v1", "v2", "v3", "v4", "v5", "v6", "v7", "v8", "v9", "v10", "v11", "v12", "v13", "v14", "v15", "v16", "v17", "v18", "v19", "v20", "v21", "v22", "v23",
        "v24", "v25", "v26", "v27", "v28", "v29", "v30", "v31", "v32", "v33", "v34", "v35", "v36", "v37", "v38", "v39", "v40", "v41", "v42", "v43", "v44", "v45", "v46", "v47",
        "v48", "v49", "v50", "v51", "v52", "v53", "v54", "v55", "v56", "v57", "v58", "v59", "v60", "v61", "v62", "v63", "v64", "v65", "v66", "v67", "v68", "v69", "v70", "v71",
        "v72", "v73", "v74", "v75", "v76", "v77", "v78", "v79", "v80", "v81", "v82", "v83", "v84", "v85", "v86", "v87", "v88", "v89", "v90", "v91", "v92", "v93", "v94", "v95",
        "v96", "v97", "v98", "v99", "v100", "v101", "v102", "v103", "v104", "v105", "v106", "v107", "v108", "v109", "v110", "v111", "v112", "v113", "v114", "v115", "v116", "v117", "v118", "v119",
        "v120", "v121", "v122", "v123", "v124", "v125", "v126", "v127", "v128", "v129", "v130", "v131", "v132", "v133", "v134", "v135", "v136", "v137", "v138", "v139", "v140", "v141", "v142", "v143",
        "v144", "v145", "v146", "v147", "v148", "v149", "v150", "v151", "v152", "v153", "v154", "v155", "v156", "v157", "v158", "v159", "v160", "v161", "v162", "v163", "v164", "v165", "v166", "v167",
        "v168", "v169", "v170", "v171", "v172", "v173", "v174", "v175", "v176", "v177", "v178", "v179", "v180", "v181", "v182", "v183", "v184", "v185", "v186", "v187", "v188", "v189", "v190", "v191",
        "v192", "v193", "v194", "v195", "v196", "v197", "v198", "v199", "v200", "v201", "v202", "v203", "v204", "v205", "v206", "v207", "v208", "v209", "v210", "v211", "v212", "v213", "v214", "v215",
        "v216", "v217", "v218", "v219", "v220", "v221", "v222", "v223", "v224", "v225", "v226", "v227", "v228", "v229", "v230", "v231", "v232", "v233", "v234", "v235", "v236", "v237", "v238", "v239",
        "v240", "v241", "v242", "v243", "v244", "v245", "v246", "v247"
    );
  } else if (w < 6) {
    const int h = w - 4;
    const v4i rs = rsrc(pw, 64u << 20);
    const int q = l >> 3;
    const unsigned voff = q * 16384 + (((l & 7) ^ ((q >> 1) & 3)) << 4);
    const unsigned v0 = voff, v1 = voff + 8 * 16384 - 1024, v2 = voff + 16 * 16384 - 2112, v3 = voff + 24 * 16384 - 3136;
    const unsigned vtab = kblock(2 * (l & 31) + h, rank, rot) * (64 * 16384) + tn * 128;
    volatile LAS unsigned* prog = (volatile LAS unsigned*)(smem + 1024 + (l & 3) * 256);
    volatile LAS unsigned char* pub = (volatile LAS unsigned char*)(smem + (l & 3) * 256 + h * 2);
    volatile LAS unsigned char* other = (volatile LAS unsigned char*)(smem + (h ^ 1) * 2);
    const unsigned base = lds0 + SYNC_BYTES + h * 4 * 2112;
    unsigned sl = 0;
#define LD_ISSUE(n, SL)                                                                                     \
    do {                                                                                                    \
      unsigned so = __builtin_amdgcn_readlane(vtab, n);                                                     \
      asm volatile("s_add_u32 m0, %[base], %[sl]\n\t"                                                       \
                   "s_add_u32 %[so], %[so], 0\n\t"                                                          \
                   "buffer_load_dwordx4 %[v0], %[rs], %[so] offen" WAUX " lds\n\t" SL                       \
                   "buffer_load_dwordx4 %[v1], %[rs], %[so] offen offset:1024" WAUX " lds\n\t" SL           \
                   "buffer_load_dwordx4 %[v2], %[rs], %[so] offen offset:2112" WAUX " lds\n\t" SL           \
                   "buffer_load_dwordx4 %[v3], %[rs], %[so] offen offset:3136" WAUX " lds\n\t" SL           \
                   "s_add_u32 m0, m0, 4224\n\t"                                                             \
                   "s_add_u32 %[so], %[so], 0x80000\n\t"                                                    \
                   "buffer_load_dwordx4 %[v0], %[rs], %[so] offen" WAUX " lds\n\t" SL                       \
                   "buffer_load_dwordx4 %[v1], %[rs], %[so] offen offset:1024" WAUX " lds\n\t" SL           \
                   "buffer_load_dwordx4 %[v2], %[rs], %[so] offen offset:2112" WAUX " lds\n\t" SL           \
                   "buffer_load_dwordx4 %[v3], %[rs], %[so] offen offset:3136" WAUX " lds\n\t"              \
                   : [so] "+s"(so)                                                                          \
                   : [base] "s"(base), [sl] "s"(sl), [rs] "s"(rs), [v0] "v"(v0), [v1] "v"(v1), [v2] "v"(v2), [v3] "v"(v3) \
                   : "memory", "scc");                                                                      \
      sl += SLOT; if (sl == RING * SLOT) sl = 0;                                                            \
    } while (0)
#define VMWAIT(n) asm volatile("s_waitcnt vmcnt(" #n ")" ::: "memory")
    const bool slow = SPF && slow_line((unsigned)(pw >> 7) + tn);
    const bool lead = h == tm;
    __builtin_amdgcn_s_setprio(2);
    LREC(0);
    LD_ISSUE(0, "");
    if (lead) {
      LD_ISSUE(1, ""); LD_ISSUE(2, "");
      LREC(1);
      VMWAIT(16); *pub = 1; VMWAIT(8); *pub = 2; VMWAIT(0); *pub = 3;
      LREC(2);
      __builtin_amdgcn_s_sleep(L34_SLEEP);
      LD_ISSUE(3, ""); LD_ISSUE(4, "");
      VMWAIT(8); *pub = 4; VMWAIT(0); *pub = 5;
    } else {
      VMWAIT(0); *pub = 1;
      __builtin_amdgcn_s_setprio(1);
      LD_ISSUE(1, ""); LD_ISSUE(2, "");
      LREC(1);
      VMWAIT(8); *pub = 2; VMWAIT(0); *pub = 3;
      LREC(2);
      __builtin_amdgcn_s_sleep(T34_SLEEP);
      LD_ISSUE(3, ""); LD_ISSUE(4, "");
      VMWAIT(8); *pub = 4; VMWAIT(0); *pub = 5;
    }
    int n = 5;
    const unsigned long long t0 = __builtin_amdgcn_s_memrealtime();
    __builtin_amdgcn_s_setprio(LPRIO);
    while (n < 32) {
      const bool pair = lead && !slow;
      int cl = n < RING ? 2 : CL; if (cl > 32 - n) cl = 32 - n;
      int need = pair ? n + cl - 1 - RING + (CL == 2 ? tm : (tm * (CL / 2) + ((jn >> 3) & (CL / 4)))) : n - THR_B;
      if (need > 25) need = 25;
      if (n >= 5 && need < 1) need = 1;
      if (n >= 7 && need < 2) need = 2;
      const int oth = (lead || slow) ? 0 : n + 1;
      bool waited = false;
      if (!__all((int)*prog > need && (int)*other >= oth)) {
        VMWAIT(0);
        *pub = (unsigned char)n;
        waited = true;
        while (!__all((int)*prog > need && (int)*other >= oth)) {
          __builtin_amdgcn_s_sleep(1);
          if (__builtin_amdgcn_s_memrealtime() - t0 > 3ull * DEADLINE_TICKS) { n = 64; break; }
        }
        if (n == 64) break;
      }
      if (pair) {
        if (STAGGER && waited) for (int i = 0; i < (jn & (CL == 4 ? 7 : 15)); i++) __builtin_amdgcn_s_sleep(CL == 4 ? 2 : 1);
        for (int i = 0; i < cl; i++) { LD_ISSUE(n, ""); n++; }
        VMWAIT(8); *pub = (unsigned char)(n - 1); VMWAIT(0); *pub = (unsigned char)n;
      }
      else {
        if (waited && n >= RING) LD_ISSUE(n, "s_sleep 1\n\t"); else LD_ISSUE(n, "");
        VMWAIT(8); *pub = (unsigned char)n; n += 1;
      }
    }
    VMWAIT(0);
    if (n == 32) *pub = 32;
  } else if (w == 6) {
    __builtin_amdgcn_s_setprio(3);
    const unsigned long long t0 = __builtin_amdgcn_s_memrealtime();
    const int cj = b >> 6, rb = (b >> 3) & 7;
    __amdgpu_buffer_rsrc_t rx = __builtin_amdgcn_make_buffer_rsrc((void*)px, 0, 4096 * 512, 0x00020000);
    __amdgpu_buffer_rsrc_t rd = __builtin_amdgcn_make_buffer_rsrc((void*)pd, 0, FLAG_OFF + 16384, 0x00020000);
    unsigned call = 0;
    v4i d[8];
    auto fetch = [&] {
      asm volatile("global_atomic_add %0, %1, %2, %3 sc0" : "=v"(call) : "v"(l * 4), "v"(1), "s"(myws + CTR_OFF + b * 512) : "memory");
#pragma unroll
      for (int f = 0; f < 8; f++)
        asm volatile("buffer_load_dwordx4 %0, %1, %2, %3 offen" : "=v"(d[f]) : "v"((l & 15) * 512 + (l >> 4) * 16), "s"(rx),
                     "s"((dest * 512 + rb * 64 + (f & 3) * 16) * 512 + cj * 128 + (f >> 2) * 64) : "memory");
    };
    const bool push = !NOPUSH && dest != (unsigned)rank;
    if (push && cj == 0) fetch();
    const unsigned idx0 = (unsigned)(pw >> 7) + (b & 7) * 16;
    const unsigned f0 = (3u - idx0) & 7u, f2 = (63u - idx0) & 127u;
    const unsigned vrow = (kblock(2 * (l & 31) + (b >> 7), rank, 0) * 64 + ((b >> 3) & 15) * 4) * 16384 + (b & 7) * 2048;
    int pf = SPF ? SPF_FIRST : 32;
    auto prefetch = [&](int cnt) {
      for (; cnt > 0 && pf < 32; cnt--, pf++) {
        const unsigned long long p = pw + __builtin_amdgcn_readlane(vrow, pf);
#define PF4(P) asm volatile("s_load_dwordx2 s[58:59], %0, 60\n\ts_load_dwordx2 s[58:59], %0, 16444\n\ts_load_dwordx2 s[58:59], %0, 32828\n\ts_load_dwordx2 s[58:59], %0, 49212" ::"s"(P) : "s58", "s59")
        PF4(p + f0 * 128);
        PF4(p + f0 * 128 + 1024);
        if (f2 < 16) PF4(p + f2 * 128);
      }
    };
    if (push) {
      if (cj > 0) {
        const unsigned tload = cj * 2 * PACE_TICKS - 40u;
        while ((unsigned)(__builtin_amdgcn_s_memrealtime() - t0) < tload) { if (pf < 32) prefetch(1); else __builtin_amdgcn_s_sleep(2); }
        fetch();
        prefetch(1);
      }
      asm volatile("s_waitcnt vmcnt(4)" : "+v"(call), "+v"(d[0]), "+v"(d[1]), "+v"(d[2]), "+v"(d[3])::"memory");
      call = __builtin_amdgcn_readfirstlane(call) + 1u;
      const unsigned so = (call & 1u) * XBUF_BYTES + (unsigned)(rank * 8 + cj * 2) * 32768u + rb * 4096;
      const unsigned fo = FLAG_OFF + ((rank * 8 + cj * 2) * 2 + (rb >> 2)) * 128 + (rb & 3) * 4;
#define PUSH(f, o) asm volatile("buffer_store_dwordx4 %0, %1, %2, %3 offen offset:%4 sc0 sc1" ::"v"(d[f]), "v"(l * 16), "s"(rd), "s"(o), "i"((f & 3) * 1024) : "memory")
      PUSH(0, so); PUSH(1, so); PUSH(2, so); PUSH(3, so);
      const unsigned long long t1 = __builtin_amdgcn_s_memrealtime();
      if (cj > 0) prefetch(1);
      while ((unsigned)(__builtin_amdgcn_s_memrealtime() - t1) < ODD_TICKS) __builtin_amdgcn_s_sleep(1);
      asm volatile("s_waitcnt vmcnt(4)" : "+v"(d[4]), "+v"(d[5]), "+v"(d[6]), "+v"(d[7])::"memory");
      PUSH(4, so + 32768u); PUSH(5, so + 32768u); PUSH(6, so + 32768u); PUSH(7, so + 32768u);
      asm volatile("s_waitcnt vmcnt(4)" ::: "memory");
      if (l == 0) asm volatile("buffer_store_dword %0, off, %1, %2 sc0 sc1" ::"v"(call), "s"(rd), "s"(fo) : "memory");
      if (cj > 0) prefetch(1);
      asm volatile("s_waitcnt vmcnt(1)" ::: "memory");
      if (l == 0) asm volatile("buffer_store_dword %0, off, %1, %2 sc0 sc1" ::"v"(call), "s"(rd), "s"(fo + 256) : "memory");
    }
    __builtin_amdgcn_s_setprio(0);
    prefetch(32);
    asm volatile("s_waitcnt lgkmcnt(0)" ::: "memory");
  } else {
    volatile LAS unsigned* sync = (volatile LAS unsigned*)smem;
    __builtin_amdgcn_s_setprio(3);
    const unsigned long long tk = __builtin_amdgcn_s_memrealtime();
    unsigned call, touch = 0;
    asm volatile("global_atomic_add %0, %1, %2, %3 sc0" : "=v"(call) : "v"(l * 4), "v"(1), "s"(myws + CTR_OFF + b * 512 + 256) : "memory");
    const float sc = a.sx[0] * a.sw[0];
    asm volatile("s_waitcnt vmcnt(0)" : "+v"(call), "+v"(touch)::"memory");
    call = __builtin_amdgcn_readfirstlane(call) + 1u;
    if (l < 4) {
      sync[l * 64 + 1] = __builtin_bit_cast(unsigned, sc);
      sync[l * 64 + 2] = __builtin_bit_cast(unsigned, sc * -1.4426950408889634f);
      ((volatile LAS unsigned char*)smem)[l * 256 + 3] = (unsigned char)(call & 1u);
    }
    const unsigned id = kblock(8 + (l < 56 ? l : 0), rank, 0);
    __amdgpu_buffer_rsrc_t rf = __builtin_amdgcn_make_buffer_rsrc(myws + FLAG_OFF, 0, 16384, 0x00020000);
    unsigned* tmo = (unsigned*)(myws + TMO_OFF);
    const unsigned long long t0 = __builtin_amdgcn_s_memrealtime();
    unsigned pub = 4, polls = 0, cnt = 0;
    bool ok = l >= 56;
    while ((unsigned)(__builtin_amdgcn_s_memrealtime() - tk) < POLL_START) __builtin_amdgcn_s_sleep(2);
    while (true) {
      if (!ok && (unsigned)l < cnt + POLL_AHEAD) {
        v4i v;
        asm volatile("buffer_load_dwordx4 %0, %1, %2, 0 offen sc0 sc1\n\ts_waitcnt vmcnt(0)" : "=&v"(v) : "v"(id * 256 + tm * 128), "s"(rf) : "memory");
        ok = (int)((v.x - call) | (v.y - call) | (v.z - call) | (v.w - call)) >= 0;
      }
      const unsigned long long miss = ~__builtin_amdgcn_ballot_w64(ok) | (1ull << 56);
      cnt = __builtin_ctzll(miss);
      unsigned ready = 4 + (cnt >> 1);
      bool bad = false;
      if (ready < 32 && (++polls & 63u) == 0u) {
        bad = __builtin_amdgcn_readfirstlane(__hip_atomic_load(tmo, __ATOMIC_RELAXED, __HIP_MEMORY_SCOPE_AGENT)) == call ||
              __builtin_amdgcn_s_memrealtime() - t0 > DEADLINE_TICKS;
        if (bad) {
          if (l == 0) __hip_atomic_store(tmo, call, __ATOMIC_RELAXED, __HIP_MEMORY_SCOPE_AGENT);
          if (l < 4) ((volatile LAS unsigned char*)smem)[l * 256 + 3] = (unsigned char)((call & 1u) | 2u);
          ready = 32;
        }
      }
      if (ready > pub) {
        pub = ready;
        if (l < 4) ((volatile LAS unsigned char*)smem)[l * 256 + 1] = (unsigned char)ready;
      }
      if (ready >= 32) break;
      __builtin_amdgcn_s_sleep(POLL_SLEEP);
    }
  }
}

extern "C" void kernel_launch(int rank, int nranks, ncclComm_t comm, hipStream_t stream, void* const* d_in, const int* in_sizes, int n_in,
                              void* d_out, int out_size, const int* in_dtypes, int out_dtype, void* const* d_out_all,
                              void* const* d_ws_all, size_t ws_bytes) {
  (void)comm; (void)in_sizes; (void)in_dtypes; (void)out_dtype; (void)d_out_all;
  if (nranks != NR || (unsigned)rank >= (unsigned)NR || n_in < 4 || out_size != 512 * 16384 || ws_bytes < WS_USED) {
    fprintf(stderr, "kernel_launch: unexpected configuration\n");
    return;
  }
  hipError_t e = hipFuncSetAttribute((const void*)a2a_gemm_kernel, hipFuncAttributeMaxDynamicSharedMemorySize, LDS_BYTES);
  if (e != hipSuccess) { fprintf(stderr, "HIP error %s\n", hipGetErrorString(e)); return; }
  Args a;
  a.x = (const signed char*)d_in[0]; a.w = (const signed char*)d_in[1];
  a.sx = (const float*)d_in[2]; a.sw = (const float*)d_in[3];
  a.out = (float*)d_out;
  for (int p = 0; p < NR; p++) a.ws[p] = (char*)d_ws_all[p];
  a.rank = rank; a.pad = 0; a.myws = (char*)d_ws_all[rank];
  a2a_gemm_kernel<<<256, 512, LDS_BYTES, stream>>>(a);
}
```
